```python
import numpy as np

kernel_name = "gpu_gemm_rs_m4096_k4096_n16384_e4m3e4m3_dqsilu"


ABSMAX_RTOL = 0.03

COMPUTE_DTYPE = "float32"
EPILOGUE = "dequant_silu"
IN_QUANT = None
OUT_QUANT = None
IN_FP8 = "e4m3e4m3"
IN_DTYPES = {"x": "e4m3", "w_mat": "e4m3",
             "scale_x": "float32", "scale_w": "float32"}
WORLD_SIZE = 8

def _snap_fp8(x, mant_bits, max_abs, min_normal):
    a = np.abs(x.astype(np.float64))
    _m, _e = np.frexp(a)
    q = np.where(a >= min_normal, np.ldexp(1.0, _e - 1 - mant_bits),
                 min_normal * 2.0 ** -mant_bits)
    snapped = np.minimum(np.round(a / q) * q, max_abs)
    return (np.sign(x) * snapped).astype(np.float32)

def setup_inputs():

    x = _snap_fp8(np.random.randn(4096, 4096).astype(np.float32), 3, 448.0, 2.0 ** -6)
    w_mat = _snap_fp8(np.random.randn(4096, 16384).astype(np.float32), 3, 448.0, 2.0 ** -6)
    scale_x = (0.02 * (1.0 + np.random.rand(1))).astype(np.float32)
    scale_w = (0.02 * (1.0 + np.random.rand(1))).astype(np.float32)
    return {"x": x, "w_mat": w_mat, "scale_x": scale_x, "scale_w": scale_w}

def reference(x, w_mat, scale_x, scale_w):
    acc = (x @ w_mat).astype(np.float32)
    y = (acc * (scale_x * scale_w)).astype(np.float32)
    return (y / (1.0 + np.exp(-np.clip(y, -60.0, 60.0)))).astype(np.float32)
```

```cpp
#include <hip/hip_runtime.h>
#include <rccl/rccl.h>
#include <cstdio>
#include <cstdint>

#define HIP_CHECK(expr) do { hipError_t _e = (expr); if (_e != hipSuccess) { \
    fprintf(stderr, "HIP error %s at %s:%d\n", hipGetErrorString(_e), __FILE__, __LINE__); \
    return; } } while (0)

namespace {

typedef unsigned u32x4 __attribute__((ext_vector_type(4)));
typedef int i32x8 __attribute__((ext_vector_type(8)));
typedef float f32x16 __attribute__((ext_vector_type(16)));
typedef float f32x4 __attribute__((ext_vector_type(4)));
#define LAS __attribute__((address_space(3)))
#define GAS __attribute__((address_space(1)))

constexpr int NR = 8;
constexpr int N_ = 16384, KS = 512;
#define NENG_ 64
#define EPW 2
constexpr int NWG = 256, NENG = NENG_, NGEMM = NWG - NENG;
#define EW 16
#define WIN 5
constexpr int SU_S = 7, SU_O = 9;
constexpr int NSU_REMOTE = 3 * SU_S + 4 * SU_O;
constexpr int NSU = NSU_REMOTE + 16;
constexpr int ITEM = 16384;
constexpr int NOUT_MAX = 4 * 8 * SU_S;

constexpr size_t OFF_CALL  = 0;
constexpr size_t OFF_TMO   = 16384;
constexpr size_t OFF_PRE   = 20480;
constexpr size_t OFF_WFLAG = 24576;
constexpr size_t OFF_XFLAG = 32768;
constexpr size_t OFF_OFLAG = 49152;
constexpr size_t OFF_DFLAG = 65536;
constexpr size_t OFF_DBG   = 262144;
constexpr size_t OFF_SB    = 1u << 20;
constexpr size_t SB_BYTES  = (size_t)KS * N_;
constexpr size_t XS_BYTES  = (size_t)NSU * NR * ITEM;
constexpr size_t WS_BYTES  = (size_t)32 * NR * 65536;
constexpr size_t OFF_XS    = OFF_SB + SB_BYTES;
constexpr size_t OFF_WS    = OFF_XS + 2 * XS_BYTES;
constexpr size_t OFF_OB    = OFF_WS + 2 * WS_BYTES;
constexpr size_t WS_TOTAL  = OFF_OB + (size_t)7 * NOUT_MAX * ITEM;
static_assert(OFF_XFLAG + NSU * 128 <= OFF_OFLAG, "flags overlap");
static_assert(OFF_DFLAG + 8 * EW * 128 <= OFF_DBG, "flags overlap");
static_assert(7 * EW <= NENG * EPW && NENG % 8 == 0 && EPW <= 8, "not enough engine waves");

constexpr int STAGE_BYTES = 49152, NSTAGE = 3, LDS_BYTES = NSTAGE * STAGE_BYTES;
constexpr unsigned long long DEADLINE = 50000000ull;

struct Params {
    const unsigned char* x; const unsigned char* w; const float* sx; const float* sw;
    char* ws[NR]; float* out[NR];
    int rank; int pad;
};

__device__ __forceinline__ unsigned ld_sys(const void* p) { return __hip_atomic_load((const GAS unsigned*)p, __ATOMIC_RELAXED, __HIP_MEMORY_SCOPE_SYSTEM); }
__device__ __forceinline__ void st_sys(void* p, unsigned v) { __hip_atomic_store((GAS unsigned*)p, v, __ATOMIC_RELAXED, __HIP_MEMORY_SCOPE_SYSTEM); }
__device__ __forceinline__ unsigned ld_agent(const void* p) { return __hip_atomic_load((const GAS unsigned*)p, __ATOMIC_RELAXED, __HIP_MEMORY_SCOPE_AGENT); }
__device__ __forceinline__ void st_agent(void* p, unsigned v) { __hip_atomic_store((GAS unsigned*)p, v, __ATOMIC_RELAXED, __HIP_MEMORY_SCOPE_AGENT); }
__device__ __forceinline__ void st16_wt(const void* base, unsigned off, u32x4 v) {
    asm volatile("global_store_dwordx4 %0, %1, %2 sc0 sc1\n\ts_nop 1" :: "v"(off), "v"(v), "s"(base) : "memory");
}
#define WAIT_VM_(n) asm volatile("s_waitcnt vmcnt(" #n ")" ::: "memory")
#define WAIT_VM(n) WAIT_VM_(n)

__device__ __forceinline__ bool give_up(char* myws, unsigned call, unsigned long long t0, unsigned& polls) {
    bool dead = __builtin_amdgcn_s_memrealtime() - t0 > DEADLINE;
    if ((++polls & 63u) == 0u) dead |= __builtin_amdgcn_readfirstlane(ld_agent(myws + OFF_TMO)) == call;
    if (dead) st_agent(myws + OFF_TMO, call);
    return dead;
}

__device__ __forceinline__ int col_block(int r, int m) { return m * 4 + ((r & 1) ? ((r >> 1) ^ m) : (r >> 1)); }
__device__ __forceinline__ int unit_row(int r, int su) {
    if (su < 3 * SU_S) return (r ^ (2 * (su / SU_S + 1))) * 512 + 32 * (su % SU_S);
    if (su < NSU_REMOTE) { const int v = su - 3 * SU_S; return (r ^ (2 * (v / SU_O) + 1)) * 512 + 32 * (SU_S + v % SU_O); }
    return r * 512 + 32 * (su - NSU_REMOTE);
}

#define STAMP(slot) do {} while (0)

__device__ __forceinline__ bool wait_pretile(char* myws, unsigned call, int lane, bool block) {
    const unsigned long long t0 = __builtin_amdgcn_s_memrealtime(); unsigned polls = 0;
    for (;;) {
        bool ok = true;
#pragma unroll
        for (int i = 0; i < 4; ++i) { const int g = lane + 64 * i; ok &= (g < NENG) | (ld_agent(myws + OFF_PRE + 4 * g) >= call); }
        if (__all(ok)) return true;
        if (!block) return false;
        if (give_up(myws, call, t0, polls)) return true;
        __builtin_amdgcn_s_sleep(4);
    }
}

struct Item { const char* src; const char* dst; char* flag; unsigned dstep; int kind; };

__device__ __forceinline__ void engine(const Params& p, int r, char* myws, unsigned call, int wave, int lane) {
    const int gw = blockIdx.x * EPW + wave;
    if (wave >= EPW || gw >= 7 * EW) return;
    const int link = gw % 7, k = gw / 7, delta = link + 1, d = (r ^ delta) & 7;
    char* dws = p.ws[delta]; char* dout = (char*)p.out[delta];
    const size_t par = call & 1;
    const bool same = (delta & 1) == 0;
    const int per_stage = same ? 8 * SU_S : 6 * SU_O, nout = 4 * per_stage;
    constexpr int J_W = NSU_REMOTE, J_XO = J_W + 128, J_OUT = J_XO + 16;
    const int nitems = J_OUT + nout;
    const unsigned lin = lane * 16;
    const unsigned ol = (lane >> 5) * (unsigned)(N_ * 4) + (lane & 31) * 16;
    bool pre_ok = false, dead = false;
    unsigned long long omask = 0;
    const int jo0 = J_OUT + ((k - J_OUT) % EW + EW) % EW;

    auto make = [&](int j, Item& it) {
        if (j < J_W || (j >= J_XO && j < J_OUT)) {
            const int su = j < J_W ? j : j - J_XO + NSU_REMOTE;
            it.kind = 0; it.src = (const char*)p.x + (size_t)unit_row(d, su) * KS;
            it.dst = dws + OFF_XS + par * XS_BYTES + ((size_t)su * NR + delta) * ITEM; it.dstep = 1024;
            it.flag = dws + OFF_XFLAG + su * 128 + delta * 4;
        } else if (j < J_XO) {
            const int q = j - J_W, t = q >> 5, kc = (q >> 3) & 3, m = (q >> 1) & 3, G = q & 1;
            it.kind = 1; it.src = myws + OFF_SB + (size_t)((col_block(d, m) * 4 + t) * 2 + G) * 65536 + kc * ITEM;
            const int tc = (m * 4 + t) * 2 + G;
            it.dst = dws + OFF_WS + par * WS_BYTES + ((size_t)tc * NR + delta) * 65536 + kc * ITEM; it.dstep = 1024;
            it.flag = dws + OFF_WFLAG + tc * 128 + (kc * 8 + delta) * 4;
        } else {
            const int n = j - J_OUT, t = n / per_stage, q = n % per_stage;
            int m, G, row;
            if (same) { const int c = q / SU_S; m = c >> 1; G = c & 1; row = 32 * (q % SU_S); }
            else { const int c = q / SU_O, mm = c >> 1; m = mm + (mm >= (delta >> 1)); G = c & 1; row = 32 * (SU_S + q % SU_O); }
            it.kind = 2; it.src = myws + OFF_OB + ((size_t)link * NOUT_MAX + n) * ITEM;
            it.dst = dout + ((size_t)row * N_ + col_block(r, m) * 1024 + t * 256 + G * 128) * 4; it.dstep = 2 * N_ * 4;
            it.flag = nullptr;
        }
    };
    auto ready = [&](int j, bool block) -> bool {
        if (j < J_W || dead) return true;
        if (j < J_OUT) {
            if (!pre_ok) pre_ok = wait_pretile(myws, call, lane, block);
            return pre_ok;
        }
        const int b = (j - jo0) / EW;
        if ((omask >> b) & 1) return true;
        const unsigned long long t0 = __builtin_amdgcn_s_memrealtime(); unsigned polls = 0;
        const int nn = jo0 - J_OUT + lane * EW;
        for (;;) {
            const unsigned f = ld_agent(myws + OFF_OFLAG + (link * 256 + (nn < nout ? nn : 0)) * 4);
            omask = __ballot(nn < nout && f >= call);
            if ((omask >> b) & 1) return true;
            if (!block) return false;
            if (give_up(myws, call, t0, polls)) { dead = true; return true; }
            __builtin_amdgcn_s_sleep(8);
        }
    };
    auto load = [&](const Item& it, u32x4 (&buf)[16]) {
#pragma unroll
        for (int q = 0; q < 16; ++q)
            asm volatile("global_load_dwordx4 %0, %1, %2 offset:%3 sc0 sc1" : "=&v"(buf[q]) : "v"(lin + (q >> 2) * 4096), "s"(it.src), "i"((q & 3) * 1024) : "memory");
    };
    auto send = [&](const Item& it, const u32x4 (&buf)[16], char* prevflag) {
        unsigned vo = it.kind == 2 ? ol : lin;
#pragma unroll
        for (int q = 0; q < 16; ++q) {
            st16_wt(it.dst, vo, buf[q]);
            vo += it.dstep;
            if (q < WIN) asm volatile("s_waitcnt vmcnt(%0)" :: "i"(16 + WIN) : "memory");
            else asm volatile("s_waitcnt vmcnt(%0)" :: "i"(WIN) : "memory");
            if (q == WIN - 1 && prevflag != nullptr && lane == 0) st_sys(prevflag, call);
        }
    };

#define PIN16(b) asm volatile("s_waitcnt vmcnt(0)" : "+v"(b[0]), "+v"(b[1]), "+v"(b[2]), "+v"(b[3]), "+v"(b[4]), "+v"(b[5]), "+v"(b[6]), "+v"(b[7]), \
                 "+v"(b[8]), "+v"(b[9]), "+v"(b[10]), "+v"(b[11]), "+v"(b[12]), "+v"(b[13]), "+v"(b[14]), "+v"(b[15]) :: "memory")
    char* prevflag = nullptr;
    int j = k;
    auto step = [&](const Item& cur, const u32x4 (&cbuf)[16], Item& nxt, u32x4 (&nbuf)[16]) -> bool {
        const bool has = j + EW < nitems;
        const bool now = has && ready(j + EW, false);
        make(now ? j + EW : j, nxt); load(nxt, nbuf);
        send(cur, cbuf, prevflag); prevflag = cur.flag;
        if (!has) return false;
        j += EW;
        if (!now) {
            WAIT_VM(0);
            if (prevflag != nullptr && lane == 0) st_sys(prevflag, call);
            prevflag = nullptr;
            ready(j, true);
            make(j, nxt); load(nxt, nbuf); PIN16(nbuf);
        }
        return true;
    };
    u32x4 bufA[16], bufB[16];
    Item a, b;
    STAMP(0);
    make(j, a); load(a, bufA); PIN16(bufA);
    for (;;) {
        if (!step(a, bufA, b, bufB)) break;
        if (!step(b, bufB, a, bufA)) break;
    }
    WAIT_VM(0);
    STAMP(4);
    if (!dead && lane == 0) st_sys(dws + OFF_DFLAG + (delta * EW + k) * 128, call);
    {
        const unsigned long long t0 = __builtin_amdgcn_s_memrealtime(); unsigned polls = 0;
        while (__builtin_amdgcn_readfirstlane(ld_sys(myws + OFF_DFLAG + (delta * EW + k) * 128)) < call) {
            if (give_up(myws, call, t0, polls)) { dead = true; break; }
            __builtin_amdgcn_s_sleep(2);
        }
    }
    if (dead && lane == 0) p.out[0][(size_t)col_block(r, 0) * 1024] = __builtin_nanf("");
    STAMP(5);
}

__device__ __forceinline__ void pretile(const Params& p, int r, char* myws, unsigned call, int wave, int lane) {
    const size_t par = call & 1;
    const int gw = (blockIdx.x - NENG) * 8 + wave;
    for (int u = gw; u < 64 * 32; u += NGEMM * 8) {
        const int strip = u & 63, kg = u >> 6;
        const unsigned* src = (const unsigned*)(p.w + (size_t)kg * 16 * N_ + strip * 256) + lane;
        unsigned v[16];
#pragma unroll
        for (int j = 0; j < 16; ++j) v[j] = __builtin_nontemporal_load(src + (size_t)j * (N_ / 4));
        u32x4 o[4];
#pragma unroll
        for (int q = 0; q < 4; ++q) {
            const unsigned t0 = __builtin_amdgcn_perm(v[4 * q + 1], v[4 * q], 0x05010400u), t1 = __builtin_amdgcn_perm(v[4 * q + 1], v[4 * q], 0x07030602u);
            const unsigned t2 = __builtin_amdgcn_perm(v[4 * q + 3], v[4 * q + 2], 0x05010400u), t3 = __builtin_amdgcn_perm(v[4 * q + 3], v[4 * q + 2], 0x07030602u);
            o[0][q] = __builtin_amdgcn_perm(t2, t0, 0x05040100u); o[1][q] = __builtin_amdgcn_perm(t2, t0, 0x07060302u);
            o[2][q] = __builtin_amdgcn_perm(t3, t1, 0x05040100u); o[3][q] = __builtin_amdgcn_perm(t3, t1, 0x07060302u);
        }
        const int cb = strip >> 2, t = strip & 3, m = cb >> 2;
        const char* dst = myws + OFF_SB + (size_t)strip * 2 * 65536 + kg * 2048;
        const unsigned lo = (lane >> 5) * 65536u + (lane & 31) * 16;
#pragma unroll
        for (int c = 0; c < 4; ++c) st16_wt(dst, lo + c * 512, o[c]);
        if (cb == col_block(r, m)) {
            const char* dst2 = myws + OFF_WS + par * WS_BYTES + (size_t)((m * 4 + t) * 2) * NR * 65536 + kg * 2048;
            const unsigned lo2 = (lane >> 5) * (unsigned)(NR * 65536) + (lane & 31) * 16;
#pragma unroll
            for (int c = 0; c < 4; ++c) st16_wt(dst2, lo2 + c * 512, o[c]);
        }
    }
    WAIT_VM(0);
    __syncthreads();
    if (threadIdx.x == 0) st_agent(myws + OFF_PRE + 4 * blockIdx.x, call);
}

__device__ __forceinline__ float dq_silu(float a, float s) {
    const float y = a * s;
    const float c = __builtin_amdgcn_fmed3f(y, -60.0f, 60.0f);
    return y * __builtin_amdgcn_rcpf(1.0f + __builtin_amdgcn_exp2f(c * -1.44269504088896341f));
}

__device__ __forceinline__ void gemm_tile(const Params& p, int r, char* myws, unsigned call, LAS char* lds, int wave, int lane,
                                          int t, int m, int G, int rb, float scale) {
    const size_t par = call & 1;
    int su, link = -1, nitem = 0;
    if (rb < 6) {
        const int v = rb * 8 + wave;
        if (v < 3 * SU_S) { su = v; link = 2 * (v / SU_S + 1) - 1; nitem = t * (8 * SU_S) + (m * 2 + G) * SU_S + v % SU_S; }
        else { const int w2 = v - 3 * SU_S, j = w2 / SU_O, jj = j + (j >= m); su = 3 * SU_S + jj * SU_O + w2 % SU_O;
               link = 2 * jj; nitem = t * (6 * SU_O) + ((m - (m > jj)) * 2 + G) * SU_O + w2 % SU_O; }
    } else su = NSU_REMOTE + (rb - 6) * 8 + wave;
    const int tc = (m * 4 + t) * 2 + G;
    const unsigned alane = (lane >> 3) * 512 + (((lane & 7) ^ (lane >> 4)) * 16);
    const char* astage = myws + OFF_XS + par * XS_BYTES + (size_t)su * NR * ITEM;
    const char* aown = (const char*)p.x + (size_t)unit_row(r, su) * KS;
    const char* bbase = myws + OFF_WS + par * WS_BYTES + (size_t)tc * NR * 65536 + wave * 2048 + lane * 16;
    const char* flagp = lane < 32 ? myws + OFF_WFLAG + tc * 128 + lane * 4 : myws + OFF_XFLAG + su * 128 + (lane & 7) * 4;
    unsigned ready = 0x01010101u;
    bool dead = false;

    auto wait_step = [&](int n, bool lazy) {
        if (!((ready >> n) & 1) && !dead) {
            const unsigned long long t0 = __builtin_amdgcn_s_memrealtime(); unsigned polls = 0;
            for (;;) {
                const unsigned long long mk = __ballot(ld_sys(flagp) >= call);
                ready = 0x01010101u | ((unsigned)mk & (((unsigned)(mk >> 32) & 0xffu) * 0x01010101u));
                if ((ready >> n) & 1) break;
                if (give_up(myws, call, t0, polls)) { dead = true; break; }
                if (lazy) __builtin_amdgcn_s_sleep(64); else __builtin_amdgcn_s_sleep(16);
            }
        }
    };
    auto stage_in = [&](int n, int buf) {
        wait_step(n, false);
        const int delta = n & 7, kc = n >> 3;
        const char* ga = (delta == 0 ? aown : astage + delta * ITEM) + kc * 128;
        const char* gb = bbase + delta * 65536 + kc * ITEM;
        LAS char* la = lds + buf * STAGE_BYTES + wave * 4096;
        LAS char* lb = lds + buf * STAGE_BYTES + 32768 + wave * 2048;
#pragma unroll
        for (int q = 0; q < 4; ++q) __builtin_amdgcn_global_load_lds((const GAS unsigned*)(ga + q * 4096 + (alane ^ ((q & 1) * 64))), (LAS unsigned*)(la + q * 1024), 16, 0, 17);
#pragma unroll
        for (int q = 0; q < 2; ++q) __builtin_amdgcn_global_load_lds((const GAS unsigned*)(gb + q * 1024), (LAS unsigned*)(lb + q * 1024), 16, 0, 17);
    };

    f32x16 acc[4];
#pragma unroll
    for (int c = 0; c < 4; ++c)
#pragma unroll
        for (int e = 0; e < 16; ++e) acc[c][e] = 0.f;

    const int aoff = wave * 4096 + (lane & 31) * 128 + (((lane >> 5) ^ ((lane >> 1) & 7)) * 16);
    const int boff = 32768 + (lane >> 5) * 2048 + (lane & 31) * 16;
    auto compute = [&](int buf) {
        const LAS char* s = lds + buf * STAGE_BYTES;
#pragma unroll
        for (int kk = 0; kk < 2; ++kk) {
            const u32x4 a0 = *(const LAS u32x4*)(s + (aoff ^ (kk * 64))), a1 = *(const LAS u32x4*)(s + (aoff ^ (kk * 64 + 32)));
            const i32x8 af = {(int)a0[0], (int)a0[1], (int)a0[2], (int)a0[3], (int)a1[0], (int)a1[1], (int)a1[2], (int)a1[3]};
#pragma unroll
            for (int c = 0; c < 4; ++c) {
                const u32x4 b0 = *(const LAS u32x4*)(s + boff + kk * 8192 + c * 512), b1 = *(const LAS u32x4*)(s + boff + kk * 8192 + 4096 + c * 512);
                const i32x8 bf = {(int)b0[0], (int)b0[1], (int)b0[2], (int)b0[3], (int)b1[0], (int)b1[1], (int)b1[2], (int)b1[3]};
                acc[c] = __builtin_amdgcn_mfma_scale_f32_32x32x64_f8f6f4(af, bf, acc[c], 0, 0, 0, 0, 0, 0);
            }
        }
    };

    if (wave == 0) wait_step(1, true);
    __builtin_amdgcn_s_barrier();
    stage_in(0, 0);
    stage_in(1, 1);
    STAMP(2);
    int buf = 0;
    for (int n = 0; n < 31; ++n) {
        WAIT_VM(6);
        __builtin_amdgcn_s_barrier();
        asm volatile("" ::: "memory");
        const int nb = buf == 0 ? 2 : buf - 1;
        if (n + 2 < 32) stage_in(n + 2, nb);
        compute(buf);
        buf = buf == 2 ? 0 : buf + 1;
    }
    WAIT_VM(0);
    __builtin_amdgcn_s_barrier();
    asm volatile("" ::: "memory");
    compute(buf);
    STAMP(3);

    const bool bad = __builtin_amdgcn_readfirstlane(ld_agent(myws + OFF_TMO)) == call;
    const float nan = __builtin_nanf("");
    const int g = lane >> 5, i = lane & 31;
    if (link < 0) {
        float* o = p.out[0] + (size_t)(32 * (su - NSU_REMOTE) + 4 * g) * N_ + col_block(r, m) * 1024 + t * 256 + G * 128 + 4 * i;
#pragma unroll
        for (int e = 0; e < 16; ++e) {
            f32x4 v = {dq_silu(acc[0][e], scale), dq_silu(acc[1][e], scale), dq_silu(acc[2][e], scale), dq_silu(acc[3][e], scale)};
            if (bad) v = (f32x4){nan, nan, nan, nan};
            *(f32x4*)(o + (size_t)((e & 3) + 8 * (e >> 2)) * N_) = v;
        }
    } else {
        const char* ob = myws + OFF_OB + ((size_t)link * NOUT_MAX + nitem) * ITEM;
#pragma unroll
        for (int e = 0; e < 16; ++e) {
            f32x4 v = {dq_silu(acc[0][e], scale), dq_silu(acc[1][e], scale), dq_silu(acc[2][e], scale), dq_silu(acc[3][e], scale)};
            if (bad) v = (f32x4){nan, nan, nan, nan};
            st16_wt(ob, ((e & 3) + 8 * (e >> 2) + 4 * g) * 512 + i * 16, __builtin_bit_cast(u32x4, v));
        }
        WAIT_VM(0);
        if (lane == 0) st_agent(myws + OFF_OFLAG + (link * 256 + nitem) * 4, call);
    }
    STAMP(4);
}

__global__ void __launch_bounds__(512, 2) fused_gemm_rs(Params p) {
    extern __shared__ __attribute__((aligned(16))) char smem[];
    const int lane = threadIdx.x & 63;
    const int wave = __builtin_amdgcn_readfirstlane(threadIdx.x >> 6);
    const int r = __builtin_amdgcn_readfirstlane(p.rank) & 7;
    char* myws = p.ws[0];
    char* cp = myws + OFF_CALL + (blockIdx.x * 8 + wave) * 4;
    const unsigned call = __builtin_amdgcn_readfirstlane(ld_agent(cp)) + 1;
    if (lane == 0) st_agent(cp, call);

    if (blockIdx.x < NENG) { engine(p, r, myws, call, wave, lane); return; }

    STAMP(0);
    pretile(p, r, myws, call, wave, lane);
    float scale = p.sx[0] * p.sw[0];
    asm volatile("" : "+v"(scale));
    if (wave == 0) wait_pretile(myws, call, lane, true);
    STAMP(1);
    const int e = blockIdx.x & 7, slot = (blockIdx.x - NENG) >> 3, m = e >> 1;
    for (int id = slot; id < 32; id += NGEMM / 8) {
        int t, G, rb;
        if (id < 24) { t = id / 6; const int q = id % 6; G = q / 3; rb = 3 * (e & 1) + q % 3; }
        else { const int q = id - 24; t = q >> 1; G = q & 1; rb = 6 + (e & 1); }
        gemm_tile(p, r, myws, call, (LAS char*)smem, wave, lane, t, m, G, rb, scale);
    }
}

}

extern "C" void kernel_launch(int rank, int nranks, ncclComm_t comm,
                              hipStream_t stream,
                              void* const* d_in, const int* in_sizes, int n_in,
                              void* d_out, int out_size,
                              const int* in_dtypes, int out_dtype,
                              void* const* d_out_all,
                              void* const* d_ws_all, size_t ws_bytes) {
    (void)comm; (void)in_dtypes; (void)out_dtype; (void)d_out;
    if (nranks != NR || (unsigned)rank >= (unsigned)NR || n_in < 4 || ws_bytes < WS_TOTAL || out_size != 512 * N_ ||
        in_sizes[0] != 4096 * KS || in_sizes[1] != KS * N_) {
        fprintf(stderr, "kernel_launch: unexpected problem geometry\n");
        return;
    }
    static bool attr_set[NR] = {};
    if (!attr_set[rank]) {
        HIP_CHECK(hipFuncSetAttribute((const void*)fused_gemm_rs, hipFuncAttributeMaxDynamicSharedMemorySize, LDS_BYTES));
        attr_set[rank] = true;
    }
    Params p;
    p.x = (const unsigned char*)d_in[0]; p.w = (const unsigned char*)d_in[1];
    p.sx = (const float*)d_in[2]; p.sw = (const float*)d_in[3];
    for (int i = 0; i < NR; ++i) { p.ws[i] = (char*)d_ws_all[rank ^ i]; p.out[i] = (float*)d_out_all[rank ^ i]; }
    p.rank = rank; p.pad = 0;
    hipLaunchKernelGGL(fused_gemm_rs, dim3(NWG), dim3(512), LDS_BYTES, stream, p);
    HIP_CHECK(hipGetLastError());
}
```

```cpp
#include <hip/hip_runtime.h>
#include <rccl/rccl.h>
#include <cstdio>
#include <cstdint>

#define HIP_CHECK(expr) do { hipError_t _e = (expr); if (_e != hipSuccess) { \
    fprintf(stderr, "HIP error %s at %s:%d\n", hipGetErrorString(_e), __FILE__, __LINE__); \
    return; } } while (0)

namespace {

typedef unsigned u32x4 __attribute__((ext_vector_type(4)));
typedef int i32x8 __attribute__((ext_vector_type(8)));
typedef float f32x16 __attribute__((ext_vector_type(16)));
typedef float f32x4 __attribute__((ext_vector_type(4)));
#define LAS __attribute__((address_space(3)))
#define GAS __attribute__((address_space(1)))

constexpr int NR = 8;
constexpr int N_ = 16384, KS = 512;
#define NENG_ 64
#define EPW 2
constexpr int NWG = 256, NENG = NENG_, NGEMM = NWG - NENG;
#define EW 16
#define WIN 4
constexpr int SU_S = 7, SU_O = 9;
constexpr int NSU_REMOTE = 3 * SU_S + 4 * SU_O;
constexpr int NSU = NSU_REMOTE + 16;
constexpr int ITEM = 16384;
constexpr int NOUT_MAX = 4 * 8 * SU_S;

#define GAUX 17
#define NRAW 12
#define UPW 4
constexpr int PK_CHUNKS = 13;

constexpr size_t OFF_CALL  = 0;
constexpr size_t OFF_TMO   = 16384;
constexpr size_t OFF_CNT   = 16384 + 128;
constexpr size_t OFF_PRE   = 20480;
constexpr size_t OFF_WFLAG = 24576;
constexpr size_t OFF_XFLAG = 32768;
constexpr size_t OFF_OQ    = 49152;
constexpr size_t OFF_RQ    = 65536;
constexpr size_t OFF_DBG   = 262144;
constexpr size_t OFF_SB    = 1u << 20;
constexpr size_t SB_BYTES  = (size_t)KS * N_;
constexpr size_t XS_BYTES  = (size_t)NR * NSU * ITEM;
constexpr size_t WS_BYTES  = (size_t)NR * 32 * 65536;
constexpr size_t OFF_XS    = OFF_SB + SB_BYTES;
constexpr size_t OFF_WS    = OFF_XS + 2 * XS_BYTES;
constexpr size_t OFF_OB    = OFF_WS + 2 * WS_BYTES;
constexpr size_t OFF_RB    = OFF_OB + (size_t)7 * NOUT_MAX * ITEM;
constexpr size_t WS_TOTAL  = OFF_RB + (size_t)NR * NOUT_MAX * ITEM;
static_assert(OFF_CNT + 17 * 128 <= OFF_PRE && OFF_XFLAG + NSU * 128 <= OFF_OQ && OFF_OQ + 7 * 2048 <= OFF_RQ && OFF_RQ + 8 * 2048 <= OFF_DBG, "flags overlap");
static_assert(7 * EW <= NENG * EPW && NENG % 8 == 0 && EPW <= 8, "not enough engine waves");
constexpr unsigned D_RAW = 1u << 11;

constexpr int STAGE_BYTES = 49152, NSTAGE = 3, LDS_BYTES = NSTAGE * STAGE_BYTES;
constexpr unsigned long long DEADLINE = 50000000ull;

struct Params {
    const unsigned char* x; const unsigned char* w; const float* sx; const float* sw;
    char* ws[NR]; float* out[NR];
    int rank; int pad;
};

__device__ __forceinline__ unsigned ld_sys(const void* p) { return __hip_atomic_load((const GAS unsigned*)p, __ATOMIC_RELAXED, __HIP_MEMORY_SCOPE_SYSTEM); }
__device__ __forceinline__ void st_sys(void* p, unsigned v) { __hip_atomic_store((GAS unsigned*)p, v, __ATOMIC_RELAXED, __HIP_MEMORY_SCOPE_SYSTEM); }
__device__ __forceinline__ unsigned ld_agent(const void* p) { return __hip_atomic_load((const GAS unsigned*)p, __ATOMIC_RELAXED, __HIP_MEMORY_SCOPE_AGENT); }
__device__ __forceinline__ void st_agent(void* p, unsigned v) { __hip_atomic_store((GAS unsigned*)p, v, __ATOMIC_RELAXED, __HIP_MEMORY_SCOPE_AGENT); }
__device__ __forceinline__ unsigned long long ld8_sys(const void* p) { return __hip_atomic_load((const GAS unsigned long long*)p, __ATOMIC_RELAXED, __HIP_MEMORY_SCOPE_SYSTEM); }
__device__ __forceinline__ void st8_sys(void* p, unsigned long long v) { __hip_atomic_store((GAS unsigned long long*)p, v, __ATOMIC_RELAXED, __HIP_MEMORY_SCOPE_SYSTEM); }
__device__ __forceinline__ unsigned ticket(void* p, int lane) {
    unsigned v = 0;
    if (lane == 0) v = __hip_atomic_fetch_add((GAS unsigned*)p, 1u, __ATOMIC_RELAXED, __HIP_MEMORY_SCOPE_AGENT);
    return __builtin_amdgcn_readfirstlane(v);
}
__device__ __forceinline__ void st16_wt(const void* base, unsigned off, u32x4 v) {
    asm volatile("global_store_dwordx4 %0, %1, %2 sc0 sc1\n\ts_nop 1" :: "v"(off), "v"(v), "s"(base) : "memory");
}
#define WAIT_VM_(n) asm volatile("s_waitcnt vmcnt(" #n ")" ::: "memory")
#define WAIT_VM(n) WAIT_VM_(n)

__device__ __forceinline__ bool give_up(char* myws, unsigned call, unsigned long long t0, unsigned& polls) {
    bool dead = __builtin_amdgcn_s_memrealtime() - t0 > DEADLINE;
    if ((++polls & 63u) == 0u) dead |= __builtin_amdgcn_readfirstlane(ld_agent(myws + OFF_TMO)) == call;
    if (dead) st_agent(myws + OFF_TMO, call);
    return dead;
}

__device__ __forceinline__ int col_block(int r, int m) { return m * 4 + ((r & 1) ? ((r >> 1) ^ m) : (r >> 1)); }
__device__ __forceinline__ int unit_row(int r, int su) {
    if (su < 3 * SU_S) return (r ^ (2 * (su / SU_S + 1))) * 512 + 32 * (su % SU_S);
    if (su < NSU_REMOTE) { const int v = su - 3 * SU_S; return (r ^ (2 * (v / SU_O) + 1)) * 512 + 32 * (SU_S + v % SU_O); }
    return r * 512 + 32 * (su - NSU_REMOTE);
}

#define STAMP(slot) do {} while (0)

__device__ __forceinline__ bool wait_pretile(char* myws, unsigned call, int lane, bool block, bool& dead) {
    const unsigned long long t0 = __builtin_amdgcn_s_memrealtime(); unsigned polls = 0;
    for (;;) {
        bool ok = true;
#pragma unroll
        for (int i = 0; i < 4; ++i) { const int g = lane + 64 * i; ok &= (g < NENG) | (ld_agent(myws + OFF_PRE + 4 * g) >= call); }
        if (__all(ok)) return true;
        if (!block) return false;
        if (give_up(myws, call, t0, polls)) { dead = true; return true; }
        __builtin_amdgcn_s_sleep(4);
    }
}

struct Item { const char* src; const char* dst; char* flag; unsigned desc; int kind; };

__device__ __forceinline__ void engine(const Params& p, int r, char* myws, unsigned callv, int wave, int lane) {
    const int gw = blockIdx.x * EPW + wave;
    if (wave >= EPW || gw >= 7 * EW) return;
    const int link = gw % 7, k = gw / 7, delta = link + 1, d = (r ^ delta) & 7;
    char* dws = p.ws[delta]; char* dout = (char*)p.out[delta];
    const int nout = (delta & 1) ? 24 * SU_O : 32 * SU_S;
    constexpr int J_W = NSU_REMOTE, J_XO = J_W + 128, J_OUT = J_XO + 16;
    const int nitems = J_OUT + nout;
    const unsigned lin = lane * 16;
    const unsigned ol = (lane >> 5) * (unsigned)(N_ * 4) + (lane & 31) * 16;
    const char* oq = myws + OFF_OQ + link * 2048;
    const char* claimp = myws + OFF_CNT + link * 128;

    auto load = [&](const char* src, u32x4 (&buf)[16]) {
#pragma unroll
        for (int q = 0; q < 16; ++q)
            asm volatile("global_load_dwordx4 %0, %1, %2 offset:%3 sc0 sc1" : "=&v"(buf[q]) : "v"(lin + (q >> 2) * 4096), "s"(src), "i"((q & 3) * 1024) : "memory");
    };
#define PIN16(b) asm volatile("s_waitcnt vmcnt(0)" : "+v"(b[0]), "+v"(b[1]), "+v"(b[2]), "+v"(b[3]), "+v"(b[4]), "+v"(b[5]), "+v"(b[6]), "+v"(b[7]), \
                 "+v"(b[8]), "+v"(b[9]), "+v"(b[10]), "+v"(b[11]), "+v"(b[12]), "+v"(b[13]), "+v"(b[14]), "+v"(b[15]) :: "memory")
    u32x4 bufA[16], bufB[16];
    load((const char*)p.x + (size_t)unit_row(d, k) * KS, bufA);
    PIN16(bufA);
    asm volatile("" : "+v"(callv));
    const unsigned call = __builtin_amdgcn_readfirstlane(callv) + 1;
    const size_t par = call & 1;
    const unsigned cbase = (call - 1) * (unsigned)(nitems - EW);
    bool pre_ok = false, dead = false;
    unsigned wlo = 0, whi = 0; int wb = 0;
    unsigned desc = 0;

    auto make = [&](int j, Item& it) {
        it.desc = 0;
        if (j < J_W || (j >= J_XO && j < J_OUT)) {
            const int su = j < J_W ? j : j - J_XO + NSU_REMOTE;
            it.kind = 0; it.src = (const char*)p.x + (size_t)unit_row(d, su) * KS;
            it.dst = dws + OFF_XS + par * XS_BYTES + ((size_t)delta * NSU + su) * ITEM;
            it.flag = dws + OFF_XFLAG + su * 128 + delta * 4;
        } else if (j < J_XO) {
            const int q = j - J_W, t = q >> 5, kc = (q >> 3) & 3, m = (q >> 1) & 3, G = q & 1;
            it.kind = 1; it.src = myws + OFF_SB + (size_t)((col_block(d, m) * 4 + t) * 2 + G) * 65536 + kc * ITEM;
            const int tc = (m * 4 + t) * 2 + G;
            it.dst = dws + OFF_WS + par * WS_BYTES + ((size_t)delta * 32 + tc) * 65536 + kc * ITEM;
            it.flag = dws + OFF_WFLAG + tc * 128 + (kc * 8 + delta) * 4;
        } else {
            const int i = j - J_OUT;
            it.desc = desc; it.src = myws + OFF_OB + ((size_t)link * NOUT_MAX + i) * ITEM;
            if (desc & D_RAW) { it.kind = 3; it.dst = dout + ((size_t)(desc & 15) * 32 * N_ + ((desc >> 4) & 127) * 128) * 4; }
            else { it.kind = 2; it.dst = dws + OFF_RB + ((size_t)delta * NOUT_MAX + i) * ITEM; }
            it.flag = dws + OFF_RQ + (delta * 256 + i) * 8;
        }
        if (dead) it.flag = nullptr;
    };
    auto ready = [&](int j, bool block) -> bool {
        if (j < J_W || dead) return true;
        if (j < J_OUT) {
            if (!pre_ok) pre_ok = wait_pretile(myws, call, lane, block, dead);
            return pre_ok;
        }
        const int i = j - J_OUT;
        if ((unsigned)(i - wb) < 64u && (unsigned)__builtin_amdgcn_readlane(whi, i - wb) == call) { desc = __builtin_amdgcn_readlane(wlo, i - wb); return true; }
        const unsigned long long t0 = __builtin_amdgcn_s_memrealtime(); unsigned polls = 0;
        for (;;) {
            const unsigned long long e = ld8_sys(oq + i * 8);
            if ((unsigned)__builtin_amdgcn_readfirstlane((unsigned)(e >> 32)) == call) { desc = __builtin_amdgcn_readfirstlane((unsigned)e); return true; }
            if (!block) return false;
            if (give_up(myws, call, t0, polls)) { dead = true; return true; }
            __builtin_amdgcn_s_sleep(4);
        }
    };
    auto publish = [&](char* flag, unsigned dsc, bool wide) {
        if (flag != nullptr && lane == 0) { if (wide) st8_sys(flag, ((unsigned long long)call << 32) | dsc); else st_sys(flag, call); }
    };
    char* pflag = nullptr; unsigned pdesc = 0; bool pwide = false;
    auto send = [&](const Item& it, const u32x4 (&buf)[16], bool open) {
        unsigned vo = it.kind == 3 ? ol : lin;
        const unsigned dstep = it.kind == 3 ? 2u * N_ * 4u : 1024u;
#pragma unroll
        for (int q = 0; q < 16; ++q) {
            if (q == PK_CHUNKS && it.kind == 2) break;
            st16_wt(it.dst, vo, buf[q]);
            vo += dstep;
            if (q < WIN) asm volatile("s_waitcnt vmcnt(%0)" :: "i"(18 + WIN) : "memory");
            else if (q == WIN || !open) asm volatile("s_waitcnt vmcnt(%0)" :: "i"(WIN) : "memory");
            if (q == WIN - 1) publish(pflag, pdesc, pwide);
        }
        pflag = it.flag; pdesc = it.desc; pwide = it.kind >= 2;
    };
    int jn = k + EW;
    auto step = [&](const Item& cur, const u32x4 (&cbuf)[16], Item& nxt, u32x4 (&nbuf)[16]) -> bool {
        const bool has = jn < nitems;
        const bool now = has && ready(jn, false);
        if (now) make(jn, nxt); else nxt = cur;
        unsigned cl; unsigned long long keep, win;
        wb = jn > J_OUT ? jn - J_OUT : 0;
        asm volatile("s_mov_b64 %1, exec\n\ts_mov_b64 exec, 1\n\tglobal_atomic_add %0, %2, %3, %4 sc0\n\ts_mov_b64 exec, %1"
                     : "=&v"(cl), "=&s"(keep) : "v"(0u), "v"(has ? 1u : 0u), "s"(claimp) : "memory");
        asm volatile("global_load_dwordx2 %0, %1, %2 sc0 sc1" : "=&v"(win) : "v"((unsigned)((wb + lane) & 255) * 8u), "s"(oq) : "memory");
        load(nxt.src, nbuf);
        send(cur, cbuf, jn + EW >= nitems);
        asm volatile("" : "+v"(cl), "+v"(win) :: "memory");
        if (!has) return false;
        wlo = (unsigned)win; whi = (unsigned)(win >> 32);
        if (!now) {
            WAIT_VM(0);
            publish(pflag, pdesc, pwide); pflag = nullptr;
            ready(jn, true);
            make(jn, nxt); load(nxt.src, nbuf); PIN16(nbuf);
        }
        const unsigned c = __builtin_amdgcn_readfirstlane(cl) - cbase;
        jn = c < (unsigned)nitems ? 2 * EW + (int)c : nitems;
        return true;
    };
    Item a, b;
    STAMP(0);
    make(k, a);
    for (;;) {
        if (!step(a, bufA, b, bufB)) break;
        if (!step(b, bufB, a, bufA)) break;
    }
    WAIT_VM(0);
    publish(pflag, pdesc, pwide);
    if (dead && lane == 0) p.out[0][(size_t)col_block(r, 0) * 1024] = __builtin_nanf("");
    STAMP(4);
}

__device__ __forceinline__ void pretile(const Params& p, int r, char* myws, unsigned call, int wave, int lane) {
    const size_t par = call & 1;
    const int gw = (blockIdx.x - NENG) * 8 + wave;
    for (int u = gw; u < 64 * 32; u += NGEMM * 8) {
        const int strip = u & 63, kg = u >> 6;
        const unsigned* src = (const unsigned*)(p.w + (size_t)kg * 16 * N_ + strip * 256) + lane;
        unsigned v[16];
#pragma unroll
        for (int j = 0; j < 16; ++j) v[j] = __builtin_nontemporal_load(src + (size_t)j * (N_ / 4));
        u32x4 o[4];
#pragma unroll
        for (int q = 0; q < 4; ++q) {
            const unsigned t0 = __builtin_amdgcn_perm(v[4 * q + 1], v[4 * q], 0x05010400u), t1 = __builtin_amdgcn_perm(v[4 * q + 1], v[4 * q], 0x07030602u);
            const unsigned t2 = __builtin_amdgcn_perm(v[4 * q + 3], v[4 * q + 2], 0x05010400u), t3 = __builtin_amdgcn_perm(v[4 * q + 3], v[4 * q + 2], 0x07030602u);
            o[0][q] = __builtin_amdgcn_perm(t2, t0, 0x05040100u); o[1][q] = __builtin_amdgcn_perm(t2, t0, 0x07060302u);
            o[2][q] = __builtin_amdgcn_perm(t3, t1, 0x05040100u); o[3][q] = __builtin_amdgcn_perm(t3, t1, 0x07060302u);
        }
        const int cb = strip >> 2, t = strip & 3, m = cb >> 2;
        const char* dst = myws + OFF_SB + (size_t)strip * 2 * 65536 + kg * 2048;
        const unsigned lo = (lane >> 5) * 65536u + (lane & 31) * 16;
#pragma unroll
        for (int c = 0; c < 4; ++c) st16_wt(dst, lo + c * 512, o[c]);
        if (cb == col_block(r, m)) {
            const char* dst2 = myws + OFF_WS + par * WS_BYTES + (size_t)((m * 4 + t) * 2) * 65536 + kg * 2048;
#pragma unroll
            for (int c = 0; c < 4; ++c) st16_wt(dst2, lo + c * 512, o[c]);
        }
    }
    WAIT_VM(0);
    __syncthreads();
    if (threadIdx.x == 0) st_agent(myws + OFF_PRE + 4 * blockIdx.x, call);
}

__device__ __forceinline__ float dq_silu(float a, float s) {
    const float y = a * s;
    const float c = __builtin_amdgcn_fmed3f(y, -60.0f, 60.0f);
    return y * __builtin_amdgcn_rcpf(1.0f + __builtin_amdgcn_exp2f(c * -1.44269504088896341f));
}

__device__ __forceinline__ void gemm_tile(const Params& p, int r, char* myws, unsigned call, LAS char* lds, int wave, int lane,
                                          int t, int m, int G, int rb, float scale) {
    const size_t par = call & 1;
    int su;
    if (rb < 6) {
        const int v = rb * 8 + wave;
        if (v < 3 * SU_S) su = v;
        else { const int w2 = v - 3 * SU_S, j = w2 / SU_O; su = 3 * SU_S + (j + (j >= m)) * SU_O + w2 % SU_O; }
    } else su = NSU_REMOTE + (rb - 6) * 8 + wave;
    const int grow = unit_row(r, su);
    const int tc = (m * 4 + t) * 2 + G;
    const unsigned alane = (lane >> 3) * 512 + (((lane & 7) ^ (lane >> 4)) * 16);
    const char* astage = myws + OFF_XS + par * XS_BYTES + (size_t)su * ITEM;
    const char* aown = (const char*)p.x + (size_t)grow * KS;
    const char* bbase = myws + OFF_WS + par * WS_BYTES + (size_t)tc * 65536 + wave * 2048 + lane * 16;
    const char* flagp = lane < 32 ? myws + OFF_WFLAG + tc * 128 + lane * 4 : myws + OFF_XFLAG + su * 128 + (lane & 7) * 4;
    unsigned ready = 0x01010101u;
    bool dead = false;

    auto wait_step = [&](int n, bool lazy) {
        if (!((ready >> n) & 1) && !dead) {
            const unsigned long long t0 = __builtin_amdgcn_s_memrealtime(); unsigned polls = 0;
            for (;;) {
                const unsigned long long mk = __ballot(ld_sys(flagp) >= call);
                ready = 0x01010101u | ((unsigned)mk & (((unsigned)(mk >> 32) & 0xffu) * 0x01010101u));
                if ((ready >> n) & 1) break;
                if (give_up(myws, call, t0, polls)) { dead = true; break; }
                if (lazy) __builtin_amdgcn_s_sleep(32); else __builtin_amdgcn_s_sleep(8);
            }
            __builtin_amdgcn_s_waitcnt(0xC07F);
        }
    };
    const char* ga; const char* gb;
    auto stage_addr = [&](int o) {
        const int kc = o < 4 ? o : (o - 4) / 7, delta = o < 4 ? 0 : 1 + (o - 4) % 7;
        wait_step(kc * 8 + delta, false);
        ga = (delta == 0 ? aown : astage + (size_t)delta * (NSU * ITEM)) + kc * 128;
        gb = bbase + (size_t)delta * (32 * 65536) + kc * ITEM;
    };
    auto stage_piece = [&](int q, int buf) {
        if (q < 4) __builtin_amdgcn_global_load_lds((const GAS unsigned*)(ga + q * 4096 + (alane ^ ((q & 1) * 64))),
                                                    (LAS unsigned*)(lds + buf * STAGE_BYTES + wave * 4096 + q * 1024), 16, 0, GAUX);
        else __builtin_amdgcn_global_load_lds((const GAS unsigned*)(gb + (q - 4) * 1024),
                                              (LAS unsigned*)(lds + buf * STAGE_BYTES + 32768 + wave * 2048 + (q - 4) * 1024), 16, 0, GAUX);
    };

    f32x16 acc[4];
#pragma unroll
    for (int c = 0; c < 4; ++c)
#pragma unroll
        for (int e = 0; e < 16; ++e) acc[c][e] = 0.f;

    const int aoff = wave * 4096 + (lane & 31) * 128 + (((lane >> 5) ^ ((lane >> 1) & 7)) * 16);
    const int boff = 32768 + (lane >> 5) * 2048 + (lane & 31) * 16;
    auto frag = [&](const LAS char* p0, const LAS char* p1) -> i32x8 {
        const u32x4 lo = *(const LAS u32x4*)p0, hi = *(const LAS u32x4*)p1;
        return (i32x8){(int)lo[0], (int)lo[1], (int)lo[2], (int)lo[3], (int)hi[0], (int)hi[1], (int)hi[2], (int)hi[3]};
    };

    i32x8 a0, b0[4], a1, b1[4];
    auto read_half = [&](int buf, int kk, i32x8& af, i32x8 (&bf)[4]) {
        const LAS char* s = lds + buf * STAGE_BYTES;
        af = frag(s + (aoff ^ (kk * 64)), s + (aoff ^ (kk * 64 + 32)));
#pragma unroll
        for (int c = 0; c < 4; ++c) bf[c] = frag(s + boff + kk * 8192 + c * 512, s + boff + kk * 8192 + 4096 + c * 512);
    };
#define WAIT_LDS() __builtin_amdgcn_s_waitcnt(0xC07F)

    __builtin_amdgcn_s_barrier();
#pragma unroll
    for (int o = 0; o < 3; ++o) {
        stage_addr(o);
#pragma unroll
        for (int q = 0; q < 6; ++q) stage_piece(q, o);
    }
    WAIT_VM(12);
    __builtin_amdgcn_s_barrier();
    asm volatile("" ::: "memory");
    read_half(0, 0, a0, b0);
    int buf = 0;
    for (int n = 0; n < 32; ++n) {
        WAIT_LDS();
        read_half(buf, 1, a1, b1);
        __builtin_amdgcn_sched_barrier(0);
#pragma unroll
        for (int c = 0; c < 4; ++c) acc[c] = __builtin_amdgcn_mfma_scale_f32_32x32x64_f8f6f4(a0, b0[c], acc[c], 0, 0, 0, 0, 0, 0);
        __builtin_amdgcn_sched_barrier(0);
        if (n == 1 && wave == 0) wait_step(1, true);
        WAIT_LDS();
        WAIT_VM(6);
        __builtin_amdgcn_s_barrier();
        asm volatile("" ::: "memory");
        const int nb = buf == 2 ? 0 : buf + 1;
        stage_addr(n + 3 < 32 ? n + 3 : 31);
        read_half(nb, 0, a0, b0);
        __builtin_amdgcn_sched_barrier(0);
#pragma unroll
        for (int c = 0; c < 4; ++c) {
            acc[c] = __builtin_amdgcn_mfma_scale_f32_32x32x64_f8f6f4(a1, b1[c], acc[c], 0, 0, 0, 0, 0, 0);
            stage_piece(c, buf);
            if (c < 2) stage_piece(4 + c, buf);
            __builtin_amdgcn_sched_barrier(0);
        }
        buf = nb;
    }
    STAMP(3);

    const int delta = (grow >> 9) ^ r, ct = col_block(r, m) * 8 + t * 2 + G;
    unsigned tk = 0;
    if (delta != 0 && lane == 0) tk = __hip_atomic_fetch_add((GAS unsigned*)(myws + OFF_CNT + (8 + delta - 1) * 128), 1u, __ATOMIC_RELAXED, __HIP_MEMORY_SCOPE_AGENT);
    const bool bad = __builtin_amdgcn_readfirstlane(ld_agent(myws + OFF_TMO)) == call;
    WAIT_VM(0);
    const int g = lane >> 5, i = lane & 31;
    unsigned desc = ((grow & 511) >> 5) | (ct << 4);
    const char* dst; unsigned pitch;
    int slot = 0;
    bool raw = true;
    if (delta == 0) { dst = (const char*)(p.out[0] + (size_t)(grow & 511) * N_ + ct * 128); pitch = N_ * 4; }
    else {
        const int nout = (delta & 1) ? 24 * SU_O : 32 * SU_S;
        tk = __builtin_amdgcn_readfirstlane(tk) - (call - 1) * (unsigned)nout;
        slot = tk < (unsigned)nout ? (int)tk : nout - 1;
        dst = myws + OFF_OB + ((size_t)(delta - 1) * NOUT_MAX + slot) * ITEM; pitch = 512;
        raw = bad || slot >= nout - NRAW;
        if (!raw) {
            unsigned mx = 0;
#pragma unroll
            for (int c = 0; c < 4; ++c)
#pragma unroll
                for (int e = 0; e < 16; ++e) mx = max(mx, __float_as_uint(acc[c][e]) & 0x7fffffffu);
#pragma unroll
            for (int o = 1; o < 64; o <<= 1) mx = max(mx, (unsigned)__shfl_xor((int)mx, o));
            const int emax = __builtin_amdgcn_readfirstlane(mx) >> 23;
            const int sh = emax > 3 ? emax - 3 : 0;
            bool ok = true;
            unsigned D[52], tops = 0;
#pragma unroll
            for (int e = 0; e < 16; ++e) {
                unsigned cd[4];
#pragma unroll
                for (int c = 0; c < 4; ++c) {
                    const float z = __builtin_amdgcn_ldexpf(acc[c][e], -sh);
                    ok &= __builtin_amdgcn_ldexpf(z, sh) == acc[c][e];
                    cd[c] = __builtin_bit_cast(unsigned, z);
                }
                D[3 * e]     = __builtin_amdgcn_perm(cd[3], cd[0], 0x04020100u);
                D[3 * e + 1] = __builtin_amdgcn_perm(cd[3], cd[1], 0x05020100u);
                D[3 * e + 2] = __builtin_amdgcn_perm(cd[3], cd[2], 0x06020100u);
                const unsigned T = __builtin_amdgcn_perm(__builtin_amdgcn_perm(cd[3], cd[2], 0x07030703u), __builtin_amdgcn_perm(cd[1], cd[0], 0x07030703u), 0x05040100u);
                tops |= T;
                if ((e & 3) == 0) D[48 + (e >> 2)] = T; else D[48 + (e >> 2)] |= __builtin_amdgcn_alignbit(T, T, 32 - 2 * (e & 3));
            }
            ok &= (tops & 0x7e7e7e7eu) == 0;
            if (__all(ok)) {
#pragma unroll
                for (int q = 0; q < PK_CHUNKS; ++q) st16_wt(dst, q * 1024 + lane * 16, (u32x4){D[4 * q], D[4 * q + 1], D[4 * q + 2], D[4 * q + 3]});
                desc |= (unsigned)sh << 12;
            } else raw = true;
        }
    }
    if (raw) {
        const float nan = __builtin_nanf("");
#pragma unroll
        for (int e = 0; e < 16; ++e) {
            f32x4 v = {dq_silu(acc[0][e], scale), dq_silu(acc[1][e], scale), dq_silu(acc[2][e], scale), dq_silu(acc[3][e], scale)};
            if (bad) v = (f32x4){nan, nan, nan, nan};
            st16_wt(dst, ((e & 3) + 8 * (e >> 2) + 4 * g) * pitch + i * 16, __builtin_bit_cast(u32x4, v));
        }
        desc |= D_RAW;
    }
    if (delta != 0) {
        WAIT_VM(0);
        if (lane == 0) st8_sys(myws + OFF_OQ + ((delta - 1) * 256 + slot) * 8, ((unsigned long long)call << 32) | desc);
    }
    STAMP(4);
}

__device__ __forceinline__ void unpack_items(const Params& p, int r, char* myws, unsigned call, int wave, int lane, float scale) {
    const unsigned base = (call - 1) * (unsigned)(7 * NOUT_MAX + NGEMM * UPW);
    bool dead = false;
    for (;;) {
        const unsigned a = ticket(myws + OFF_CNT + 16 * 128, lane) - base;
        if (a >= 7u * NOUT_MAX) break;
        const int delta = a % 7 + 1, slot = a / 7;
        const int nout = (delta & 1) ? 24 * SU_O : 32 * SU_S;
        if (slot >= nout || dead) continue;
        unsigned desc;
        const unsigned long long t0 = __builtin_amdgcn_s_memrealtime(); unsigned polls = 0;
        for (;;) {
            const unsigned long long e = ld8_sys(myws + OFF_RQ + (delta * 256 + slot) * 8);
            desc = __builtin_amdgcn_readfirstlane((unsigned)e);
            if ((unsigned)__builtin_amdgcn_readfirstlane((unsigned)(e >> 32)) == call) break;
            if (give_up(myws, call, t0, polls)) { dead = true; break; }
            if (slot >= nout - 2 * NRAW) __builtin_amdgcn_s_sleep(4); else __builtin_amdgcn_s_sleep(32);
        }
        if (dead || (desc & D_RAW)) continue;
        const __amdgpu_buffer_rsrc_t rs = __builtin_amdgcn_make_buffer_rsrc(myws + OFF_RB + ((size_t)delta * NOUT_MAX + slot) * ITEM, 0, ITEM, 0x00020000);
        u32x4 in[PK_CHUNKS];
#pragma unroll
        for (int q = 0; q < PK_CHUNKS; ++q) in[q] = __builtin_amdgcn_raw_buffer_load_b128(rs, lane * 16, q * 1024, 17);
        const int sh = (desc >> 12) & 255;
        const char* dst = (const char*)(p.out[0] + (size_t)(desc & 15) * 32 * N_ + ((desc >> 4) & 127) * 128);
#pragma unroll
        for (int e = 0; e < 16; ++e) {
            const unsigned d0 = in[(3 * e) >> 2][(3 * e) & 3], d1 = in[(3 * e + 1) >> 2][(3 * e + 1) & 3], d2 = in[(3 * e + 2) >> 2][(3 * e + 2) & 3];
            const unsigned H = in[12][e >> 2];
            const unsigned T = ((e & 3) ? __builtin_amdgcn_alignbit(H, H, 2 * (e & 3)) : H) & 0x81818181u;
            const unsigned c0 = __builtin_amdgcn_perm(T, d0, 0x04020100u), c1 = __builtin_amdgcn_perm(T, d1, 0x05020100u), c2 = __builtin_amdgcn_perm(T, d2, 0x06020100u);
            const unsigned c3 = __builtin_amdgcn_perm(__builtin_amdgcn_perm(T, d2, 0x07030703u), __builtin_amdgcn_perm(d1, d0, 0x07030703u), 0x05040100u);
            const f32x4 v = {dq_silu(__builtin_amdgcn_ldexpf(__builtin_bit_cast(float, c0), sh), scale), dq_silu(__builtin_amdgcn_ldexpf(__builtin_bit_cast(float, c1), sh), scale),
                             dq_silu(__builtin_amdgcn_ldexpf(__builtin_bit_cast(float, c2), sh), scale), dq_silu(__builtin_amdgcn_ldexpf(__builtin_bit_cast(float, c3), sh), scale)};
            st16_wt(dst, ((e & 3) + 8 * (e >> 2) + 4 * (lane >> 5)) * (unsigned)(N_ * 4) + (lane & 31) * 16, __builtin_bit_cast(u32x4, v));
        }
    }
    if (dead && lane == 0) p.out[0][(size_t)col_block(r, 0) * 1024] = __builtin_nanf("");
    STAMP(5);
}

__global__ void __launch_bounds__(512, 2) fused_gemm_rs(Params p) {
    extern __shared__ __attribute__((aligned(16))) char smem[];
    const int lane = threadIdx.x & 63;
    const int wave = __builtin_amdgcn_readfirstlane(threadIdx.x >> 6);
    const int r = __builtin_amdgcn_readfirstlane(p.rank) & 7;
    char* myws = p.ws[0];
    char* cp = myws + OFF_CALL + (blockIdx.x * 8 + wave) * 4;
    const unsigned callv = ld_agent(cp);
    if (blockIdx.x < NENG) {
        engine(p, r, myws, callv, wave, lane);
        st_agent(cp, callv + 1);
        return;
    }
    const unsigned call = __builtin_amdgcn_readfirstlane(callv) + 1;
    st_agent(cp, call);

    STAMP(0);
    pretile(p, r, myws, call, wave, lane);
    float scale = p.sx[0] * p.sw[0];
    asm volatile("" : "+v"(scale));
    bool dead = false;
    if (wave == 0) wait_pretile(myws, call, lane, true, dead);
    STAMP(1);
    const int e = blockIdx.x & 7, slot = (blockIdx.x - NENG) >> 3, m = e >> 1;
    for (int id = slot; id < 32; id += NGEMM / 8) {
        int t, G, rb;
        if (id < 24) { t = id / 6; const int q = id % 6; G = q / 3; rb = 3 * (e & 1) + q % 3; }
        else { const int q = id - 24; t = q >> 1; G = q & 1; rb = 6 + (e & 1); }
        gemm_tile(p, r, myws, call, (LAS char*)smem, wave, lane, t, m, G, rb, scale);
    }
    if (wave < UPW) unpack_items(p, r, myws, call, wave, lane, scale);
}

}

extern "C" void kernel_launch(int rank, int nranks, ncclComm_t comm,
                              hipStream_t stream,
                              void* const* d_in, const int* in_sizes, int n_in,
                              void* d_out, int out_size,
                              const int* in_dtypes, int out_dtype,
                              void* const* d_out_all,
                              void* const* d_ws_all, size_t ws_bytes) {
    (void)comm; (void)in_dtypes; (void)out_dtype; (void)d_out;
    if (nranks != NR || (unsigned)rank >= (unsigned)NR || n_in < 4 || ws_bytes < WS_TOTAL || out_size != 512 * N_ ||
        in_sizes[0] != 4096 * KS || in_sizes[1] != KS * N_) {
        fprintf(stderr, "kernel_launch: unexpected problem geometry\n");
        return;
    }
    static bool attr_set[NR] = {};
    if (!attr_set[rank]) {
        HIP_CHECK(hipFuncSetAttribute((const void*)fused_gemm_rs, hipFuncAttributeMaxDynamicSharedMemorySize, LDS_BYTES));
        attr_set[rank] = true;
    }
    Params p;
    p.x = (const unsigned char*)d_in[0]; p.w = (const unsigned char*)d_in[1];
    p.sx = (const float*)d_in[2]; p.sw = (const float*)d_in[3];
    for (int i = 0; i < NR; ++i) { p.ws[i] = (char*)d_ws_all[rank ^ i]; p.out[i] = (float*)d_out_all[rank ^ i]; }
    p.rank = rank; p.pad = 0;
    hipLaunchKernelGGL(fused_gemm_rs, dim3(NWG), dim3(512), LDS_BYTES, stream, p);
    HIP_CHECK(hipGetLastError());
}
```

```cpp
#include <hip/hip_runtime.h>
#include <rccl/rccl.h>
#include <cstdio>
#include <cstdint>

#define HIP_CHECK(expr) do { hipError_t _e = (expr); if (_e != hipSuccess) { \
    fprintf(stderr, "HIP error %s at %s:%d\n", hipGetErrorString(_e), __FILE__, __LINE__); \
    return; } } while (0)

namespace {

typedef unsigned u32x4 __attribute__((ext_vector_type(4)));
typedef int i32x8 __attribute__((ext_vector_type(8)));
typedef float f32x16 __attribute__((ext_vector_type(16)));
typedef float f32x4 __attribute__((ext_vector_type(4)));
#define LAS __attribute__((address_space(3)))
#define GAS __attribute__((address_space(1)))

constexpr int NR = 8;
constexpr int N_ = 16384, KS = 512;
#define NENG_ 48
#define EPW 2
constexpr int NWG = 256, NENG = NENG_, NGEMM = NWG - NENG;
#define EW 12
#define WIN 6
#define PWIN 5
#define PACK_DELAY 1
#define GAUX 17
constexpr int SU_S = 7, SU_O = 9;
constexpr int NSU_REMOTE = 3 * SU_S + 4 * SU_O;
constexpr int NSU = NSU_REMOTE + 16;
constexpr int ITEM = 16384;
constexpr int NOUT_MAX = 4 * 8 * SU_S;
constexpr int NPUSH = 2 * EW;
constexpr int J_X = 32, J_W1 = J_X + NSU_REMOTE - NPUSH, J_XO = J_W1 + 96, J_OUT = J_XO + 16, RQN = 512;
constexpr int EXC_MAX = 256;
constexpr unsigned BIAS4 = 0x08080808u;

constexpr size_t OFF_CALL  = 0;
constexpr size_t OFF_TMO   = 16384;
constexpr size_t OFF_CNT   = 16384 + 128;
constexpr size_t OFF_WFLAG = 24576;
constexpr size_t OFF_XFLAG = 32768;
constexpr size_t OFF_RQ    = 65536;
constexpr size_t OFF_DBG   = 262144;
constexpr size_t SBW_BYTES = (size_t)512 * ITEM;
constexpr size_t SBX_BYTES = (size_t)128 * ITEM;
constexpr size_t OB_BYTES  = (size_t)7 * NOUT_MAX * ITEM;
constexpr size_t SEND_BYTES = SBW_BYTES + SBX_BYTES + OB_BYTES;
constexpr size_t XS_BYTES  = (size_t)NR * NSU * ITEM;
constexpr size_t WS_BYTES  = (size_t)NR * 32 * 65536;
constexpr size_t OFF_SEND  = 1u << 20;
constexpr size_t OFF_XS    = OFF_SEND + 2 * SEND_BYTES;
constexpr size_t OFF_WS    = OFF_XS + 2 * XS_BYTES;
constexpr size_t WS_TOTAL  = OFF_WS + WS_BYTES;
static_assert(OFF_CNT + 16 * 128 <= OFF_WFLAG && OFF_XFLAG + NSU * 128 <= OFF_RQ && OFF_RQ + 2 * NR * RQN * 8 <= OFF_DBG, "flags overlap");
static_assert(7 * EW <= NENG * EPW && NENG % 8 == 0 && EPW <= 8 && NPUSH <= NSU_REMOTE && NGEMM >= 160 && J_OUT + NOUT_MAX <= RQN, "geometry");
constexpr unsigned D_RAW = 1u << 11;

constexpr int STAGE_BYTES = 49152, NSTAGE = 3, LDS_BYTES = NSTAGE * STAGE_BYTES;
constexpr unsigned long long DEADLINE = 50000000ull;

struct Params {
    const unsigned char* x; const unsigned char* w; const float* sx; const float* sw;
    char* ws[NR];
    float* out;
    int rank; int pad;
};

__device__ __forceinline__ unsigned ld_sys(const void* p) { return __hip_atomic_load((const GAS unsigned*)p, __ATOMIC_RELAXED, __HIP_MEMORY_SCOPE_SYSTEM); }
__device__ __forceinline__ void st_sys(void* p, unsigned v) { __hip_atomic_store((GAS unsigned*)p, v, __ATOMIC_RELAXED, __HIP_MEMORY_SCOPE_SYSTEM); }
__device__ __forceinline__ unsigned ld_agent(const void* p) { return __hip_atomic_load((const GAS unsigned*)p, __ATOMIC_RELAXED, __HIP_MEMORY_SCOPE_AGENT); }
__device__ __forceinline__ void st_agent(void* p, unsigned v) { __hip_atomic_store((GAS unsigned*)p, v, __ATOMIC_RELAXED, __HIP_MEMORY_SCOPE_AGENT); }
__device__ __forceinline__ unsigned long long ld8_sys(const void* p) { return __hip_atomic_load((const GAS unsigned long long*)p, __ATOMIC_RELAXED, __HIP_MEMORY_SCOPE_SYSTEM); }
__device__ __forceinline__ void st8_sys(void* p, unsigned long long v) { __hip_atomic_store((GAS unsigned long long*)p, v, __ATOMIC_RELAXED, __HIP_MEMORY_SCOPE_SYSTEM); }
__device__ __forceinline__ void st16_wt(const void* base, unsigned off, u32x4 v) {
    asm volatile("global_store_dwordx4 %0, %1, %2 sc0 sc1\n\ts_nop 1" :: "v"(off), "v"(v), "s"(base) : "memory");
}
#define WAIT_VM(n) asm volatile("s_waitcnt vmcnt(%0)" :: "i"(n) : "memory")
__device__ __forceinline__ unsigned bfi(unsigned mask, unsigned a, unsigned b) { return (a & mask) | (b & ~mask); }
__device__ __forceinline__ unsigned shl(unsigned v, int s) { return s >= 0 ? v << s : v >> -s; }

__device__ __forceinline__ bool give_up(char* myws, unsigned call, unsigned long long t0, unsigned& polls) {
    bool dead = __builtin_amdgcn_s_memrealtime() - t0 > DEADLINE;
    if ((++polls & 63u) == 0u) dead |= __builtin_amdgcn_readfirstlane(ld_agent(myws + OFF_TMO)) == call;
    if (dead) st_agent(myws + OFF_TMO, call);
    return dead;
}

__device__ __forceinline__ int col_block(int r, int m) { return m * 4 + ((r & 1) ? ((r >> 1) ^ m) : (r >> 1)); }
__device__ __forceinline__ int unit_of(int r, int su) {
    if (su >= NSU_REMOTE) return r * 16 + su - NSU_REMOTE;
    const int g = su / 19, pos = su % 19;
    if (pos < 2 || (pos >= 10 && pos < 15)) { const int sg = g * 7 + (pos < 2 ? pos : pos - 8); return (r ^ (2 * (sg % 3 + 1))) * 16 + sg / 3; }
    const int jj = pos < 10 ? (pos - 2) >> 1 : pos - 15, idx = g * 3 + (pos < 10 ? (pos & 1) : 2);
    return (r ^ (2 * jj + 1)) * 16 + SU_S + idx;
}
__device__ __forceinline__ int staged_of(int r, int gu) {
    const int rel = (gu >> 4) ^ r, u = gu & 15;
    if (rel == 0) return NSU_REMOTE + u;
    if (((rel & 1) != 0) != (u >= SU_S)) return -1;
    if (u < SU_S) { const int sg = u * 3 + (rel >> 1) - 1, rho = sg % 7; return (sg / 7) * 19 + (rho < 2 ? rho : rho + 8); }
    const int idx = u - SU_S, rho = idx % 3, jj = rel >> 1;
    return (idx / 3) * 19 + (rho < 2 ? 2 + 2 * jj + rho : 15 + jj);
}

#define STAMP(slot) do {} while (0)
#define STAMPC(slot) do {} while (0)

__device__ __forceinline__ float dq_silu(float a, float s) {
    const float y = a * s;
    const float c = __builtin_amdgcn_fmed3f(y, -60.0f, 60.0f);
    return y * __builtin_amdgcn_rcpf(1.0f + __builtin_amdgcn_exp2f(c * -1.44269504088896341f));
}

template <int V> struct IC { static constexpr int value = V; };
constexpr int eng_extra(int s) { return ((s == 4 + WIN || s == 8 + WIN || s == WIN - 4 || s == WIN - 1) ? 4 : 0) + (s == 0 ? 2 : 0); }
constexpr int eng_wait(int o) { int n = WIN - 1; for (int s = o - WIN; s < o; ++s) n += eng_extra((s + 16) % 16); return n; }
static_assert(WIN >= 4 && WIN <= 7, "engine schedule");

__device__ __forceinline__ void engine(const Params& p, int r, char* myws, char* callp, LAS char* lds, int wave, int lane) {
    const int gw = blockIdx.x * EPW + wave;
    if (wave >= EPW || gw >= 7 * EW) return;
    const int link = gw % 7, k = gw / 7, delta = link + 1, d = (r ^ delta) & 7;
    char* dws = p.ws[delta];
    const int nitems = J_OUT + ((delta & 1) ? 24 * SU_O : 32 * SU_S);
    const unsigned lin = lane * 16;
    const char* claimp = myws + OFF_CNT + link * 128;
    auto srd = [](const void* base, int bytes) { return __builtin_amdgcn_make_buffer_rsrc((void*)base, 0, bytes, 0x00020000); };
    auto ld = [&](u32x4& v, int c, __amdgpu_buffer_rsrc_t src, unsigned skip) {
        asm volatile("buffer_load_dwordx4 %0, %1, %2, 0 offen offset:%3 sc0 sc1" : "=&v"(v) : "v"(lin + (c >> 2) * 4096 + skip), "s"(src), "i"((c & 3) * 1024) : "memory");
    };
    auto st = [&](__amdgpu_buffer_rsrc_t dst, unsigned off, u32x4 v) {
        asm volatile("buffer_store_dwordx4 %0, %1, %2, 0 offen sc0 sc1\n\ts_nop 1" :: "v"(v), "v"(off), "s"(dst) : "memory");
    };
#define PIN4(a, b) asm volatile("" : "+v"(a[4 * (b)]), "+v"(a[4 * (b) + 1]), "+v"(a[4 * (b) + 2]), "+v"(a[4 * (b) + 3]) :: "memory")
    u32x4 R[16];
    unsigned long long win;
    int j = k;
    float scale;
    unsigned call;
    {
        u32x4 Q[16];
        unsigned callv; float sxv, swv;
        const __amdgpu_buffer_rsrc_t sa = srd(p.x + (size_t)unit_of(d, k) * ITEM, ITEM), sb = srd(p.x + (size_t)unit_of(d, k + EW) * ITEM, ITEM);
        asm volatile("global_load_dword %0, %3, %4 sc1\n\tglobal_load_dword %1, %3, %5\n\tglobal_load_dword %2, %3, %6"
                     : "=&v"(callv), "=&v"(sxv), "=&v"(swv) : "v"(0u), "s"(callp), "s"(p.sx), "s"(p.sw) : "memory");
#pragma unroll
        for (int c = 0; c < 16; ++c) ld(Q[c], c, sa, 0);
#pragma unroll
        for (int c = 0; c < 16; ++c) ld(R[c], c, sb, 0);
        WAIT_VM(16);
        asm volatile("" : "+v"(callv), "+v"(sxv), "+v"(swv) :: "memory");
        PIN4(Q, 0); PIN4(Q, 1); PIN4(Q, 2); PIN4(Q, 3);
        call = __builtin_amdgcn_readfirstlane(callv) + 1;
        scale = sxv * swv;
        STAMP(0);
        const char* dst = dws + OFF_XS + (size_t)(call & 1) * XS_BYTES + ((size_t)delta * NSU + k) * ITEM;
#pragma unroll
        for (int c = 0; c < 32; ++c) {
            if (c == 16) { PIN4(R, 0); PIN4(R, 1); PIN4(R, 2); PIN4(R, 3); }
            st16_wt(dst, lin + (c & 15) * 1024 + (c >> 4) * (EW * ITEM), c < 16 ? Q[c & 15] : R[c & 15]);
            if (c >= PWIN - 1) WAIT_VM(PWIN - 1);
            if (c == 16 + PWIN - 2 && lane == 0) st_sys(dws + OFF_XFLAG + k * 128 + delta * 4, call);
            if (c == 24) asm volatile("global_load_dwordx2 %0, %1, %2 sc0 sc1" : "=&v"(win)
                                      : "v"((unsigned)(j + lane) * 8u), "s"(myws + OFF_RQ + ((size_t)(call & 1) * NR + delta) * (RQN * 8)) : "memory");
        }
        asm volatile("" : "+v"(win) :: "memory");
    }
    STAMP(1);
    const size_t par = call & 1;
    const char* rq = myws + OFF_RQ + (par * NR + delta) * (RQN * 8);
    const char* send = dws + OFF_SEND + par * SEND_BYTES;
    const unsigned cbase = (call - 1) * (unsigned)nitems;
    LAS unsigned* bm = (LAS unsigned*)(lds + wave * 4096);
    bool dead = false;
    int wb = j;
    int kind = 0, sh = 0; const char* dst = myws; char* flag = dws + OFF_XFLAG + (k + EW) * 128 + delta * 4;
    bool busy = true;
    u32x4 aux = {0, 0, 0, 0};
    for (;;) {
        const bool has = j < nitems;
        if (!has && !busy) break;
        unsigned desc = 0;
        bool now = has;
        if (has) {
            if ((unsigned)(j - wb) < 64u && (unsigned)__builtin_amdgcn_readlane((unsigned)(win >> 32), j - wb) == call) desc = __builtin_amdgcn_readlane((unsigned)win, j - wb);
            else if (busy) now = false;
            else if (!dead) {
                const unsigned long long t0 = __builtin_amdgcn_s_memrealtime(); unsigned polls = 0;
                for (;;) {
                    const unsigned long long e = ld8_sys(rq + j * 8);
                    desc = __builtin_amdgcn_readfirstlane((unsigned)e);
                    if ((unsigned)__builtin_amdgcn_readfirstlane((unsigned)(e >> 32)) == call) break;
                    if (give_up(myws, call, t0, polls)) { dead = true; break; }
                    __builtin_amdgcn_s_sleep(4);
                }
            }
        }
        const int pkind = kind, psh = sh; const char* pdst = dst; char* pflag = flag;
        const bool raw = (desc & D_RAW) != 0, is_out = j >= J_OUT;
        const char* src = send;
        flag = nullptr;
        if (is_out) {
            if (has) src = send + SBW_BYTES + SBX_BYTES + ((size_t)link * NOUT_MAX + (j - J_OUT)) * ITEM;
            dst = (const char*)(p.out + (size_t)(desc & 15) * 32 * N_ + ((desc >> 4) & 127) * 128);
        } else if (j < J_X || (j >= J_W1 && j < J_XO)) {
            const int q = j < J_X ? j : j - J_W1 + 32, t = q >> 5, kc = (q >> 3) & 3, m = (q >> 1) & 3, G = q & 1, tc = (m * 4 + t) * 2 + G;
            src = send + (size_t)(((col_block(r, m) * 4 + t) * 2 + G) * 4 + kc) * ITEM;
            dst = myws + OFF_WS + ((size_t)delta * 32 + tc) * 65536 + kc * ITEM;
            flag = myws + OFF_WFLAG + tc * 128 + (kc * 8 + delta) * 4;
        } else {
            const int su = j < J_W1 ? j - J_X + NPUSH : j - J_XO + NSU_REMOTE;
            src = send + SBW_BYTES + (size_t)unit_of(r, su) * ITEM;
            dst = myws + OFF_XS + par * XS_BYTES + ((size_t)delta * NSU + su) * ITEM;
            flag = myws + OFF_XFLAG + su * 128 + delta * 4;
        }
        kind = (!now || dead) ? 0 : (raw ? 1 : 3) + (is_out ? 1 : 0);
        if (kind == 0) flag = nullptr;
        sh = (desc >> 12) & 255;
        const __amdgpu_buffer_rsrc_t ssrc = srd(src, kind == 0 ? 0 : kind == 3 ? 15 * 1024 + 2 * EXC_MAX : ITEM);
        const unsigned skip3 = kind == 4 ? 2u * ITEM : 0u, skip7 = kind >= 3 ? 2u * ITEM : 0u;
        u32x4 last = {0, 0, 0, 0};
        auto quarter = [&](int b, int kd, const char* base, int shift) __attribute__((always_inline)) {
            const __amdgpu_buffer_rsrc_t sd = srd(base, kd == 0 ? 0 : (kd & 1) ? ITEM : 31 * N_ * 4 + 512);
            if (kd == 4) {
#pragma unroll
                for (int e = 0; e < 4; ++e) {
                    const unsigned d0 = R[4 * b + ((3 * e) >> 2)][(3 * e) & 3], d1 = R[4 * b + ((3 * e + 1) >> 2)][(3 * e + 1) & 3], d2 = R[4 * b + ((3 * e + 2) >> 2)][(3 * e + 2) & 3];
                    const unsigned T = (e ? __builtin_amdgcn_alignbit(aux[b], aux[b], 2 * e) : aux[b]) & 0x81818181u;
                    const unsigned c0 = __builtin_amdgcn_perm(T, d0, 0x04020100u), c1 = __builtin_amdgcn_perm(T, d1, 0x05020100u), c2 = __builtin_amdgcn_perm(T, d2, 0x06020100u);
                    const unsigned c3 = __builtin_amdgcn_perm(__builtin_amdgcn_perm(T, d2, 0x07030703u), __builtin_amdgcn_perm(d1, d0, 0x07030703u), 0x05040100u);
                    const f32x4 v = {dq_silu(__builtin_amdgcn_ldexpf(__uint_as_float(c0), shift), scale), dq_silu(__builtin_amdgcn_ldexpf(__uint_as_float(c1), shift), scale),
                                     dq_silu(__builtin_amdgcn_ldexpf(__uint_as_float(c2), shift), scale), dq_silu(__builtin_amdgcn_ldexpf(__uint_as_float(c3), shift), scale)};
                    st(sd, (e + 8 * b + 4 * (lane >> 5)) * (unsigned)(N_ * 4) + (lane & 31) * 16, __builtin_bit_cast(u32x4, v));
                }
            } else if (kd == 3) {
                if (b == 0) {
                    *(LAS u32x4*)(bm + lane * 8) = (u32x4){0, 0, 0, 0}; *(LAS u32x4*)(bm + lane * 8 + 4) = (u32x4){0, 0, 0, 0};
                    if (lane < 32) {
#pragma unroll
                        for (int e = 0; e < 8; ++e) {
                            const unsigned v = (aux[e >> 1] >> (16 * (e & 1))) & 0xffffu;
                            __hip_atomic_fetch_or(bm + min(v >> 5, 512u), 1u << (v & 31), __ATOMIC_RELAXED, __HIP_MEMORY_SCOPE_WAVEFRONT);
                        }
                    }
                    asm volatile("" ::: "memory");
                }
                const int pp = b >> 1;
                const u32x4 M = *(LAS u32x4*)(bm + lane * 8 + pp * 4);
                if ((b & 1) == 0) last = (u32x4){0, 0, 0, 0};
#pragma unroll
                for (int q = 4 * (b & 1); q < 4 * (b & 1) + 4; ++q) {
                    u32x4 o;
#pragma unroll
                    for (int c = 0; c < 4; ++c) {
                        unsigned v;
                        if (q < 7) { v = R[pp * 8 + q][c]; const int pl = q < 6 ? q : 7; last[c] |= shl(v, pl - 6) & (0x01010101u << pl); v &= 0xbfbfbfbfu; }
                        else v = last[c];
                        o[c] = (v + BIAS4) ^ (shl(M[c], 6 - q) & 0x40404040u);
                    }
                    st(sd, (lane >> 5) * 4096 + (lane & 31) * 16 + pp * 8192 + q * 512, o);
                }
            } else {
                const unsigned lo = kd == 2 ? (lane >> 5) * (unsigned)(N_ * 4) + (lane & 31) * 16 : lin, step = kd == 2 ? 2u * N_ * 4u : 1024u;
#pragma unroll
                for (int q = 4 * b; q < 4 * b + 4; ++q) st(sd, lo + q * step, q == 15 ? aux : R[q]);
            }
        };
        unsigned cl; unsigned long long keep;
        auto slot = [&](auto oc) __attribute__((always_inline)) {
            constexpr int o = decltype(oc)::value;
            WAIT_VM(eng_wait(o));
            if constexpr (o == WIN) { asm volatile("" : "+v"(R[15]) :: "memory"); aux = R[15]; asm volatile("" : "+v"(aux)); }
            if constexpr (o == 2 * WIN) {
                if (!now) WAIT_VM(0);
                if (pflag != nullptr && lane == 0) st_sys(pflag, call);
            }
            constexpr int c = (o + 15) & 15;
            ld(R[c], c, ssrc, c == 7 ? skip7 : (c == 3 || c == 11) ? skip3 : 0u);
            if constexpr (o == 0) {
                wb = j;
                asm volatile("s_mov_b64 %1, exec\n\ts_mov_b64 exec, 1\n\tglobal_atomic_add %0, %2, %3, %4 sc0\n\ts_mov_b64 exec, %1"
                             : "=&v"(cl), "=&s"(keep) : "v"(0u), "v"(now ? 1u : 0u), "s"(claimp) : "memory");
                asm volatile("global_load_dwordx2 %0, %1, %2 sc0 sc1" : "=&v"(win) : "v"((unsigned)((wb + lane) & (RQN - 1)) * 8u), "s"(rq) : "memory");
            }
            if constexpr (o == WIN - 4) { PIN4(R, 2); quarter(2, pkind, pdst, psh); }
            if constexpr (o == WIN - 1) { asm volatile("" : "+v"(R[12]), "+v"(R[13]), "+v"(R[14]) :: "memory"); quarter(3, pkind, pdst, psh); }
            if constexpr (o == WIN + 4) { PIN4(R, 0); quarter(0, kind, dst, sh); }
            if constexpr (o == WIN + 8) { PIN4(R, 1); quarter(1, kind, dst, sh); }
        };
        slot(IC<0>{}); slot(IC<1>{}); slot(IC<2>{}); slot(IC<3>{}); slot(IC<4>{}); slot(IC<5>{}); slot(IC<6>{}); slot(IC<7>{});
        slot(IC<8>{}); slot(IC<9>{}); slot(IC<10>{}); slot(IC<11>{}); slot(IC<12>{}); slot(IC<13>{}); slot(IC<14>{}); slot(IC<15>{});
        asm volatile("" : "+v"(cl), "+v"(win) :: "memory");
        busy = kind != 0;
        if (!now) continue;
        const unsigned c = __builtin_amdgcn_readfirstlane(cl) - cbase;
        j = c < (unsigned)nitems ? EW + (int)c : nitems;
    }
    WAIT_VM(0);
    st_agent(callp, call);
    if (dead && lane == 0) p.out[(size_t)col_block(r, 0) * 1024] = __builtin_nanf("");
    STAMP(4);
}

__device__ __forceinline__ void pack_operands(const Params& p, int r, char* myws, unsigned callv, LAS char* lds, int wave, int lane) {
    const int task = blockIdx.x - NENG;
    if (task >= 160) return;
    if (task >= 32) __builtin_amdgcn_s_sleep(PACK_DELAY);
    const int pair = task * 2 + (wave >> 2), pr = wave & 3, G = lane >> 5, i = lane & 31;
    LAS unsigned* cnt = (LAS unsigned*)lds;
    LAS unsigned short* list = (LAS unsigned short*)(lds + 64);
    if (threadIdx.x < 4) cnt[threadIdx.x] = 0;
    ((LAS unsigned*)list)[threadIdx.x] = 0xffffffffu;
    u32x4 o[8];
    const bool is_w = pair < 256;
    const int strip = ((pair >> 2) & 15) * 4 + ((pair >> 6) & 3), kc = pair & 3;
    if (is_w) {
        const unsigned* src = (const unsigned*)(p.w + (size_t)(kc * 128 + pr * 32) * N_ + strip * 256) + lane;
        unsigned v[32];
#pragma unroll
        for (int j = 0; j < 32; ++j) v[j] = __builtin_nontemporal_load(src + (size_t)j * (N_ / 4));
#pragma unroll
        for (int q = 0; q < 8; ++q) {
            const unsigned t0 = __builtin_amdgcn_perm(v[4 * q + 1], v[4 * q], 0x05010400u), t1 = __builtin_amdgcn_perm(v[4 * q + 1], v[4 * q], 0x07030602u);
            const unsigned t2 = __builtin_amdgcn_perm(v[4 * q + 3], v[4 * q + 2], 0x05010400u), t3 = __builtin_amdgcn_perm(v[4 * q + 3], v[4 * q + 2], 0x07030602u);
            o[(q & 4) + 0][q & 3] = __builtin_amdgcn_perm(t2, t0, 0x05040100u); o[(q & 4) + 1][q & 3] = __builtin_amdgcn_perm(t2, t0, 0x07060302u);
            o[(q & 4) + 2][q & 3] = __builtin_amdgcn_perm(t3, t1, 0x05040100u); o[(q & 4) + 3][q & 3] = __builtin_amdgcn_perm(t3, t1, 0x07060302u);
        }
    } else {
        const char* src = (const char*)p.x + (size_t)((pair - 256) * 2 + G) * ITEM + pr * 4096 + i * 16;
#pragma unroll
        for (int q = 0; q < 8; ++q) o[q] = *(const u32x4*)(src + q * 512);
    }
    u32x4 pk[7], M = {0, 0, 0, 0};
#pragma unroll
    for (int c = 0; c < 4; ++c) {
        unsigned cd[8];
#pragma unroll
        for (int q = 0; q < 8; ++q) {
            cd[q] = bfi(0x7f7f7f7fu, (o[q][c] | 0x80808080u) - BIAS4, o[q][c]);
            M[c] |= shl(cd[q], q - 6) & (0x01010101u << q);
        }
#pragma unroll
        for (int q = 0; q < 7; ++q) pk[q][c] = bfi(0x40404040u, shl(cd[7], 6 - (q < 6 ? q : 7)), cd[q]);
    }
    __syncthreads();
    asm volatile("" : "+v"(callv));
    const unsigned call = __builtin_amdgcn_readfirstlane(callv) + 1;
    const size_t par = call & 1;
    const int it = (wave >> 2) * 2 + G;
    {
        const unsigned n = __builtin_popcount(M[0]) + __builtin_popcount(M[1]) + __builtin_popcount(M[2]) + __builtin_popcount(M[3]);
        unsigned pos = n ? __hip_atomic_fetch_add(cnt + it, n, __ATOMIC_RELAXED, __HIP_MEMORY_SCOPE_WORKGROUP) : 0u;
        if (pos + n <= (unsigned)EXC_MAX) {
            const unsigned id = ((((pr & 1) * 32 + i) * 8) + (pr >> 1) * 4) << 5;
#pragma unroll
            for (int c = 0; c < 4; ++c)
                for (unsigned mm = M[c]; mm; mm &= mm - 1) list[it * EXC_MAX + pos++] = (unsigned short)(id + c * 32 + __builtin_ctz(mm));
        }
    }
    __syncthreads();
    const bool rawv = cnt[it] > (unsigned)EXC_MAX;
    const char* slot = myws + OFF_SEND + par * SEND_BYTES + (is_w ? (size_t)(strip * 8 + kc) * ITEM : SBW_BYTES + (size_t)(pair - 256) * 2 * ITEM);
    const unsigned ib = G * (is_w ? 4u * ITEM : (unsigned)ITEM) + i * 16;
#pragma unroll
    for (int q = 0; q < 7; ++q) st16_wt(slot, ib + (rawv ? pr * 4096 + q * 512 : ((pr >> 1) * 8 + q) * 1024 + (pr & 1) * 512), rawv ? o[q] : pk[q]);
    if (rawv || pr == 0) st16_wt(slot, ib + (rawv ? pr * 4096 + 7 * 512 : 15 * 1024), rawv ? o[7] : *(LAS u32x4*)(list + it * EXC_MAX + i * 8));
    const int cb = strip >> 2, t = strip & 3, m = cb >> 2;
    const bool mine = is_w && cb == col_block(r, m);
    if (mine) {
        const char* dst = myws + OFF_WS + (size_t)((m * 4 + t) * 2) * 65536 + kc * ITEM + pr * 4096;
#pragma unroll
        for (int q = 0; q < 8; ++q) st16_wt(dst, G * 65536 + q * 512 + i * 16, o[q]);
    }
    WAIT_VM(0);
    __syncthreads();
    if (pr == 0 && i < NR) {
        const int d = r ^ i;
        int j = -1;
        if (is_w) { if (cb == col_block(d, m)) { const int q = t * 32 + kc * 8 + m * 2 + G; j = t == 0 ? q : q - 32 + J_W1; } }
        else { const int su = staged_of(d, (pair - 256) * 2 + G); if (su >= NPUSH) j = su < NSU_REMOTE ? su - NPUSH + J_X : su - NSU_REMOTE + J_XO; }
        if (j >= 0) {
            if (i != 0) st8_sys(p.ws[i] + OFF_RQ + ((par * NR + i) * RQN + j) * 8, ((unsigned long long)call << 32) | (rawv ? D_RAW : 0u));
            else if (is_w) st_sys(myws + OFF_WFLAG + ((m * 4 + t) * 2 + G) * 128 + kc * 32, call);
        }
    }
}

__device__ __forceinline__ void gemm_tile(const Params& p, int r, char* myws, unsigned call, LAS char* lds, int wave, int lane,
                                          int t, int m, int G, int rb, float scale) {
    const size_t par = call & 1;
    int su;
    if (rb < 3) { const int j = (wave - 2) >> 1; su = rb * 19 + (wave < 2 ? wave : 2 + 2 * (j + (j >= m)) + (wave & 1)); }
    else if (rb < 6) { const int j = wave - 5; su = (rb - 3) * 19 + (wave < 5 ? 10 + wave : 15 + j + (j >= m)); }
    else su = NSU_REMOTE + (rb - 6) * 8 + wave;
    const int grow = unit_of(r, su) * 32;
    const int tc = (m * 4 + t) * 2 + G;
    const unsigned alane = (lane >> 3) * 512 + (((lane & 7) ^ (lane >> 4)) * 16);
    const char* astage = myws + OFF_XS + par * XS_BYTES + (size_t)su * ITEM;
    const char* aown = (const char*)p.x + (size_t)grow * KS;
    const char* bbase = myws + OFF_WS + (size_t)tc * 65536 + wave * 2048 + lane * 16;
    const char* flagp = lane < 32 ? myws + OFF_WFLAG + tc * 128 + lane * 4 : myws + OFF_XFLAG + su * 128 + (lane & 7) * 4;
    unsigned ready = 0;
    bool dead = false;

    auto wait_step = [&](int n, bool lazy) {
        if (!((ready >> n) & 1) && !dead) {
            const unsigned long long t0 = __builtin_amdgcn_s_memrealtime(); unsigned polls = 0;
            for (;;) {
                const unsigned long long mk = __ballot(ld_sys(flagp) >= call);
                ready = (unsigned)mk & ((((unsigned)(mk >> 32) & 0xffu) | 1u) * 0x01010101u);
                if ((ready >> n) & 1) break;
                if (give_up(myws, call, t0, polls)) { dead = true; break; }
                if (lazy) __builtin_amdgcn_s_sleep(32); else __builtin_amdgcn_s_sleep(8);
            }
            __builtin_amdgcn_s_waitcnt(0xC07F);
        }
    };
    const char* ga; const char* gb;
    auto stage_addr = [&](int o) {
        const int kc = o < 4 ? o : (o - 4) / 7, delta = o < 4 ? 0 : 1 + (o - 4) % 7;
        wait_step(kc * 8 + delta, false);
        ga = (delta == 0 ? aown : astage + (size_t)delta * (NSU * ITEM)) + kc * 128;
        gb = bbase + (size_t)delta * (32 * 65536) + kc * ITEM;
    };
    auto stage_piece = [&](int q, int buf) {
        if (q < 4) __builtin_amdgcn_global_load_lds((const GAS unsigned*)(ga + q * 4096 + (alane ^ ((q & 1) * 64))),
                                                    (LAS unsigned*)(lds + buf * STAGE_BYTES + wave * 4096 + q * 1024), 16, 0, GAUX);
        else __builtin_amdgcn_global_load_lds((const GAS unsigned*)(gb + (q - 4) * 1024),
                                              (LAS unsigned*)(lds + buf * STAGE_BYTES + 32768 + wave * 2048 + (q - 4) * 1024), 16, 0, GAUX);
    };

    f32x16 acc[4];
#pragma unroll
    for (int c = 0; c < 4; ++c)
#pragma unroll
        for (int e = 0; e < 16; ++e) acc[c][e] = 0.f;

    const int aoff = wave * 4096 + (lane & 31) * 128 + (((lane >> 5) ^ ((lane >> 1) & 7)) * 16);
    const int boff = 32768 + (lane >> 5) * 2048 + (lane & 31) * 16;
    auto frag = [&](const LAS char* p0, const LAS char* p1) -> i32x8 {
        const u32x4 lo = *(const LAS u32x4*)p0, hi = *(const LAS u32x4*)p1;
        return (i32x8){(int)lo[0], (int)lo[1], (int)lo[2], (int)lo[3], (int)hi[0], (int)hi[1], (int)hi[2], (int)hi[3]};
    };

    i32x8 a0, b0[4], a1, b1[4];
    auto read_half = [&](int buf, int kk, i32x8& af, i32x8 (&bf)[4]) {
        const LAS char* s = lds + buf * STAGE_BYTES;
        af = frag(s + (aoff ^ (kk * 64)), s + (aoff ^ (kk * 64 + 32)));
#pragma unroll
        for (int c = 0; c < 4; ++c) bf[c] = frag(s + boff + kk * 8192 + c * 512, s + boff + kk * 8192 + 4096 + c * 512);
    };
#define WAIT_LDS() __builtin_amdgcn_s_waitcnt(0xC07F)

    __builtin_amdgcn_s_barrier();
#pragma unroll
    for (int o = 0; o < 3; ++o) {
        stage_addr(o);
#pragma unroll
        for (int q = 0; q < 6; ++q) stage_piece(q, o);
    }
    WAIT_VM(12);
    __builtin_amdgcn_s_barrier();
    asm volatile("" ::: "memory");
    read_half(0, 0, a0, b0);
    int buf = 0;
    for (int n = 0; n < 32; ++n) {
        WAIT_LDS();
        read_half(buf, 1, a1, b1);
        __builtin_amdgcn_sched_barrier(0);
#pragma unroll
        for (int c = 0; c < 4; ++c) acc[c] = __builtin_amdgcn_mfma_scale_f32_32x32x64_f8f6f4(a0, b0[c], acc[c], 0, 0, 0, 0, 0, 0);
        __builtin_amdgcn_sched_barrier(0);
        if (n == 1 && wave == 0) wait_step(1, true);
        WAIT_LDS();
        WAIT_VM(6);
        __builtin_amdgcn_s_barrier();
        asm volatile("" ::: "memory");
        const int nb = buf == 2 ? 0 : buf + 1;
        stage_addr(n + 3 < 32 ? n + 3 : 31);
        read_half(nb, 0, a0, b0);
        __builtin_amdgcn_sched_barrier(0);
#pragma unroll
        for (int c = 0; c < 4; ++c) {
            acc[c] = __builtin_amdgcn_mfma_scale_f32_32x32x64_f8f6f4(a1, b1[c], acc[c], 0, 0, 0, 0, 0, 0);
            stage_piece(c, buf);
            if (c < 2) stage_piece(4 + c, buf);
            __builtin_amdgcn_sched_barrier(0);
        }
        buf = nb;
    }
    STAMP(3);

    const int delta = (grow >> 9) ^ r, ct = col_block(r, m) * 8 + t * 2 + G;
    unsigned tk = 0;
    if (delta != 0 && lane == 0) tk = __hip_atomic_fetch_add((GAS unsigned*)(myws + OFF_CNT + (8 + delta - 1) * 128), 1u, __ATOMIC_RELAXED, __HIP_MEMORY_SCOPE_AGENT);
    const bool bad = __builtin_amdgcn_readfirstlane(ld_agent(myws + OFF_TMO)) == call;
    WAIT_VM(0);
    const int g = lane >> 5, i = lane & 31;
    unsigned desc = ((grow & 511) >> 5) | (ct << 4);
    const char* dst; unsigned pitch;
    int slot = 0;
    bool raw = true;
    if (delta == 0) { dst = (const char*)(p.out + (size_t)(grow & 511) * N_ + ct * 128); pitch = N_ * 4; }
    else {
        const int nout = (delta & 1) ? 24 * SU_O : 32 * SU_S;
        tk = __builtin_amdgcn_readfirstlane(tk) - (call - 1) * (unsigned)nout;
        slot = tk < (unsigned)nout ? (int)tk : nout - 1;
        dst = myws + OFF_SEND + par * SEND_BYTES + SBW_BYTES + SBX_BYTES + ((size_t)(delta - 1) * NOUT_MAX + slot) * ITEM; pitch = 512;
        raw = bad;
        if (!raw) {
            unsigned mx = 0;
#pragma unroll
            for (int c = 0; c < 4; ++c)
#pragma unroll
                for (int e = 0; e < 16; ++e) mx = max(mx, __float_as_uint(acc[c][e]) & 0x7fffffffu);
#pragma unroll
            for (int o = 1; o < 64; o <<= 1) mx = max(mx, (unsigned)__shfl_xor((int)mx, o));
            const int emax = __builtin_amdgcn_readfirstlane(mx) >> 23;
            const int sh = emax > 3 ? emax - 3 : 0;
            bool ok = true;
            unsigned D[52], tops = 0;
#pragma unroll
            for (int e = 0; e < 16; ++e) {
                unsigned cd[4];
#pragma unroll
                for (int c = 0; c < 4; ++c) {
                    const float z = __builtin_amdgcn_ldexpf(acc[c][e], -sh);
                    ok &= __builtin_amdgcn_ldexpf(z, sh) == acc[c][e];
                    cd[c] = __builtin_bit_cast(unsigned, z);
                }
                D[3 * e]     = __builtin_amdgcn_perm(cd[3], cd[0], 0x04020100u);
                D[3 * e + 1] = __builtin_amdgcn_perm(cd[3], cd[1], 0x05020100u);
                D[3 * e + 2] = __builtin_amdgcn_perm(cd[3], cd[2], 0x06020100u);
                const unsigned T = __builtin_amdgcn_perm(__builtin_amdgcn_perm(cd[3], cd[2], 0x07030703u), __builtin_amdgcn_perm(cd[1], cd[0], 0x07030703u), 0x05040100u);
                tops |= T;
                if ((e & 3) == 0) D[48 + (e >> 2)] = T; else D[48 + (e >> 2)] |= __builtin_amdgcn_alignbit(T, T, 32 - 2 * (e & 3));
            }
            ok &= (tops & 0x7e7e7e7eu) == 0;
            if (__all(ok)) {
#pragma unroll
                for (int q = 0; q < 13; ++q)
                    st16_wt(dst, (q < 12 ? q + q / 3 : 15) * 1024 + lane * 16, (u32x4){D[4 * q], D[4 * q + 1], D[4 * q + 2], D[4 * q + 3]});
                desc |= (unsigned)sh << 12;
            } else raw = true;
        }
    }
    if (raw) {
        const float nan = __builtin_nanf("");
#pragma unroll
        for (int e = 0; e < 16; ++e) {
            f32x4 v = {dq_silu(acc[0][e], scale), dq_silu(acc[1][e], scale), dq_silu(acc[2][e], scale), dq_silu(acc[3][e], scale)};
            if (bad) v = (f32x4){nan, nan, nan, nan};
            st16_wt(dst, ((e & 3) + 8 * (e >> 2) + 4 * g) * pitch + i * 16, __builtin_bit_cast(u32x4, v));
        }
        desc |= D_RAW;
    }
    if (delta != 0) {
        WAIT_VM(0);
        if (lane == 0) st8_sys(p.ws[delta] + OFF_RQ + ((par * NR + delta) * RQN + J_OUT + slot) * 8, ((unsigned long long)call << 32) | desc);
    }
    STAMP(4);
}

__global__ void __launch_bounds__(512, 2) fused_gemm_rs(Params p) {
    extern __shared__ __attribute__((aligned(16))) char smem[];
    const int lane = threadIdx.x & 63;
    const int wave = __builtin_amdgcn_readfirstlane(threadIdx.x >> 6);
    const int r = __builtin_amdgcn_readfirstlane(p.rank) & 7;
    char* myws = p.ws[0];
    char* cp = myws + OFF_CALL + (blockIdx.x * 8 + wave) * 4;
    if (blockIdx.x < NENG) { engine(p, r, myws, cp, (LAS char*)smem, wave, lane); return; }
    const unsigned callv = ld_agent(cp);
    STAMP(0);
    pack_operands(p, r, myws, callv, (LAS char*)smem, wave, lane);
    const unsigned call = __builtin_amdgcn_readfirstlane(callv) + 1;
    st_agent(cp, call);
    float scale = p.sx[0] * p.sw[0];
    asm volatile("" : "+v"(scale));
    STAMP(1);
    const int e = blockIdx.x & 7, slot = (blockIdx.x - NENG) >> 3, m = e >> 1;
    for (int id = slot; id < 32; id += NGEMM / 8) {
        int t, G, rb;
        if (id < 24) { t = id / 6; const int q = id % 6; G = q & 1; rb = 3 * (e & 1) + (q >> 1); }
        else { const int q = id - 24; t = q >> 1; G = q & 1; rb = 6 + (e & 1); }
        gemm_tile(p, r, myws, call, (LAS char*)smem, wave, lane, t, m, G, rb, scale);
    }
}

}

extern "C" void kernel_launch(int rank, int nranks, ncclComm_t comm,
                              hipStream_t stream,
                              void* const* d_in, const int* in_sizes, int n_in,
                              void* d_out, int out_size,
                              const int* in_dtypes, int out_dtype,
                              void* const* d_out_all,
                              void* const* d_ws_all, size_t ws_bytes) {
    (void)comm; (void)in_dtypes; (void)out_dtype; (void)d_out_all;
    if (nranks != NR || (unsigned)rank >= (unsigned)NR || n_in < 4 || ws_bytes < WS_TOTAL || out_size != 512 * N_ ||
        in_sizes[0] != 4096 * KS || in_sizes[1] != KS * N_) {
        fprintf(stderr, "kernel_launch: unexpected problem geometry\n");
        return;
    }
    static bool attr_set[NR] = {};
    if (!attr_set[rank]) {
        HIP_CHECK(hipFuncSetAttribute((const void*)fused_gemm_rs, hipFuncAttributeMaxDynamicSharedMemorySize, LDS_BYTES));
        attr_set[rank] = true;
    }
    Params p;
    p.x = (const unsigned char*)d_in[0]; p.w = (const unsigned char*)d_in[1];
    p.sx = (const float*)d_in[2]; p.sw = (const float*)d_in[3];
    for (int i = 0; i < NR; ++i) p.ws[i] = (char*)d_ws_all[rank ^ i];
    p.out = (float*)d_out;
    p.rank = rank; p.pad = 0;
    hipLaunchKernelGGL(fused_gemm_rs, dim3(NWG), dim3(512), LDS_BYTES, stream, p);
    HIP_CHECK(hipGetLastError());
}
```

```cpp
#include <hip/hip_runtime.h>
#include <rccl/rccl.h>
#include <cstdio>
#include <cstdint>

#define HIP_CHECK(expr) do { hipError_t _e = (expr); if (_e != hipSuccess) { \
    fprintf(stderr, "HIP error %s at %s:%d\n", hipGetErrorString(_e), __FILE__, __LINE__); \
    return; } } while (0)

namespace {

typedef unsigned u32x4 __attribute__((ext_vector_type(4)));
typedef int i32x8 __attribute__((ext_vector_type(8)));
typedef float f32x16 __attribute__((ext_vector_type(16)));
typedef float f32x4 __attribute__((ext_vector_type(4)));
#define LAS __attribute__((address_space(3)))
#define GAS __attribute__((address_space(1)))

constexpr int NR = 8;
constexpr int N_ = 16384, KS = 512;
#define NENG_ 96
#define EPW 1
constexpr int NWG = 256, NENG = NENG_, NGEMM = NWG - NENG;
#define EW 13
#define WIN 6
#define PWIN 5
#define GAUX 17
constexpr int SU_S = 7, SU_O = 9;
constexpr int NSU_REMOTE = 3 * SU_S + 4 * SU_O;
constexpr int NSU = NSU_REMOTE + 16;
constexpr int ITEM = 16384;
constexpr int NOUT_MAX = 4 * 8 * SU_S;
constexpr int NPUSH = 2 * EW;
constexpr int J_X = 32, J_W1 = J_X + NSU_REMOTE - NPUSH, J_XO = J_W1 + 96, J_OUT = J_XO + 16, RQN = 512;
constexpr int EXC_MAX = 512;
constexpr unsigned BIAS4 = 0x05050505u;

constexpr size_t OFF_CALL  = 0;
constexpr size_t OFF_TMO   = 16384;
constexpr size_t OFF_CNT   = 16384 + 128;
constexpr size_t OFF_WFLAG = 24576;
constexpr size_t OFF_XFLAG = 32768;
constexpr size_t OFF_RQ    = 65536;
constexpr size_t OFF_DBG   = 262144;
constexpr size_t SBW_BYTES = (size_t)512 * ITEM;
constexpr size_t SBX_BYTES = (size_t)128 * ITEM;
constexpr size_t OB_BYTES  = (size_t)7 * NOUT_MAX * ITEM;
constexpr size_t SEND_BYTES = SBW_BYTES + SBX_BYTES + OB_BYTES;
constexpr size_t XS_BYTES  = (size_t)NR * NSU * ITEM;
constexpr size_t WS_BYTES  = (size_t)NR * 32 * 65536;
constexpr size_t OFF_SEND  = 1u << 20;
constexpr size_t OFF_XS    = OFF_SEND + 2 * SEND_BYTES;
constexpr size_t OFF_WS    = OFF_XS + 2 * XS_BYTES;
constexpr size_t WS_TOTAL  = OFF_WS + WS_BYTES;
static_assert(OFF_CNT + 16 * 128 <= OFF_WFLAG && OFF_XFLAG + NSU * 128 <= OFF_RQ && OFF_RQ + 2 * NR * RQN * 8 <= OFF_DBG, "flags overlap");
static_assert(EW <= (NENG / 7) * EPW && NENG % 8 == 0 && EPW <= 8 && NPUSH <= NSU_REMOTE && J_OUT + NOUT_MAX <= RQN, "geometry");
constexpr unsigned D_RAW = 1u << 11, D_WIDE = 1u << 30;

constexpr int STAGE_BYTES = 49152, NSTAGE = 3, LDS_BYTES = NSTAGE * STAGE_BYTES;
constexpr unsigned long long DEADLINE = 50000000ull;

struct Params {
    const unsigned char* x; const unsigned char* w; const float* sx; const float* sw;
    char* ws[NR];
    float* out;
    int rank; int pad;
};

__device__ __forceinline__ unsigned ld_sys(const void* p) { return __hip_atomic_load((const GAS unsigned*)p, __ATOMIC_RELAXED, __HIP_MEMORY_SCOPE_SYSTEM); }
__device__ __forceinline__ void st_sys(void* p, unsigned v) { __hip_atomic_store((GAS unsigned*)p, v, __ATOMIC_RELAXED, __HIP_MEMORY_SCOPE_SYSTEM); }
__device__ __forceinline__ unsigned ld_agent(const void* p) { return __hip_atomic_load((const GAS unsigned*)p, __ATOMIC_RELAXED, __HIP_MEMORY_SCOPE_AGENT); }
__device__ __forceinline__ void st_agent(void* p, unsigned v) { __hip_atomic_store((GAS unsigned*)p, v, __ATOMIC_RELAXED, __HIP_MEMORY_SCOPE_AGENT); }
__device__ __forceinline__ unsigned long long ld8_sys(const void* p) { return __hip_atomic_load((const GAS unsigned long long*)p, __ATOMIC_RELAXED, __HIP_MEMORY_SCOPE_SYSTEM); }
__device__ __forceinline__ void st8_sys(void* p, unsigned long long v) { __hip_atomic_store((GAS unsigned long long*)p, v, __ATOMIC_RELAXED, __HIP_MEMORY_SCOPE_SYSTEM); }
__device__ __forceinline__ void st16_wt(const void* base, unsigned off, u32x4 v) {
    asm volatile("global_store_dwordx4 %0, %1, %2 sc0 sc1\n\ts_nop 1" :: "v"(off), "v"(v), "s"(base) : "memory");
}
#define WAIT_VM(n) asm volatile("s_waitcnt vmcnt(%0)" :: "i"(n) : "memory")
__device__ __forceinline__ unsigned bfi(unsigned mask, unsigned a, unsigned b) { return (a & mask) | (b & ~mask); }
__device__ __forceinline__ unsigned shl(unsigned v, int s) { return s >= 0 ? v << s : v >> -s; }

__device__ __forceinline__ bool give_up(char* myws, unsigned call, unsigned long long t0, unsigned& polls) {
    bool dead = __builtin_amdgcn_s_memrealtime() - t0 > DEADLINE;
    if ((++polls & 63u) == 0u) dead |= __builtin_amdgcn_readfirstlane(ld_agent(myws + OFF_TMO)) == call;
    if (dead) st_agent(myws + OFF_TMO, call);
    return dead;
}

__device__ __forceinline__ int col_block(int r, int m) { return m * 4 + ((r & 1) ? ((r >> 1) ^ m) : (r >> 1)); }
__device__ __forceinline__ int unit_of(int r, int su) {
    if (su >= NSU_REMOTE) return r * 16 + su - NSU_REMOTE;
    const int g = su / 19, pos = su % 19;
    if (pos < 2 || (pos >= 10 && pos < 15)) { const int sg = g * 7 + (pos < 2 ? pos : pos - 8); return (r ^ (2 * (sg % 3 + 1))) * 16 + sg / 3; }
    const int jj = pos < 10 ? (pos - 2) >> 1 : pos - 15, idx = g * 3 + (pos < 10 ? (pos & 1) : 2);
    return (r ^ (2 * jj + 1)) * 16 + SU_S + idx;
}
__device__ __forceinline__ int staged_of(int r, int gu) {
    const int rel = (gu >> 4) ^ r, u = gu & 15;
    if (rel == 0) return NSU_REMOTE + u;
    if (((rel & 1) != 0) != (u >= SU_S)) return -1;
    if (u < SU_S) { const int sg = u * 3 + (rel >> 1) - 1, rho = sg % 7; return (sg / 7) * 19 + (rho < 2 ? rho : rho + 8); }
    const int idx = u - SU_S, rho = idx % 3, jj = rel >> 1;
    return (idx / 3) * 19 + (rho < 2 ? 2 + 2 * jj + rho : 15 + jj);
}

#define STAMP(slot) do {} while (0)

__device__ __forceinline__ f32x4 dq_silu(f32x4 a, float s) {
    constexpr float L2E = 1.44269504088896341f;
    const f32x4 y = a * s, t = y * -L2E;
    f32x4 e;
#pragma unroll
    for (int c = 0; c < 4; ++c) e[c] = __builtin_amdgcn_exp2f(__builtin_amdgcn_fmed3f(t[c], -60.0f * L2E, 60.0f * L2E));
    e += 1.0f;
#pragma unroll
    for (int c = 0; c < 4; ++c) e[c] = __builtin_amdgcn_rcpf(e[c]);
    return y * e;
}
__device__ __forceinline__ unsigned lds_add(LAS unsigned* p, unsigned v) {
    unsigned old;
    asm volatile("ds_add_rtn_u32 %0, %1, %2\n\ts_waitcnt lgkmcnt(0)" : "=v"(old) : "v"((unsigned)(size_t)p), "v"(v) : "memory");
    return old;
}

template <int V> struct IC { static constexpr int value = V; };
constexpr int eng_extra(int s) { return ((s == 4 + WIN || s == 8 + WIN || s == WIN - 4 || s == WIN - 1) ? 4 : 0) + (s == 0 ? 1 : 0) + (s == 14 - WIN ? 1 : 0); }
constexpr int eng_wait(int o) { int n = WIN - 1; for (int s = o - WIN; s < o; ++s) n += eng_extra((s + 16) % 16); return n; }
static_assert(WIN >= 4 && WIN <= 7, "engine schedule");

__device__ __forceinline__ void engine(const Params& p, int r, char* myws, char* callp, LAS char* lds, int wave, int lane) {
    const int link = blockIdx.x % 7, k = (blockIdx.x / 7) * EPW + wave;
    if (wave >= EPW || k >= EW) return;
    const int gw = k * 7 + link;
    const int delta = link + 1, d = (r ^ delta) & 7;
    char* dws = p.ws[delta];
    const int nitems = J_OUT + ((delta & 1) ? 24 * SU_O : 32 * SU_S);
    const unsigned lin = lane * 16; (void)gw;
    const char* claimp = myws + OFF_CNT + link * 128;
    auto srd = [](const void* base, int bytes) { return __builtin_amdgcn_make_buffer_rsrc((void*)base, 0, bytes, 0x00020000); };
    auto ld = [&](u32x4& v, int c, __amdgpu_buffer_rsrc_t src, unsigned skip) {
        asm volatile("buffer_load_dwordx4 %0, %1, %2, 0 offen offset:%3 sc0 sc1" : "=&v"(v) : "v"(lin + (c >> 2) * 4096 + skip), "s"(src), "i"((c & 3) * 1024) : "memory");
    };
    auto st = [&](__amdgpu_buffer_rsrc_t dst, unsigned off, u32x4 v) {
        asm volatile("buffer_store_dwordx4 %0, %1, %2, 0 offen sc0 sc1\n\ts_nop 1" :: "v"(v), "v"(off), "s"(dst) : "memory");
    };
#define PIN4(a, b) asm volatile("" : "+v"(a[4 * (b)]), "+v"(a[4 * (b) + 1]), "+v"(a[4 * (b) + 2]), "+v"(a[4 * (b) + 3]) :: "memory")
    u32x4 R[16];
    unsigned long long win;
    int j = k;
    float scale;
    unsigned call;
    {
        u32x4 Q[16];
        unsigned callv; float sxv, swv;
        const __amdgpu_buffer_rsrc_t sa = srd(p.x + (size_t)unit_of(d, k) * ITEM, ITEM), sb = srd(p.x + (size_t)unit_of(d, k + EW) * ITEM, ITEM);
        asm volatile("global_load_dword %0, %3, %4 sc1\n\tglobal_load_dword %1, %3, %5\n\tglobal_load_dword %2, %3, %6"
                     : "=&v"(callv), "=&v"(sxv), "=&v"(swv) : "v"(0u), "s"(callp), "s"(p.sx), "s"(p.sw) : "memory");
#pragma unroll
        for (int c = 0; c < 16; ++c) ld(Q[c], c, sa, 0);
#pragma unroll
        for (int c = 0; c < 16; ++c) ld(R[c], c, sb, 0);
        WAIT_VM(16);
        asm volatile("" : "+v"(callv), "+v"(sxv), "+v"(swv) :: "memory");
        PIN4(Q, 0); PIN4(Q, 1); PIN4(Q, 2); PIN4(Q, 3);
        call = __builtin_amdgcn_readfirstlane(callv) + 1;
        scale = sxv * swv;
        STAMP(0);
        const char* dst = dws + OFF_XS + (size_t)(call & 1) * XS_BYTES + ((size_t)delta * NSU + k) * ITEM;
#pragma unroll
        for (int c = 0; c < 32; ++c) {
            if (c == 16) { PIN4(R, 0); PIN4(R, 1); PIN4(R, 2); PIN4(R, 3); }
            st16_wt(dst, lin + (c & 15) * 1024 + (c >> 4) * (EW * ITEM), c < 16 ? Q[c & 15] : R[c & 15]);
            if (c >= PWIN - 1) WAIT_VM(PWIN - 1);
            if (c == 16 + PWIN - 2 && lane == 0) st_sys(dws + OFF_XFLAG + k * 128 + delta * 4, call);
            if (c == 32 - PWIN) asm volatile("global_load_dwordx2 %0, %1, %2 sc0 sc1" : "=&v"(win)
                                      : "v"((unsigned)(j + lane) * 8u), "s"(myws + OFF_RQ + ((size_t)(call & 1) * NR + delta) * (RQN * 8)) : "memory");
        }
        asm volatile("" : "+v"(win) :: "memory");
    }
    STAMP(1);
    const size_t par = call & 1;
    const char* rq = myws + OFF_RQ + (par * NR + delta) * (RQN * 8);
    const char* send = dws + OFF_SEND + par * SEND_BYTES;
    const unsigned cbase = (call - 1) * (unsigned)nitems;
    LAS unsigned* bm = (LAS unsigned*)(lds + wave * 4096);
    bool dead = false;
    int wb = j;
    int kind = 0; unsigned dsc = 0; const char* dst = myws; char* flag = dws + OFF_XFLAG + (k + EW) * 128 + delta * 4;
    bool busy = true;
    u32x4 aux = {0, 0, 0, 0};
    for (;;) {
        const bool has = j < nitems;
        if (!has && !busy) break;
        unsigned desc = 0;
        bool now = has;
        if (has) {
            if ((unsigned)(j - wb) < 64u && (unsigned)__builtin_amdgcn_readlane((unsigned)(win >> 32), j - wb) == call) desc = __builtin_amdgcn_readlane((unsigned)win, j - wb);
            else if (busy) now = false;
            else if (!dead) {
                const unsigned long long t0 = __builtin_amdgcn_s_memrealtime(); unsigned polls = 0;
                for (;;) {
                    const unsigned long long e = ld8_sys(rq + j * 8);
                    desc = __builtin_amdgcn_readfirstlane((unsigned)e);
                    if ((unsigned)__builtin_amdgcn_readfirstlane((unsigned)(e >> 32)) == call) break;
                    if (give_up(myws, call, t0, polls)) { dead = true; break; }
                    __builtin_amdgcn_s_sleep(4);
                }
            }
        }
        const int pkind = kind; const unsigned pdsc = dsc; const char* pdst = dst; char* pflag = flag;
        const bool raw = (desc & D_RAW) != 0, is_out = j >= J_OUT;
        const char* src = send;
        flag = nullptr;
        if (is_out) {
            if (has) src = send + SBW_BYTES + SBX_BYTES + ((size_t)link * NOUT_MAX + (j - J_OUT)) * ITEM;
            dst = (const char*)(p.out + (size_t)(desc & 15) * 32 * N_ + ((desc >> 4) & 127) * 128);
        } else if (j < J_X || (j >= J_W1 && j < J_XO)) {
            const int q = j < J_X ? j : j - J_W1 + 32, t = q >> 5, kc = (q >> 3) & 3, m = (q >> 1) & 3, G = q & 1, tc = (m * 4 + t) * 2 + G;
            src = send + (size_t)(((col_block(r, m) * 4 + t) * 2 + G) * 4 + kc) * ITEM;
            dst = myws + OFF_WS + ((size_t)delta * 32 + tc) * 65536 + kc * ITEM;
            flag = myws + OFF_WFLAG + tc * 128 + (kc * 8 + delta) * 4;
        } else {
            const int su = j < J_W1 ? j - J_X + NPUSH : j - J_XO + NSU_REMOTE;
            src = send + SBW_BYTES + (size_t)unit_of(r, su) * ITEM;
            dst = myws + OFF_XS + par * XS_BYTES + ((size_t)delta * NSU + su) * ITEM;
            flag = myws + OFF_XFLAG + su * 128 + delta * 4;
        }
        kind = (!now || dead) ? 0 : (raw ? 1 : 3) + (is_out ? 1 : 0);
        if (kind == 0) flag = nullptr;
        dsc = desc;
        const unsigned nexc = min((desc >> 20) & 1023u, (unsigned)EXC_MAX);
        const __amdgpu_buffer_rsrc_t ssrc = srd(src, kind == 0 ? 0 : kind >= 3 ? 15 * 1024 + ((nexc + 7) >> 3) * 16 : ITEM);
        const unsigned skip3 = kind == 4 ? 2u * ITEM : 0u, skip7 = kind >= 3 ? 2u * ITEM : 0u;
        u32x4 last = {0, 0, 0, 0};
        auto quarter = [&](int b, int kd, const char* base, unsigned ds) __attribute__((always_inline)) {
            const __amdgpu_buffer_rsrc_t sd = srd(base, kd == 0 ? 0 : (kd & 1) ? ITEM : 31 * N_ * 4 + 512);
            if (kd == 4) {
                if (b == 0) {
#pragma unroll
                    for (int q = 0; q < 4; ++q) *(LAS u32x4*)(bm + q * 256 + lane * 4) = (u32x4){0, 0, 0, 0};
                    if ((unsigned)lane * 8u < nexc) {
#pragma unroll
                        for (int e = 0; e < 8; ++e) {
                            const unsigned v = aux[e >> 1] >> (16 * (e & 1));
                            __hip_atomic_fetch_or(bm + ((v >> 6) & 1023u), (((v << 4) & 0xe0u) | (v & 1u)) << ((v >> 1) & 24u), __ATOMIC_RELAXED, __HIP_MEMORY_SCOPE_WAVEFRONT);
                        }
                    }
                    asm volatile("" ::: "memory");
                }
                const u32x4 L = *(LAS u32x4*)(bm + b * 256 + lane * 4);
                const float s2 = __builtin_amdgcn_ldexpf(scale, (int)((ds >> 12) & 255u) - 127 - 31);
                u32x4 y[4]; f32x4 v[4];
#pragma unroll
                for (int e = 0; e < 4; ++e) {
                    const unsigned d0 = R[4 * b + ((3 * e) >> 2)][(3 * e) & 3], d1 = R[4 * b + ((3 * e + 1) >> 2)][(3 * e + 1) & 3], d2 = R[4 * b + ((3 * e + 2) >> 2)][(3 * e + 2) & 3];
                    const unsigned T = L[e] & 0xe0e0e0e0u;
                    y[e] = (u32x4){__builtin_amdgcn_perm(T, d0, 0x03020104u), __builtin_amdgcn_perm(T, d1, 0x03020105u), __builtin_amdgcn_perm(T, d2, 0x03020106u),
                                   __builtin_amdgcn_perm(__builtin_amdgcn_perm(d2, d1, 0x04000400u), __builtin_amdgcn_perm(T, d0, 0x00070007u), 0x05040100u)};
#pragma unroll
                    for (int c = 0; c < 4; ++c) v[e][c] = (float)(int)y[e][c];
                }
                if (ds & D_WIDE) {
#pragma unroll
                    for (int e = 0; e < 4; ++e)
#pragma unroll
                        for (int c = 0; c < 4; ++c)
                            v[e][c] += __uint_as_float((0xcf800000u ^ (y[e][c] & 0x80000000u)) & (unsigned)__builtin_amdgcn_sbfe((int)L[e], 8 * c, 1));
                }
#pragma unroll
                for (int e = 0; e < 4; ++e) v[e] = dq_silu(v[e], s2);
#pragma unroll
                for (int e = 0; e < 4; ++e) st(sd, (e + 8 * b + 4 * (lane >> 5)) * (unsigned)(N_ * 4) + (lane & 31) * 16, __builtin_bit_cast(u32x4, v[e]));
            } else if (kd == 3) {
                if (b == 0) {
                    *(LAS u32x4*)(bm + lane * 8) = (u32x4){0, 0, 0, 0}; *(LAS u32x4*)(bm + lane * 8 + 4) = (u32x4){0, 0, 0, 0};
                    if ((unsigned)lane * 8u < nexc) {
#pragma unroll
                        for (int e = 0; e < 8; ++e) {
                            const unsigned v = aux[e >> 1] >> (16 * (e & 1));
                            __hip_atomic_fetch_or(bm + ((v >> 5) & 511u), ((v >> 15) & 1u) << (v & 31), __ATOMIC_RELAXED, __HIP_MEMORY_SCOPE_WAVEFRONT);
                        }
                    }
                    asm volatile("" ::: "memory");
                }
                const int pp = b >> 1;
                const u32x4 M = *(LAS u32x4*)(bm + lane * 8 + pp * 4);
                if ((b & 1) == 0) last = (u32x4){0, 0, 0, 0};
#pragma unroll
                for (int q = 4 * (b & 1); q < 4 * (b & 1) + 4; ++q) {
                    u32x4 o;
#pragma unroll
                    for (int c = 0; c < 4; ++c) {
                        unsigned v;
                        if (q < 7) { v = R[pp * 8 + q][c]; const int pl = q < 6 ? q : 7; last[c] |= shl(v, pl - 6) & (0x01010101u << pl); v &= 0xbfbfbfbfu; }
                        else v = last[c];
                        o[c] = (v + BIAS4) ^ (shl(M[c], 6 - q) & 0x40404040u);
                    }
                    st(sd, (lane >> 5) * 4096 + (lane & 31) * 16 + pp * 8192 + q * 512, o);
                }
            } else {
                const unsigned lo = kd == 2 ? (lane >> 5) * (unsigned)(N_ * 4) + (lane & 31) * 16 : lin, step = kd == 2 ? 2u * N_ * 4u : 1024u;
#pragma unroll
                for (int q = 4 * b; q < 4 * b + 4; ++q) st(sd, lo + q * step, q == 15 ? aux : R[q]);
            }
        };
        unsigned cl; unsigned long long keep;
        auto slot = [&](auto oc) __attribute__((always_inline)) {
            constexpr int o = decltype(oc)::value;
            WAIT_VM(eng_wait(o));
            if constexpr (o == WIN) { asm volatile("" : "+v"(R[15]) :: "memory"); aux = R[15]; asm volatile("" : "+v"(aux)); }
            if constexpr (o == 2 * WIN) {
                if (!now) WAIT_VM(0);
                if (pflag != nullptr && lane == 0) st_sys(pflag, call);
            }
            constexpr int c = (o + 15) & 15;
            ld(R[c], c, ssrc, c == 7 ? skip7 : (c == 3 || c == 11) ? skip3 : 0u);
            if constexpr (o == 0)
                asm volatile("s_mov_b64 %1, exec\n\ts_mov_b64 exec, 1\n\tglobal_atomic_add %0, %2, %3, %4 sc0\n\ts_mov_b64 exec, %1"
                             : "=&v"(cl), "=&s"(keep) : "v"(0u), "v"(now ? 1u : 0u), "s"(claimp) : "memory");
            if constexpr (o == 14 - WIN) {
                wb = j;
                asm volatile("global_load_dwordx2 %0, %1, %2 sc0 sc1" : "=&v"(win) : "v"((unsigned)((wb + lane) & (RQN - 1)) * 8u), "s"(rq) : "memory");
            }
            if constexpr (o == WIN - 4) { PIN4(R, 2); quarter(2, pkind, pdst, pdsc); }
            if constexpr (o == WIN - 1) { asm volatile("" : "+v"(R[12]), "+v"(R[13]), "+v"(R[14]) :: "memory"); quarter(3, pkind, pdst, pdsc); }
            if constexpr (o == WIN + 4) { PIN4(R, 0); quarter(0, kind, dst, dsc); }
            if constexpr (o == WIN + 8) { PIN4(R, 1); quarter(1, kind, dst, dsc); }
        };
        slot(IC<0>{}); slot(IC<1>{}); slot(IC<2>{}); slot(IC<3>{}); slot(IC<4>{}); slot(IC<5>{}); slot(IC<6>{}); slot(IC<7>{});
        slot(IC<8>{}); slot(IC<9>{}); slot(IC<10>{}); slot(IC<11>{}); slot(IC<12>{}); slot(IC<13>{}); slot(IC<14>{}); slot(IC<15>{});
        asm volatile("" : "+v"(cl), "+v"(win) :: "memory");
        busy = kind != 0;
        if (!now) continue;
        const unsigned c = __builtin_amdgcn_readfirstlane(cl) - cbase;
        j = c < (unsigned)nitems ? EW + (int)c : nitems;
    }
    WAIT_VM(0);
    st_agent(callp, call);
    if (dead && lane == 0) p.out[(size_t)col_block(r, 0) * 1024] = __builtin_nanf("");
    STAMP(4);
}

__device__ __forceinline__ void pack_operands(const Params& p, int r, char* myws, unsigned callv, LAS char* lds, int wave, int lane, int task) {
    const int pair = (task < 32 ? task : task < 64 ? task + 96 : task - 32) * 2 + (wave >> 2), pr = wave & 3, G = lane >> 5, i = lane & 31;
    LAS unsigned* cnt = (LAS unsigned*)lds;
    LAS unsigned short* list = (LAS unsigned short*)(lds + 64);
    if (threadIdx.x < 4) cnt[threadIdx.x] = 0;
    ((LAS unsigned long long*)list)[threadIdx.x] = 0;
    u32x4 o[8];
    const bool is_w = pair < 256;
    const int strip = ((pair >> 2) & 15) * 4 + ((pair >> 6) & 3), kc = pair & 3;
    if (is_w) {
        const unsigned* src = (const unsigned*)(p.w + (size_t)(kc * 128 + pr * 32) * N_ + strip * 256) + lane;
        unsigned v[32];
#pragma unroll
        for (int j = 0; j < 32; ++j) v[j] = __builtin_nontemporal_load(src + (size_t)j * (N_ / 4));
#pragma unroll
        for (int q = 0; q < 8; ++q) {
            const unsigned t0 = __builtin_amdgcn_perm(v[4 * q + 1], v[4 * q], 0x05010400u), t1 = __builtin_amdgcn_perm(v[4 * q + 1], v[4 * q], 0x07030602u);
            const unsigned t2 = __builtin_amdgcn_perm(v[4 * q + 3], v[4 * q + 2], 0x05010400u), t3 = __builtin_amdgcn_perm(v[4 * q + 3], v[4 * q + 2], 0x07030602u);
            o[(q & 4) + 0][q & 3] = __builtin_amdgcn_perm(t2, t0, 0x05040100u); o[(q & 4) + 1][q & 3] = __builtin_amdgcn_perm(t2, t0, 0x07060302u);
            o[(q & 4) + 2][q & 3] = __builtin_amdgcn_perm(t3, t1, 0x05040100u); o[(q & 4) + 3][q & 3] = __builtin_amdgcn_perm(t3, t1, 0x07060302u);
        }
    } else {
        const char* src = (const char*)p.x + (size_t)((pair - 256) * 2 + G) * ITEM + pr * 4096 + i * 16;
#pragma unroll
        for (int q = 0; q < 8; ++q) o[q] = *(const u32x4*)(src + q * 512);
    }
    u32x4 pk[7], M = {0, 0, 0, 0};
#pragma unroll
    for (int c = 0; c < 4; ++c) {
        unsigned cd[8];
#pragma unroll
        for (int q = 0; q < 8; ++q) {
            cd[q] = bfi(0x7f7f7f7fu, (o[q][c] | 0x80808080u) - BIAS4, o[q][c]);
            M[c] |= shl(cd[q], q - 6) & (0x01010101u << q);
        }
#pragma unroll
        for (int q = 0; q < 7; ++q) pk[q][c] = bfi(0x40404040u, shl(cd[7], 6 - (q < 6 ? q : 7)), cd[q]);
    }
    __syncthreads();
    asm volatile("" : "+v"(callv));
    const unsigned call = __builtin_amdgcn_readfirstlane(callv) + 1;
    const size_t par = call & 1;
    const int it = (wave >> 2) * 2 + G;
    {
        const unsigned n = __builtin_popcount(M[0]) + __builtin_popcount(M[1]) + __builtin_popcount(M[2]) + __builtin_popcount(M[3]);
        unsigned pos = lds_add(cnt + it, n);
        if (pos + n <= (unsigned)EXC_MAX) {
            const unsigned id = 0x8000u | (((((pr & 1) * 32 + i) * 8) + (pr >> 1) * 4) << 5);
#pragma unroll
            for (int c = 0; c < 4; ++c)
                for (unsigned mm = M[c]; mm; mm &= mm - 1) list[it * EXC_MAX + pos++] = (unsigned short)(id + c * 32 + __builtin_ctz(mm));
        }
    }
    __syncthreads();
    const unsigned nexc = cnt[it];
    const bool rawv = nexc > (unsigned)EXC_MAX;
    const char* slot = myws + OFF_SEND + par * SEND_BYTES + (is_w ? (size_t)(strip * 8 + kc) * ITEM : SBW_BYTES + (size_t)(pair - 256) * 2 * ITEM);
    const unsigned ib = G * (is_w ? 4u * ITEM : (unsigned)ITEM) + i * 16;
#pragma unroll
    for (int q = 0; q < 7; ++q) st16_wt(slot, ib + (rawv ? pr * 4096 + q * 512 : ((pr >> 1) * 8 + q) * 1024 + (pr & 1) * 512), rawv ? o[q] : pk[q]);
    if (rawv || pr < 2) st16_wt(slot, ib + (rawv ? pr * 4096 + 7 * 512 : 15 * 1024 + pr * 512), rawv ? o[7] : *(LAS u32x4*)(list + it * EXC_MAX + (pr & 1) * 256 + i * 8));
    const int cb = strip >> 2, t = strip & 3, m = cb >> 2;
    const bool mine = is_w && cb == col_block(r, m);
    if (mine) {
        const char* dst = myws + OFF_WS + (size_t)((m * 4 + t) * 2) * 65536 + kc * ITEM + pr * 4096;
#pragma unroll
        for (int q = 0; q < 8; ++q) st16_wt(dst, G * 65536 + q * 512 + i * 16, o[q]);
    }
    WAIT_VM(0);
    __syncthreads();
    if (pr == 0 && i < NR) {
        const int d = r ^ i;
        int j = -1;
        if (is_w) { if (cb == col_block(d, m)) { const int q = t * 32 + kc * 8 + m * 2 + G; j = t == 0 ? q : q - 32 + J_W1; } }
        else { const int su = staged_of(d, (pair - 256) * 2 + G); if (su >= NPUSH) j = su < NSU_REMOTE ? su - NPUSH + J_X : su - NSU_REMOTE + J_XO; }
        if (j >= 0) {
            if (i != 0) st8_sys(p.ws[i] + OFF_RQ + ((par * NR + i) * RQN + j) * 8, ((unsigned long long)call << 32) | (rawv ? D_RAW : nexc << 20));
            else if (is_w) st_sys(myws + OFF_WFLAG + ((m * 4 + t) * 2 + G) * 128 + kc * 32, call);
        }
    }
}

__device__ __forceinline__ void gemm_tile(const Params& p, int r, char* myws, unsigned call, LAS char* lds, int wave, int lane,
                                          int t, int m, int G, int rb, float scale, int so) {
    (void)so;
    const size_t par = call & 1;
    int su;
    if (rb < 3) { const int j = (wave - 2) >> 1; su = rb * 19 + (wave < 2 ? wave : 2 + 2 * (j + (j >= m)) + (wave & 1)); }
    else if (rb < 6) { const int j = wave - 5; su = (rb - 3) * 19 + (wave < 5 ? 10 + wave : 15 + j + (j >= m)); }
    else su = NSU_REMOTE + (rb - 6) * 8 + wave;
    const int grow = unit_of(r, su) * 32;
    const int tc = (m * 4 + t) * 2 + G;
    const unsigned alane = (lane >> 3) * 512 + (((lane & 7) ^ (lane >> 4)) * 16);
    const char* astage = myws + OFF_XS + par * XS_BYTES + (size_t)su * ITEM;
    const char* aown = (const char*)p.x + (size_t)grow * KS;
    const char* bbase = myws + OFF_WS + (size_t)tc * 65536 + wave * 2048 + lane * 16;
    const char* flagp = lane < 32 ? myws + OFF_WFLAG + tc * 128 + lane * 4 : myws + OFF_XFLAG + su * 128 + (lane & 7) * 4;
    unsigned ready = 0;
    bool dead = false;

    auto wait_step = [&](int n, bool lazy) {
        if (!((ready >> n) & 1) && !dead) {
            const unsigned long long t0 = __builtin_amdgcn_s_memrealtime(); unsigned polls = 0;
            for (;;) {
                const unsigned long long mk = __ballot(ld_sys(flagp) >= call);
                ready = (unsigned)mk & ((((unsigned)(mk >> 32) & 0xffu) | 1u) * 0x01010101u);
                if ((ready >> n) & 1) break;
                if (give_up(myws, call, t0, polls)) { dead = true; break; }
                if (lazy) __builtin_amdgcn_s_sleep(32); else __builtin_amdgcn_s_sleep(8);
            }
            __builtin_amdgcn_s_waitcnt(0xC07F);
        }
    };
    const char* ga; const char* gb;
    int skc = 0, sdl = 0;
    auto stage_addr = [&]() {
        wait_step(skc * 8 + sdl, false);
        ga = (sdl == 0 ? aown : astage + (size_t)sdl * (NSU * ITEM)) + skc * 128;
        gb = bbase + (size_t)sdl * (32 * 65536) + skc * ITEM;
        if (sdl == 0) { if (++skc == 4) { skc = 0; sdl = 1; } }
        else if (sdl < 7) ++sdl;
        else if (skc < 3) { ++skc; sdl = 1; }
    };
    auto stage_piece = [&](int q, int buf) {
        if (q < 4) __builtin_amdgcn_global_load_lds((const GAS unsigned*)(ga + q * 4096 + (alane ^ ((q & 1) * 64))),
                                                    (LAS unsigned*)(lds + buf * STAGE_BYTES + wave * 4096 + q * 1024), 16, 0, GAUX);
        else __builtin_amdgcn_global_load_lds((const GAS unsigned*)(gb + (q - 4) * 1024),
                                              (LAS unsigned*)(lds + buf * STAGE_BYTES + 32768 + wave * 2048 + (q - 4) * 1024), 16, 0, GAUX);
    };

    f32x16 acc[4];
#pragma unroll
    for (int c = 0; c < 4; ++c)
#pragma unroll
        for (int e = 0; e < 16; ++e) acc[c][e] = 0.f;

    const int aoff = wave * 4096 + (lane & 31) * 128 + (((lane >> 5) ^ ((lane >> 1) & 7)) * 16);
    const int boff = 32768 + (lane >> 5) * 2048 + (lane & 31) * 16;
    auto frag = [&](const LAS char* p0, const LAS char* p1) -> i32x8 {
        const u32x4 lo = *(const LAS u32x4*)p0, hi = *(const LAS u32x4*)p1;
        return (i32x8){(int)lo[0], (int)lo[1], (int)lo[2], (int)lo[3], (int)hi[0], (int)hi[1], (int)hi[2], (int)hi[3]};
    };

    i32x8 a0, b0[4], a1, b1[4];
    auto read_half = [&](int buf, int kk, i32x8& af, i32x8 (&bf)[4]) {
        const LAS char* s = lds + buf * STAGE_BYTES;
        af = frag(s + (aoff ^ (kk * 64)), s + (aoff ^ (kk * 64 + 32)));
#pragma unroll
        for (int c = 0; c < 4; ++c) bf[c] = frag(s + boff + kk * 8192 + c * 512, s + boff + kk * 8192 + 4096 + c * 512);
    };
#define WAIT_LDS() __builtin_amdgcn_s_waitcnt(0xC07F)

    __builtin_amdgcn_s_barrier();
#pragma unroll
    for (int o = 0; o < 3; ++o) {
        stage_addr();
#pragma unroll
        for (int q = 0; q < 6; ++q) stage_piece(q, o);
    }
    WAIT_VM(12);
    __builtin_amdgcn_s_barrier();
    asm volatile("" ::: "memory");
    read_half(0, 0, a0, b0);
    int buf = 0;
#define MFMA(c, a, b) acc[c] = __builtin_amdgcn_mfma_scale_f32_32x32x64_f8f6f4(a, b[c], acc[c], 0, 0, 0, 0, 0, 0)
#define FENCE() __builtin_amdgcn_sched_barrier(0)
    for (int n = 0; n < 32; ++n) {
        WAIT_LDS();
        MFMA(0, a0, b0); FENCE();
        read_half(buf, 1, a1, b1); FENCE();
        MFMA(1, a0, b0); MFMA(2, a0, b0); FENCE();
        if (n == 1) {
            if (wave == 0) wait_step(1, true);
            __builtin_amdgcn_s_barrier();
        }
        stage_addr(); FENCE();
        MFMA(3, a0, b0); FENCE();
        WAIT_LDS();
        MFMA(0, a1, b1);
        asm volatile("" : "+v"(acc[0])); FENCE();
        WAIT_VM(6);
        __builtin_amdgcn_s_barrier();
        asm volatile("" ::: "memory");
        const int nb = buf == 2 ? 0 : buf + 1;
        read_half(nb, 0, a0, b0); FENCE();
#pragma unroll
        for (int c = 1; c < 4; ++c) {
            MFMA(c, a1, b1);
            stage_piece(2 * c - 2, buf); stage_piece(2 * c - 1, buf); FENCE();
        }
        buf = nb;
    }
    STAMP(3 + so);

    const int delta = (grow >> 9) ^ r, ct = col_block(r, m) * 8 + t * 2 + G;
    unsigned tk = 0;
    if (delta != 0 && lane == 0) tk = __hip_atomic_fetch_add((GAS unsigned*)(myws + OFF_CNT + (8 + delta - 1) * 128), 1u, __ATOMIC_RELAXED, __HIP_MEMORY_SCOPE_AGENT);
    const unsigned tmov = ld_agent(myws + OFF_TMO);
    float sum = 0.f, mx = 0.f;
#pragma unroll
    for (int e = 0; e < 16; ++e) {
        sum += __builtin_fabsf(acc[e & 3][e]);
        mx = __builtin_fmaxf(__builtin_fmaxf(mx, __builtin_fmaxf(__builtin_fabsf(acc[0][e]), __builtin_fabsf(acc[1][e]))), __builtin_fmaxf(__builtin_fabsf(acc[2][e]), __builtin_fabsf(acc[3][e])));
    }
    sum += __uint_as_float(__builtin_amdgcn_update_dpp(0u, __float_as_uint(sum), 0x111, 0xf, 0xf, true));
    sum += __uint_as_float(__builtin_amdgcn_update_dpp(0u, __float_as_uint(sum), 0x112, 0xf, 0xf, true));
    sum += __uint_as_float(__builtin_amdgcn_update_dpp(0u, __float_as_uint(sum), 0x114, 0xf, 0xf, true));
    sum += __uint_as_float(__builtin_amdgcn_update_dpp(0u, __float_as_uint(sum), 0x118, 0xf, 0xf, true));
    const int tb = min(max((int)((unsigned)__builtin_amdgcn_readlane(__float_as_uint(sum * (3.13f / 256.f)), 15) >> 23) + 1, 26), 254);
    const bool wide = __any((int)(__float_as_uint(mx) >> 23) >= tb);
    const bool huge = __any((int)(__float_as_uint(mx) >> 23) > tb);
    asm volatile("" ::: "memory");
    const bool bad = __builtin_amdgcn_readfirstlane(tmov) == call;
    WAIT_VM(0);
    const int g = lane >> 5, i = lane & 31;
    unsigned desc = ((grow & 511) >> 5) | (ct << 4);
    const char* dst; unsigned pitch;
    int slot = 0;
    bool raw = true;
    if (delta == 0) { dst = (const char*)(p.out + (size_t)(grow & 511) * N_ + ct * 128); pitch = N_ * 4; }
    else {
        const int nout = (delta & 1) ? 24 * SU_O : 32 * SU_S;
        tk = __builtin_amdgcn_readfirstlane(tk) - (call - 1) * (unsigned)nout;
        slot = tk < (unsigned)nout ? (int)tk : nout - 1;
        dst = myws + OFF_SEND + par * SEND_BYTES + SBW_BYTES + SBX_BYTES + ((size_t)(delta - 1) * NOUT_MAX + slot) * ITEM; pitch = 512;
        raw = bad || huge;
        if (!raw) {
            const float up = __uint_as_float((unsigned)(280 - tb) << 23);
            unsigned P[4][2] = {}, bad_bits = 0;
#pragma unroll
            for (int b = 0; b < 4; ++b) {
                unsigned y[4][4], D[12];
#pragma unroll
                for (int c = 0; c < 4; ++c) {
                    const f32x4 z = (f32x4){acc[c][4 * b], acc[c][4 * b + 1], acc[c][4 * b + 2], acc[c][4 * b + 3]} * up;
#pragma unroll
                    for (int e = 0; e < 4; ++e) {
                        int q;
                        const float ze = z[e];
                        asm("v_cvt_i32_f32 %0, %1" : "=v"(q) : "v"(ze));
                        bad_bits |= __float_as_uint((float)q) ^ __float_as_uint(ze);
                        y[e][c] = (unsigned)q << 5;
                    }
                }
#pragma unroll
                for (int e = 0; e < 4; ++e) {
                    D[3 * e]     = __builtin_amdgcn_perm(y[e][3], y[e][0], 0x03020105u);
                    D[3 * e + 1] = __builtin_amdgcn_perm(y[e][3], y[e][1], 0x03020106u);
                    D[3 * e + 2] = __builtin_amdgcn_perm(y[e][3], y[e][2], 0x03020107u);
                    const unsigned L = __builtin_amdgcn_perm(__builtin_amdgcn_perm(y[e][3], y[e][2], 0x04000400u), __builtin_amdgcn_perm(y[e][1], y[e][0], 0x04000400u), 0x05040100u);
                    const int s = (4 * b + e) & 7;
#pragma unroll
                    for (int k = 0; k < 3; ++k) P[k][b >> 1] |= shl(L, s - (7 - k)) & (0x01010101u << s);
                }
#pragma unroll
                for (int q = 0; q < 3; ++q)
                    st16_wt(dst, (4 * b + q) * 1024 + lane * 16, (u32x4){D[4 * q], D[4 * q + 1], D[4 * q + 2], D[4 * q + 3]});
                asm volatile("" : "+v"(bad_bits), "+v"(P[0][b >> 1]), "+v"(P[1][b >> 1]), "+v"(P[2][b >> 1]));
            }
            if (wide) {
                asm volatile("" ::: "memory");
                const float lim = __uint_as_float((unsigned)tb << 23);
#pragma unroll
                for (int e = 0; e < 16; ++e)
#pragma unroll
                    for (int c = 0; c < 4; ++c) P[3][e >> 3] |= (acc[c][e] >= lim || acc[c][e] < -lim) ? 1u << (8 * c + (e & 7)) : 0u;
            }
            const unsigned nz[2] = {P[0][0] | P[1][0] | P[2][0] | P[3][0], P[0][1] | P[1][1] | P[2][1] | P[3][1]};
            LAS unsigned* cnt = (LAS unsigned*)(lds + wave * 4096);
            LAS unsigned short* list = (LAS unsigned short*)(lds + STAGE_BYTES + wave * 4096);
            *cnt = 0;
            *(LAS u32x4*)(list + lane * 8) = (u32x4){0, 0, 0, 0};
            unsigned pos = lds_add(cnt, __builtin_popcount(nz[0]) + __builtin_popcount(nz[1]));
            const unsigned nexc = __builtin_amdgcn_readfirstlane(*(volatile LAS unsigned*)cnt);
            const float s2 = __builtin_amdgcn_ldexpf(scale, tb - 158);
            const bool s2_ok = __builtin_amdgcn_ldexpf(s2, 158 - tb) == scale && (__float_as_uint(s2) & 0x7f800000u) != 0;
            if (!__any(bad_bits != 0 || !s2_ok) && nexc <= (unsigned)EXC_MAX) {
#pragma unroll
                for (int h = 0; h < 2; ++h)
                    for (unsigned mm = nz[h]; mm; mm &= mm - 1) {
                        const unsigned b = __builtin_ctz(mm);
                        const unsigned val = (((P[0][h] >> b) & 1u) << 3) | (((P[1][h] >> b) & 1u) << 2) | (((P[2][h] >> b) & 1u) << 1) | ((P[3][h] >> b) & 1u);
                        list[pos++] = (unsigned short)(((h * 2048 + (b & 4u) * 256 + lane * 16 + (b & 3u) * 4 + (b >> 3)) << 4) | val);
                    }
                asm volatile("" ::: "memory");
                st16_wt(dst, 15 * 1024 + lane * 16, *(LAS u32x4*)(list + lane * 8));
                desc |= ((unsigned)tb << 12) | (nexc << 20) | (wide ? D_WIDE : 0u);
            } else { raw = true; WAIT_VM(0); }
        }
    }
    if (raw) {
        const float nan = __builtin_nanf("");
#pragma unroll
        for (int e = 0; e < 16; ++e) {
            f32x4 v = dq_silu((f32x4){acc[0][e], acc[1][e], acc[2][e], acc[3][e]}, scale);
            if (bad) v = (f32x4){nan, nan, nan, nan};
            st16_wt(dst, ((e & 3) + 8 * (e >> 2) + 4 * g) * pitch + i * 16, __builtin_bit_cast(u32x4, v));
        }
        desc |= D_RAW;
    }
    if (delta != 0) {
        WAIT_VM(0);
        if (lane == 0) st8_sys(p.ws[delta] + OFF_RQ + ((par * NR + delta) * RQN + J_OUT + slot) * 8, ((unsigned long long)call << 32) | desc);
    }
    STAMP(4 + so);
}

__global__ void __launch_bounds__(512, 2) fused_gemm_rs(Params p) {
    extern __shared__ __attribute__((aligned(16))) char smem[];
    const int lane = threadIdx.x & 63;
    const int wave = __builtin_amdgcn_readfirstlane(threadIdx.x >> 6);
    const int r = __builtin_amdgcn_readfirstlane(p.rank) & 7;
    char* myws = p.ws[0];
    char* cp = myws + OFF_CALL + (blockIdx.x * 8 + wave) * 4;
    if (blockIdx.x < NENG) { engine(p, r, myws, cp, (LAS char*)smem, wave, lane); return; }
    const unsigned callv = ld_agent(cp);
    STAMP(0);
    for (int task = blockIdx.x - NENG; task < 160; task += NGEMM) pack_operands(p, r, myws, callv, (LAS char*)smem, wave, lane, task);
    const unsigned call = __builtin_amdgcn_readfirstlane(callv) + 1;
    st_agent(cp, call);
    float scale = p.sx[0] * p.sw[0];
    asm volatile("" : "+v"(scale));
    STAMP(1);
    const int e = blockIdx.x & 7, slot = (blockIdx.x - NENG) >> 3, m = e >> 1;
    for (int id = slot; id < 32; id += NGEMM / 8) {
        int t, G, rb;
        if (id < 24) { t = id / 6; const int q = id % 6; G = q & 1; rb = 3 * (e & 1) + (q >> 1); }
        else { const int q = id - 24; t = q >> 1; G = q & 1; rb = 6 + (e & 1); }
        gemm_tile(p, r, myws, call, (LAS char*)smem, wave, lane, t, m, G, rb, scale, id == slot ? 0 : 5);
    }
}

}

extern "C" void kernel_launch(int rank, int nranks, ncclComm_t comm,
                              hipStream_t stream,
                              void* const* d_in, const int* in_sizes, int n_in,
                              void* d_out, int out_size,
                              const int* in_dtypes, int out_dtype,
                              void* const* d_out_all,
                              void* const* d_ws_all, size_t ws_bytes) {
    (void)comm; (void)in_dtypes; (void)out_dtype; (void)d_out_all;
    if (nranks != NR || (unsigned)rank >= (unsigned)NR || n_in < 4 || ws_bytes < WS_TOTAL || out_size != 512 * N_ ||
        in_sizes[0] != 4096 * KS || in_sizes[1] != KS * N_) {
        fprintf(stderr, "kernel_launch: unexpected problem geometry\n");
        return;
    }
    static bool attr_set[NR] = {};
    if (!attr_set[rank]) {
        HIP_CHECK(hipFuncSetAttribute((const void*)fused_gemm_rs, hipFuncAttributeMaxDynamicSharedMemorySize, LDS_BYTES));
        attr_set[rank] = true;
    }
    Params p;
    p.x = (const unsigned char*)d_in[0]; p.w = (const unsigned char*)d_in[1];
    p.sx = (const float*)d_in[2]; p.sw = (const float*)d_in[3];
    for (int i = 0; i < NR; ++i) p.ws[i] = (char*)d_ws_all[rank ^ i];
    p.out = (float*)d_out;
    p.rank = rank; p.pad = 0;
    hipLaunchKernelGGL(fused_gemm_rs, dim3(NWG), dim3(512), LDS_BYTES, stream, p);
    HIP_CHECK(hipGetLastError());
}
```

```cpp
#include <hip/hip_runtime.h>
#include <rccl/rccl.h>
#include <cstdio>
#include <cstdint>

#define HIP_CHECK(expr) do { hipError_t _e = (expr); if (_e != hipSuccess) { \
    fprintf(stderr, "HIP error %s at %s:%d\n", hipGetErrorString(_e), __FILE__, __LINE__); \
    return; } } while (0)

namespace {

typedef unsigned u32x4 __attribute__((ext_vector_type(4)));
typedef int i32x8 __attribute__((ext_vector_type(8)));
typedef float f32x16 __attribute__((ext_vector_type(16)));
typedef float f32x4 __attribute__((ext_vector_type(4)));
#define LAS __attribute__((address_space(3)))
#define GAS __attribute__((address_space(1)))

constexpr int NR = 8;
constexpr int N_ = 16384, KS = 512;
#define NENG_ 112
#define EPW 1
constexpr int NWG = 256, NENG = NENG_, NGEMM = NWG - NENG;
#define EW 16
#define WIN 4
#define PWIN 5
#define GAUX 17
constexpr int SU_S = 7, SU_O = 9;
constexpr int NSU_REMOTE = 3 * SU_S + 4 * SU_O;
constexpr int NSU = NSU_REMOTE + 16;
constexpr int ITEM = 16384;
constexpr int NOUT_MAX = 4 * 8 * SU_S;
#define NPUSH_ 18
constexpr int NPUSH = NPUSH_;
#define PACE_X 0
#define PACE_W 300
#define PACE_WS 80
#define PACE_W0 0
#define PACE_W1 0
#define PACE_P0 1900
#define PACE_P1 220
#define JXP 0
#define JX0 16
#define NHALF 0
constexpr int E_X = 128, E_OUT = 208, RQN = 512;
constexpr int EXC_MAX = 512;
constexpr unsigned BIAS4 = 0x05050505u;

constexpr size_t OFF_CALL  = 0;
constexpr size_t OFF_TMO   = 16384;
constexpr size_t OFF_CNT   = 16384 + 128;
constexpr size_t OFF_LFLAG = 20480;
constexpr size_t OFF_WFLAG = 24576;
constexpr size_t OFF_XFLAG = 32768;
constexpr size_t OFF_RQ    = 65536;
constexpr size_t OFF_DBG   = 262144;
constexpr size_t SBW_BYTES = (size_t)512 * ITEM;
constexpr size_t SBX_BYTES = (size_t)128 * ITEM;
constexpr size_t OB_BYTES  = (size_t)7 * 32 * SU_O * ITEM;
constexpr size_t SEND_BYTES = SBW_BYTES + SBX_BYTES + OB_BYTES;
constexpr size_t XS_BYTES  = (size_t)NR * NSU * ITEM;
constexpr size_t WS_BYTES  = (size_t)NR * 32 * 65536;
constexpr size_t OFF_SEND  = 1u << 20;
constexpr size_t OFF_XS    = OFF_SEND + 2 * SEND_BYTES;
constexpr size_t OFF_WS    = OFF_XS + 2 * XS_BYTES;
constexpr size_t OFF_WL    = OFF_WS + WS_BYTES;
constexpr size_t WS_TOTAL  = OFF_WL + (size_t)128 * 65536;
static_assert(OFF_CNT + 17 * 128 <= OFF_LFLAG && OFF_LFLAG + 2048 <= OFF_WFLAG && OFF_XFLAG + NSU * 128 <= OFF_RQ && OFF_RQ + 2 * NR * RQN * 8 <= OFF_DBG, "flags overlap");
static_assert(EW <= (NENG / 7) * EPW && NENG % 8 == 0 && EPW <= 8 && NPUSH <= 2 * EW && NPUSH >= EW && EPW == 1 && NGEMM >= 128 && E_OUT + NOUT_MAX <= RQN, "geometry");
constexpr unsigned D_RAW = 1u << 11, D_WIDE = 1u << 30;
__device__ __forceinline__ size_t out_slot(int delta, unsigned desc) {
    const unsigned ru = desc & 15u, ct = (desc >> 4) & 127u;
    return SBW_BYTES + SBX_BYTES + (size_t)((((delta - 1) * 32 + (ct >> 5) * 8 + (ct & 7u)) * SU_O) + (ru >= SU_S ? ru - SU_S : ru)) * ITEM;
}

constexpr int STAGE_BYTES = 49152, NSTAGE = 3, LDS_BYTES = NSTAGE * STAGE_BYTES;
constexpr unsigned long long DEADLINE = 50000000ull;

struct Params {
    const unsigned char* x; const unsigned char* w; const float* sx; const float* sw;
    char* ws[NR];
    float* out;
    int rank; int pad;
};

__device__ __forceinline__ unsigned ld_sys(const void* p) { return __hip_atomic_load((const GAS unsigned*)p, __ATOMIC_RELAXED, __HIP_MEMORY_SCOPE_SYSTEM); }
__device__ __forceinline__ void st_sys(void* p, unsigned v) { __hip_atomic_store((GAS unsigned*)p, v, __ATOMIC_RELAXED, __HIP_MEMORY_SCOPE_SYSTEM); }
__device__ __forceinline__ unsigned ld_agent(const void* p) { return __hip_atomic_load((const GAS unsigned*)p, __ATOMIC_RELAXED, __HIP_MEMORY_SCOPE_AGENT); }
__device__ __forceinline__ void st_agent(void* p, unsigned v) { __hip_atomic_store((GAS unsigned*)p, v, __ATOMIC_RELAXED, __HIP_MEMORY_SCOPE_AGENT); }
__device__ __forceinline__ unsigned long long ld8_sys(const void* p) { return __hip_atomic_load((const GAS unsigned long long*)p, __ATOMIC_RELAXED, __HIP_MEMORY_SCOPE_SYSTEM); }
__device__ __forceinline__ void st8_sys(void* p, unsigned long long v) { __hip_atomic_store((GAS unsigned long long*)p, v, __ATOMIC_RELAXED, __HIP_MEMORY_SCOPE_SYSTEM); }
__device__ __forceinline__ void st16_wt(const void* base, unsigned off, u32x4 v) {
    asm volatile("s_nop 4\n\tglobal_store_dwordx4 %0, %1, %2 sc0 sc1\n\ts_nop 1" :: "v"(off), "v"(v), "s"(base) : "memory");
}
#define WAIT_VM(n) asm volatile("s_waitcnt vmcnt(%0)" :: "i"(n) : "memory")
__device__ __forceinline__ unsigned bfi(unsigned mask, unsigned a, unsigned b) { return (a & mask) | (b & ~mask); }
__device__ __forceinline__ unsigned shl(unsigned v, int s) { return s >= 0 ? v << s : v >> -s; }

__device__ __forceinline__ bool give_up(char* myws, unsigned call, unsigned long long t0, unsigned& polls) {
    bool dead = __builtin_amdgcn_s_memrealtime() - t0 > DEADLINE;
    if ((++polls & 63u) == 0u) dead |= __builtin_amdgcn_readfirstlane(ld_agent(myws + OFF_TMO)) == call;
    if (dead) st_agent(myws + OFF_TMO, call);
    return dead;
}

__device__ __forceinline__ int col_block(int r, int m) { return m * 4 + ((r & 1) ? ((r >> 1) ^ m) : (r >> 1)); }
__device__ __forceinline__ int unit_of(int r, int su) {
    if (su >= NSU_REMOTE) return r * 16 + su - NSU_REMOTE;
    const int g = su / 19, pos = su % 19;
    if (pos < 2 || (pos >= 10 && pos < 15)) { const int sg = g * 7 + (pos < 2 ? pos : pos - 8); return (r ^ (2 * (sg % 3 + 1))) * 16 + sg / 3; }
    const int jj = pos < 10 ? (pos - 2) >> 1 : pos - 15, idx = g * 3 + (pos < 10 ? (pos & 1) : 2);
    return (r ^ (2 * jj + 1)) * 16 + SU_S + idx;
}
__device__ __forceinline__ int staged_of(int r, int gu) {
    const int rel = (gu >> 4) ^ r, u = gu & 15;
    if (rel == 0) return NSU_REMOTE + u;
    if (((rel & 1) != 0) != (u >= SU_S)) return -1;
    if (u < SU_S) { const int sg = u * 3 + (rel >> 1) - 1, rho = sg % 7; return (sg / 7) * 19 + (rho < 2 ? rho : rho + 8); }
    const int idx = u - SU_S, rho = idx % 3, jj = rel >> 1;
    return (idx / 3) * 19 + (rho < 2 ? 2 + 2 * jj + rho : 15 + jj);
}

__device__ __forceinline__ int kept_su(int delta, int i) {
    if (delta & 1) return i;
    int su = i;
#pragma unroll
    for (int u = 0; u < SU_S; ++u) { const int sg = u * 3 + (delta >> 1) - 1, rho = sg % 7; su += ((sg / 7) * 19 + (rho < 2 ? rho : rho + 8)) <= su; }
    return su;
}

#define STAMP(slot) do {} while (0)

__device__ __forceinline__ f32x4 dq_silu(f32x4 a, float s) {
    constexpr float L2E = 1.44269504088896341f;
    const f32x4 y = a * s, t = y * -L2E;
    f32x4 e;
#pragma unroll
    for (int c = 0; c < 4; ++c) e[c] = __builtin_amdgcn_exp2f(__builtin_amdgcn_fmed3f(t[c], -60.0f * L2E, 60.0f * L2E));
    e += 1.0f;
#pragma unroll
    for (int c = 0; c < 4; ++c) e[c] = __builtin_amdgcn_rcpf(e[c]);
    return y * e;
}
__device__ __forceinline__ unsigned lds_add(LAS unsigned* p, unsigned v) {
    unsigned old;
    asm volatile("ds_add_rtn_u32 %0, %1, %2\n\ts_waitcnt lgkmcnt(0)" : "=v"(old) : "v"((unsigned)(size_t)p), "v"(v) : "memory");
    return old;
}

#define HOTNOP ""
template <int V> struct IC { static constexpr int value = V; };
constexpr int eng_extra(int s) {
    return ((s == 4 + WIN || s == 8 + WIN || s == WIN - 4 || s == WIN - 1) ? 4 : 0) + ((s == 3 || s == 7 || s == 11 || s == 14) ? 4 : 0) +
           (s == 0 ? 1 : 0) + (s == 14 - WIN ? 1 : 0);
}
constexpr int eng_wait(int o) { int n = WIN - 1; for (int s = o - WIN; s < o; ++s) n += eng_extra((s + 16) % 16); return n; }
static_assert(WIN >= 4 && WIN <= 6, "engine schedule");

__device__ __forceinline__ void engine(const Params& p, int r, char* myws, char* callp, LAS char* lds, int wave, int lane) {
    const int link = blockIdx.x % 7, k = blockIdx.x / 7;
    if (wave >= 2 || k >= EW) return;
    const int gw = k * 7 + link;
    const int delta = link + 1, d = (r ^ delta) & 7;
    char* dws = p.ws[delta];
    const bool same = (delta & 1) == 0;
    const int jW1 = JX0 + (same ? NSU_REMOTE - SU_S : NSU_REMOTE) - NPUSH, jXO = jW1 + 128 - JX0, jOUT = jXO + 16;
    const int jH = jOUT + (same ? 32 * SU_S : 24 * SU_O) - NHALF, nitems = jH + 2 * NHALF;
    auto entry_of = [&](int j) {
        if (j >= jH) return j < nitems ? (E_OUT + jH - jOUT + ((j - jH) >> 1)) | ((1 + ((j - jH) & 1)) << 16) : -1;
        if (j >= jOUT) return E_OUT + j - jOUT;
        if (j >= jXO) return E_X + NSU_REMOTE + j - jXO;
        if (j >= jW1) return j - jW1 + JX0;
        return (j >= JXP && j < JXP + JX0) ? j - JXP : E_X + kept_su(delta, (j < JXP ? j : j - JX0) + NPUSH);
    };
    const unsigned lin = lane * 16; (void)gw;
    const unsigned rowl = (lane >> 5) * (unsigned)(4 * N_ * 4) + (lane & 31) * 16;
    const char* claimp = myws + OFF_CNT + link * 128;
    auto srd = [](const void* base, int bytes) { return __builtin_amdgcn_make_buffer_rsrc((void*)base, 0, bytes, 0x00020000); };
    auto ld = [&](u32x4& v, int c, __amdgpu_buffer_rsrc_t src, unsigned skip) {
        asm volatile(HOTNOP "buffer_load_dwordx4 %0, %1, %2, 0 offen offset:%3 sc0 sc1" : "=&v"(v) : "v"(lin + (c >> 2) * 4096 + skip), "s"(src), "i"((c & 3) * 1024) : "memory");
    };
    auto st = [&](__amdgpu_buffer_rsrc_t dst, unsigned off, u32x4 v) {
        asm volatile(HOTNOP "buffer_store_dwordx4 %0, %1, %2, 0 offen sc0 sc1\n\ts_nop 1" :: "v"(v), "v"(off), "s"(dst) : "memory");
    };
#define PIN4(a, b) asm volatile("" : "+v"(a[4 * (b)]), "+v"(a[4 * (b) + 1]), "+v"(a[4 * (b) + 2]), "+v"(a[4 * (b) + 3]) :: "memory")
    u32x4 R[16], P[16];
    unsigned long long win;
    int en = entry_of(k);
    unsigned call = 0;
    auto look = [&](int e) {
        const unsigned off = lane == 1 ? (unsigned)(OFF_CNT + 16 * 128) : (unsigned)(OFF_RQ + (((call & 1) * NR + delta) * RQN + (e & (RQN - 1))) * 8);
        asm volatile("s_nop 4\n\tglobal_load_dwordx2 %0, %1, %2 sc0 sc1" : "=&v"(win) : "v"(off), "s"(myws) : "memory");
    };
    if (wave == 1) {
        const int su0 = kept_su(delta, k), su1 = kept_su(delta, k + EW);
        const bool two = k + EW < NPUSH;
        unsigned callv;
        const __amdgpu_buffer_rsrc_t sa = srd(p.x + (size_t)unit_of(d, su0) * ITEM, ITEM), sb = srd(p.x + (size_t)unit_of(d, su1) * ITEM, two ? ITEM : 0);
        asm volatile("s_nop 4\n\tglobal_load_dword %0, %1, %2 sc1" : "=&v"(callv) : "v"(0u), "s"(callp) : "memory");
#pragma unroll
        for (int c = 0; c < 16; ++c) ld(P[c], c, sa, 0);
#pragma unroll
        for (int c = 0; c < 16; ++c) ld(R[c], c, sb, 0);
        WAIT_VM(24);
        asm volatile("" : "+v"(callv) :: "memory");
        PIN4(P, 0); PIN4(P, 1);
        call = __builtin_amdgcn_readfirstlane(callv) + 1;
        STAMP(0);
        const char* dst = dws + OFF_XS + (size_t)(call & 1) * XS_BYTES + (size_t)delta * NSU * ITEM;
#pragma unroll
        for (int c = 0; c < 16; ++c) {
            if (c == 8) { PIN4(P, 2); PIN4(P, 3); PIN4(R, 0); PIN4(R, 1); PIN4(R, 2); PIN4(R, 3); }
            st16_wt(dst, lin + c * 1024 + su0 * ITEM, P[c]);
            if (c >= PWIN - 1) WAIT_VM(PWIN - 1);
        }
        if (two) {
#pragma unroll
            for (int c = 0; c < 16; ++c) {
                st16_wt(dst, lin + c * 1024 + su1 * ITEM, R[c]);
                WAIT_VM(PWIN - 1);
                if (c == PWIN - 2 && lane == 0) st_sys(dws + OFF_XFLAG + su0 * 128 + delta * 4, call);
            }
        }
        WAIT_VM(0);
        if (lane == 0) st_sys(dws + OFF_XFLAG + (two ? su1 : su0) * 128 + delta * 4, call);
        st_agent(callp, call);
        STAMP(1);
        return;
    }
    static_assert(PWIN <= 8, "push window");
    call = __builtin_amdgcn_readfirstlane(ld_agent(callp)) + 1;
    float scale = p.sx[0] * p.sw[0];
    asm volatile("" : "+v"(scale));
    win = 0;
#pragma unroll
    for (int c = 0; c < 16; ++c) { R[c] = (u32x4){0, 0, 0, 0}; P[c] = (u32x4){0, 0, 0, 0}; }
    const size_t par = call & 1;
    const char* send = dws + OFF_SEND + par * SEND_BYTES;
    const unsigned cbase = (call - 1) * (unsigned)nitems;
    LAS unsigned* bm = (LAS unsigned*)(lds + wave * 4096);
    bool dead = false;
    int khi = 0; unsigned dsc = 0; const char* dst = myws; char* flag = nullptr;
    bool busy = false;
    u32x4 aux = {0, 0, 0, 0};
    for (;;) {
        const bool has = en >= 0;
        if (!has && !busy) break;
        const int e = en & 0xffff, half = en >> 16;
        const bool is_out = e >= E_OUT;
        auto there = [&]() { return (unsigned)__builtin_amdgcn_readfirstlane((unsigned)(win >> 32)) == call &&
                                    (!(same && is_out) || (unsigned)__builtin_amdgcn_readlane((unsigned)(win >> 32), 1) == call); };
        bool now = has;
        if (has && !there()) {
            if (busy) now = false;
            else if (!dead) {
                const unsigned long long t0 = __builtin_amdgcn_s_memrealtime(); unsigned polls = 0;
                for (;;) {
                    look(e);
                    asm volatile("s_waitcnt vmcnt(0)" : "+v"(win) :: "memory");
                    if (there()) break;
                    if (give_up(myws, call, t0, polls)) { dead = true; break; }
                    __builtin_amdgcn_s_sleep(2);
                }
            }
        }
        const unsigned desc = __builtin_amdgcn_readfirstlane((unsigned)win);
        const int pkind = khi; const unsigned pdsc = dsc; const char* pdst = dst; char* pflag = flag;
        const bool raw = (desc & D_RAW) != 0;
        const char* src;
        flag = nullptr;
        if (is_out) {
            src = send + out_slot(delta, desc);
            dst = (const char*)(p.out + (size_t)(desc & 15) * 32 * N_ + ((desc >> 4) & 127) * 128);
        } else if (e < E_X) {
            const int t = e >> 5, G = (e >> 4) & 1, kc = (e >> 2) & 3, m = e & 3, tc = (m * 4 + t) * 2 + G;
            src = send + (size_t)(((col_block(r, m) * 4 + t) * 2 + G) * 4 + kc) * ITEM;
            dst = myws + OFF_WS + ((size_t)delta * 32 + tc) * 65536 + kc * ITEM;
            flag = myws + OFF_WFLAG + tc * 128 + (kc * 8 + delta) * 4;
        } else {
            const int su = e - E_X;
            src = send + SBW_BYTES + (size_t)unit_of(r, su) * ITEM;
            dst = myws + OFF_XS + par * XS_BYTES + ((size_t)delta * NSU + su) * ITEM;
            flag = myws + OFF_XFLAG + su * 128 + delta * 4;
        }
        const int kind = (!now || dead) ? 0 : (raw ? 1 : 3) + (is_out ? 1 : 0);
        const int klo = half == 2 ? 0 : kind;
        khi = half == 1 ? 0 : kind;
        if (kind == 0) flag = nullptr;
        dsc = desc;
        const unsigned nexc = min((desc >> 20) & 1023u, (unsigned)EXC_MAX);
        const __amdgpu_buffer_rsrc_t ssrc = srd(src, kind == 0 ? 0 : kind >= 3 ? 15 * 1024 + ((nexc + 7) >> 3) * 16 : ITEM);
        constexpr unsigned SKIP = 2u * ITEM;
        const unsigned slo = klo ? 0u : SKIP, shi = khi ? 0u : SKIP;
        const unsigned s3 = klo == 4 ? SKIP : slo, s7 = klo >= 3 ? SKIP : slo, s11 = khi == 4 ? SKIP : shi;
        constexpr int OUT_SPAN = 31 * N_ * 4 + 512;
        auto own_part = [&](int b, int kd) __attribute__((always_inline)) {
            const __amdgpu_buffer_rsrc_t sp = srd(dst, (same && kd != 0 && !(kd & 1)) ? OUT_SPAN : 0);
#pragma unroll
            for (int q = 0; q < 4; ++q)
                asm volatile(HOTNOP "buffer_load_dwordx4 %0, %1, %2, 0 offen sc0 sc1" : "=&v"(P[4 * b + q]) : "v"(rowl + (q + 8 * b) * (unsigned)(N_ * 4)), "s"(sp) : "memory");
        };
        auto scatter = [&]() __attribute__((always_inline)) {
            if (kind == 4) {
#pragma unroll
                for (int q = 0; q < 4; ++q) *(LAS u32x4*)(bm + q * 256 + lane * 4) = (u32x4){0, 0, 0, 0};
                if ((unsigned)lane * 8u < nexc) {
#pragma unroll
                    for (int q = 0; q < 8; ++q) {
                        const unsigned v = aux[q >> 1] >> (16 * (q & 1));
                        __hip_atomic_fetch_or(bm + ((v >> 6) & 1023u), (((v << 4) & 0xe0u) | (v & 1u)) << ((v >> 1) & 24u), __ATOMIC_RELAXED, __HIP_MEMORY_SCOPE_WAVEFRONT);
                    }
                }
            } else if (kind == 3) {
                *(LAS u32x4*)(bm + lane * 8) = (u32x4){0, 0, 0, 0}; *(LAS u32x4*)(bm + lane * 8 + 4) = (u32x4){0, 0, 0, 0};
                if ((unsigned)lane * 8u < nexc) {
#pragma unroll
                    for (int q = 0; q < 8; ++q) {
                        const unsigned v = aux[q >> 1] >> (16 * (q & 1));
                        __hip_atomic_fetch_or(bm + ((v >> 5) & 511u), ((v >> 15) & 1u) << (v & 31), __ATOMIC_RELAXED, __HIP_MEMORY_SCOPE_WAVEFRONT);
                    }
                }
            }
            asm volatile("" ::: "memory");
        };
        u32x4 last = {0, 0, 0, 0};
        auto quarter = [&](int b, int kd, const char* base, unsigned ds) __attribute__((always_inline)) {
            const __amdgpu_buffer_rsrc_t sd = srd(base, kd == 0 ? 0 : (kd & 1) ? ITEM : OUT_SPAN);
            if (kd == 4 || kd == 2) {
                f32x4 v[4];
                float unit = 1.0f;
                if (kd == 4) {
                    const u32x4 L = *(LAS u32x4*)(bm + b * 256 + lane * 4);
                    unit = __uint_as_float((((ds >> 12) & 255u) - 31u) << 23);
                    u32x4 y[4];
#pragma unroll
                    for (int q = 0; q < 4; ++q) {
                        const unsigned d0 = R[4 * b + ((3 * q) >> 2)][(3 * q) & 3], d1 = R[4 * b + ((3 * q + 1) >> 2)][(3 * q + 1) & 3], d2 = R[4 * b + ((3 * q + 2) >> 2)][(3 * q + 2) & 3];
                        const unsigned T = L[q] & 0xe0e0e0e0u;
                        y[q] = (u32x4){__builtin_amdgcn_perm(T, d0, 0x03020104u), __builtin_amdgcn_perm(T, d1, 0x03020105u), __builtin_amdgcn_perm(T, d2, 0x03020106u),
                                       __builtin_amdgcn_perm(__builtin_amdgcn_perm(d2, d1, 0x04000400u), __builtin_amdgcn_perm(T, d0, 0x00070007u), 0x05040100u)};
#pragma unroll
                        for (int c = 0; c < 4; ++c) v[q][c] = (float)(int)y[q][c];
                    }
                    if (ds & D_WIDE) {
#pragma unroll
                        for (int q = 0; q < 4; ++q)
#pragma unroll
                            for (int c = 0; c < 4; ++c)
                                v[q][c] += __uint_as_float((0xcf800000u ^ (y[q][c] & 0x80000000u)) & (unsigned)__builtin_amdgcn_sbfe((int)L[q], 8 * c, 1));
                    }
                } else {
#pragma unroll
                    for (int q = 0; q < 4; ++q) v[q] = __builtin_bit_cast(f32x4, 4 * b + q == 15 ? aux : R[4 * b + q]);
                }
#pragma unroll
                for (int q = 0; q < 4; ++q)
                    v[q] = dq_silu(__builtin_elementwise_fma(v[q], (f32x4){unit, unit, unit, unit}, __builtin_bit_cast(f32x4, P[4 * b + q])), scale);
#pragma unroll
                for (int q = 0; q < 4; ++q) st(sd, rowl + (q + 8 * b) * (unsigned)(N_ * 4), __builtin_bit_cast(u32x4, v[q]));
            } else if (kd == 3) {
                const int pp = b >> 1;
                const u32x4 M = *(LAS u32x4*)(bm + lane * 8 + pp * 4);
                if ((b & 1) == 0) last = (u32x4){0, 0, 0, 0};
#pragma unroll
                for (int q = 4 * (b & 1); q < 4 * (b & 1) + 4; ++q) {
                    u32x4 o;
#pragma unroll
                    for (int c = 0; c < 4; ++c) {
                        unsigned v;
                        if (q < 7) { v = R[pp * 8 + q][c]; const int pl = q < 6 ? q : 7; last[c] |= shl(v, pl - 6) & (0x01010101u << pl); v &= 0xbfbfbfbfu; }
                        else v = last[c];
                        o[c] = (v + BIAS4) ^ (shl(M[c], 6 - q) & 0x40404040u);
                    }
                    st(sd, (lane >> 5) * 4096 + (lane & 31) * 16 + pp * 8192 + q * 512, o);
                }
            } else {
#pragma unroll
                for (int q = 4 * b; q < 4 * b + 4; ++q) st(sd, lin + q * 1024, q == 15 ? aux : R[q]);
            }
        };
        unsigned cl; unsigned long long keep;
        auto slot = [&](auto oc) __attribute__((always_inline)) {
            constexpr int o = decltype(oc)::value;
            WAIT_VM(eng_wait(o));
            if constexpr (o == WIN) { asm volatile("" : "+v"(R[15]) :: "memory"); aux = R[15]; asm volatile("" : "+v"(aux)); }
            if constexpr (o == 2 * WIN) {
                if (pflag != nullptr && lane == 0) st_sys(pflag, call);
            }
            constexpr int c = (o + 15) & 15;
            ld(R[c], c, ssrc, c == 15 ? 0u : c == 3 ? s3 : c == 7 ? s7 : c == 11 ? s11 : c < 8 ? slo : shi);
            if constexpr (o == 0)
                asm volatile("s_mov_b64 %1, exec\n\ts_mov_b64 exec, 1\n\ts_nop 2\n\tglobal_atomic_add %0, %2, %3, %4 sc0\n\ts_mov_b64 exec, %1"
                             : "=&v"(cl), "=&s"(keep) : "v"(0u), "v"(now ? 1u : 0u), "s"(claimp) : "memory");
            if constexpr (o == 14 - WIN) {
                asm volatile("" : "+v"(cl) :: "memory");
                const unsigned cn = __builtin_amdgcn_readfirstlane(cl) - cbase;
                en = cn < (unsigned)nitems ? entry_of(EW + (int)cn) : -1;
                look(en);
            }
            if constexpr (o == 3) own_part(0, klo);
            if constexpr (o == 7) own_part(1, klo);
            if constexpr (o == 11) own_part(2, khi);
            if constexpr (o == 14) own_part(3, khi);
            if constexpr (o == WIN - 4) { PIN4(R, 2); PIN4(P, 2); quarter(2, pkind, pdst, pdsc); }
            if constexpr (o == WIN - 1) { asm volatile("" : "+v"(R[12]), "+v"(R[13]), "+v"(R[14]) :: "memory"); PIN4(P, 3); quarter(3, pkind, pdst, pdsc); }
            if constexpr (o == WIN + 2) scatter();
            if constexpr (o == WIN + 4) { PIN4(R, 0); PIN4(P, 0); quarter(0, klo, dst, dsc); }
            if constexpr (o == WIN + 8) { PIN4(R, 1); PIN4(P, 1); quarter(1, klo, dst, dsc); }
        };
        slot(IC<0>{}); slot(IC<1>{}); slot(IC<2>{}); slot(IC<3>{});
        if constexpr (WIN > 4) slot(IC<4>{});
        if constexpr (WIN > 5) slot(IC<5>{});
        if (!now) {
            WAIT_VM(0);
            if (pflag != nullptr && lane == 0) st_sys(pflag, call);
            asm volatile("" : "+v"(cl), "+v"(R[15]), "+v"(R[0]), "+v"(R[1]), "+v"(R[2]), "+v"(R[3]), "+v"(R[4]) :: "memory");
            PIN4(P, 0);
            asm volatile("" : "=v"(R[8]), "=v"(R[9]), "=v"(R[10]), "=v"(R[11]), "=v"(R[12]), "=v"(R[13]), "=v"(R[14]));
            asm volatile("" : "=v"(P[8]), "=v"(P[9]), "=v"(P[10]), "=v"(P[11]), "=v"(P[12]), "=v"(P[13]), "=v"(P[14]), "=v"(P[15]));
            busy = false;
            continue;
        }
        if constexpr (WIN <= 4) slot(IC<4>{});
        if constexpr (WIN <= 5) slot(IC<5>{});
        slot(IC<6>{}); slot(IC<7>{}); slot(IC<8>{}); slot(IC<9>{}); slot(IC<10>{}); slot(IC<11>{}); slot(IC<12>{}); slot(IC<13>{}); slot(IC<14>{}); slot(IC<15>{});
        asm volatile("" : "+v"(win) :: "memory");
        busy = khi != 0;
    }
    WAIT_VM(0);
    st_agent(callp, call);
    if (dead && lane == 0) p.out[(size_t)col_block(r, 0) * 1024] = __builtin_nanf("");
    STAMP(4);
}

__device__ __forceinline__ void pack_operands(const Params& p, int r, char* myws, unsigned callv, LAS char* lds, int wave, int lane, int task, unsigned long long later) {
    const int pair = (task < 32 ? task : task < 64 ? task + 96 : task - 32) * 2 + (wave >> 2), pr = wave & 3, G = lane >> 5, i = lane & 31;
    LAS unsigned* cnt = (LAS unsigned*)lds;
    LAS unsigned short* list = (LAS unsigned short*)(lds + 64);
    if (threadIdx.x < 4) cnt[threadIdx.x] = 0;
    ((LAS unsigned long long*)list)[threadIdx.x] = 0;
    u32x4 o[8];
    const bool is_w = pair < 256;
    const int strip = ((pair >> 2) & 15) * 4 + ((pair >> 6) & 3), kc = pair & 3;
    if (is_w) {
        const __amdgpu_buffer_rsrc_t src = __builtin_amdgcn_make_buffer_rsrc((void*)(p.w + (size_t)(kc * 128 + pr * 32) * N_ + strip * 256), 0, 31 * N_ + 256, 0x00020000);
        unsigned v[32];
#pragma unroll
        for (int j = 0; j < 32; ++j) v[j] = __builtin_amdgcn_raw_buffer_load_b32(src, lane * 4, j * N_, 2);
        __builtin_amdgcn_sched_barrier(0);
#pragma unroll
        for (int q = 0; q < 8; ++q) {
            const unsigned t0 = __builtin_amdgcn_perm(v[4 * q + 1], v[4 * q], 0x05010400u), t1 = __builtin_amdgcn_perm(v[4 * q + 1], v[4 * q], 0x07030602u);
            const unsigned t2 = __builtin_amdgcn_perm(v[4 * q + 3], v[4 * q + 2], 0x05010400u), t3 = __builtin_amdgcn_perm(v[4 * q + 3], v[4 * q + 2], 0x07030602u);
            o[(q & 4) + 0][q & 3] = __builtin_amdgcn_perm(t2, t0, 0x05040100u); o[(q & 4) + 1][q & 3] = __builtin_amdgcn_perm(t2, t0, 0x07060302u);
            o[(q & 4) + 2][q & 3] = __builtin_amdgcn_perm(t3, t1, 0x05040100u); o[(q & 4) + 3][q & 3] = __builtin_amdgcn_perm(t3, t1, 0x07060302u);
        }
    } else {
        const char* src = (const char*)p.x + (size_t)((pair - 256) * 2 + G) * ITEM + pr * 4096 + i * 16;
#pragma unroll
        for (int q = 0; q < 8; ++q) o[q] = *(const u32x4*)(src + q * 512);
    }
    u32x4 pk[7], M = {0, 0, 0, 0};
#pragma unroll
    for (int c = 0; c < 4; ++c) {
        unsigned cd[8];
#pragma unroll
        for (int q = 0; q < 8; ++q) {
            cd[q] = bfi(0x7f7f7f7fu, (o[q][c] | 0x80808080u) - BIAS4, o[q][c]);
            M[c] |= shl(cd[q], q - 6) & (0x01010101u << q);
        }
#pragma unroll
        for (int q = 0; q < 7; ++q) pk[q][c] = bfi(0x40404040u, shl(cd[7], 6 - (q < 6 ? q : 7)), cd[q]);
    }
    __syncthreads();
    asm volatile("" : "+v"(callv));
    const unsigned call = __builtin_amdgcn_readfirstlane(callv) + 1;
    const size_t par = call & 1;
    const int it = (wave >> 2) * 2 + G;
    {
        const unsigned n = __builtin_popcount(M[0]) + __builtin_popcount(M[1]) + __builtin_popcount(M[2]) + __builtin_popcount(M[3]);
        unsigned pos = lds_add(cnt + it, n);
        if (pos + n <= (unsigned)EXC_MAX) {
            const unsigned id = 0x8000u | (((((pr & 1) * 32 + i) * 8) + (pr >> 1) * 4) << 5);
#pragma unroll
            for (int c = 0; c < 4; ++c)
                for (unsigned mm = M[c]; mm; mm &= mm - 1) list[it * EXC_MAX + pos++] = (unsigned short)(id + c * 32 + __builtin_ctz(mm));
        }
    }
    __syncthreads();
    const unsigned nexc = cnt[it];
    const bool rawv = nexc > (unsigned)EXC_MAX;
    const char* slot = myws + OFF_SEND + par * SEND_BYTES + (is_w ? (size_t)(strip * 8 + kc) * ITEM : SBW_BYTES + (size_t)(pair - 256) * 2 * ITEM);
    const unsigned ib = G * (is_w ? 4u * ITEM : (unsigned)ITEM) + i * 16;
#pragma unroll
    for (int q = 0; q < 7; ++q) st16_wt(slot, ib + (rawv ? pr * 4096 + q * 512 : ((pr >> 1) * 8 + q) * 1024 + (pr & 1) * 512), rawv ? o[q] : pk[q]);
    if (rawv || pr < 2) st16_wt(slot, ib + (rawv ? pr * 4096 + 7 * 512 : 15 * 1024 + pr * 512), rawv ? o[7] : *(LAS u32x4*)(list + it * EXC_MAX + (pr & 1) * 256 + i * 8));
    WAIT_VM(0);
    __syncthreads();
    if (pr == 0 && i != 0 && i < NR) {
        const int d = r ^ i, m = strip >> 4;
        const int e = is_w ? ((strip >> 2) == col_block(d, m) ? (strip & 3) * 32 + G * 16 + kc * 4 + m : -1) : staged_of(d, (pair - 256) * 2 + G);
        if (e >= 0) st8_sys(p.ws[i] + OFF_RQ + ((par * NR + i) * RQN + (is_w ? e : E_X + e)) * 8, ((unsigned long long)call << 32) | (rawv ? D_RAW : nexc << 20));
    }
    if (is_w) {
        if (task >= 32) while (__builtin_amdgcn_s_memrealtime() < later) __builtin_amdgcn_s_sleep(8);
        const char* dst = myws + OFF_WL + (size_t)strip * 2 * 65536 + kc * ITEM + pr * 4096;
#pragma unroll
        for (int q = 0; q < 8; ++q) st16_wt(dst, G * 65536 + q * 512 + i * 16, o[q]);
        WAIT_VM(0);
        __syncthreads();
        if (pr == 0 && i == 0) st_sys(myws + OFF_LFLAG + ((strip * 2 + G) * 4 + kc) * 4, call);
    }
}

__device__ __forceinline__ void gemm_tile(const Params& p, int r, char* myws, unsigned call, LAS char* lds, int wave, int lane,
                                          int ct, int rb, bool part, float scale, int so) {
    (void)so;
    const size_t par = call & 1;
    asm volatile("" : "+v"(lane));
    const int m = ct >> 5, tc = m * 8 + (ct & 7);
    auto su_of = [&](int u) {
        if (rb < 3) { const int j = (u - 2) >> 1; return rb * 19 + (u < 2 ? u : 2 + 2 * (j + (j >= m)) + (u & 1)); }
        if (rb < 6) { const int j = u - 5; return (rb - 3) * 19 + (u < 5 ? 10 + u : 15 + j + (j >= m)); }
        return NSU_REMOTE + (rb - 6) * 8 + u;
    };
    const int w4 = wave & 3;
    const int su0 = su_of(w4), su1 = su_of(w4 + 4), gu0 = unit_of(r, su0), gu1 = unit_of(r, su1);
    const int dl0 = (gu0 >> 4) ^ r, dl1 = (gu1 >> 4) ^ r;
    const int sk0 = (dl0 & 1) ? 0 : dl0, sk1 = (dl1 & 1) ? 0 : dl1;
    const int nsteps = part ? 4 : 32;
    f32x16 acc[4];
#define WAIT_LDS() __builtin_amdgcn_s_waitcnt(0xC07F)
#define FENCE() __builtin_amdgcn_sched_barrier(0)
    LAS char* xch = lds + 81920 + w4 * 16384 + lane * 16;
    __builtin_amdgcn_s_barrier();
    if (wave >= 4) {
        const unsigned alane = (lane >> 3) * 512 + (((lane & 7) ^ (lane >> 4)) * 16);
        const char* xs = myws + OFF_XS + par * XS_BYTES;
        const char* bbase = myws + OFF_WS + (size_t)tc * 65536 + w4 * 4096 + lane * 16;
        const char* bown = myws + OFF_WL + (size_t)ct * 65536 + w4 * 4096 + lane * 16;
        const char* flagp = lane >= 32 ? myws + OFF_XFLAG + ((lane & 8) ? su1 : su0) * 128 + (lane & 7) * 4
                          : (lane & 7) ? myws + OFF_WFLAG + tc * 128 + lane * 4 : myws + OFF_LFLAG + (ct * 4 + (lane >> 3)) * 4;
        unsigned ready = 0;
        bool dead = false;
        int skc = 0, sdl = 0;
        auto stage = [&](int buf) {
            const int n = skc * 8 + sdl;
            if (!((ready >> n) & 1) && !dead) {
                const unsigned long long t0 = __builtin_amdgcn_s_memrealtime(); unsigned polls = 0;
                for (;;) {
                    const unsigned long long mk = __ballot(ld_sys(flagp) >= call);
                    const unsigned xm = (((unsigned)(mk >> 32) | 1u | (1u << sk0)) & ((unsigned)(mk >> 40) | 1u | (1u << sk1))) & 0xffu;
                    ready = (unsigned)mk & (xm * 0x01010101u);
                    if ((ready >> n) & 1) break;
                    if (give_up(myws, call, t0, polls)) { dead = true; break; }
                    __builtin_amdgcn_s_sleep(8);
                }
            }
            const char* ga0 = (sdl == 0 ? (const char*)p.x + (size_t)gu0 * ITEM : xs + (size_t)(sdl * NSU + su0) * ITEM) + skc * 128;
            const char* ga1 = (sdl == 0 ? (const char*)p.x + (size_t)gu1 * ITEM : xs + (size_t)(sdl * NSU + su1) * ITEM) + skc * 128;
            const char* gb = (sdl == 0 ? bown : bbase + (size_t)sdl * (32 * 65536)) + skc * ITEM;
            LAS char* l = lds + buf * STAGE_BYTES + w4 * 4096;
#pragma unroll
            for (int q = 0; q < 4; ++q) {
                __builtin_amdgcn_global_load_lds((const GAS unsigned*)(gb + q * 1024), (LAS unsigned*)(l + 32768 + q * 1024), 16, 0, GAUX);
                __builtin_amdgcn_global_load_lds((const GAS unsigned*)(ga0 + q * 4096 + (alane ^ ((q & 1) * 64))), (LAS unsigned*)(l + q * 1024), 16, 0, GAUX);
                __builtin_amdgcn_global_load_lds((const GAS unsigned*)(ga1 + q * 4096 + (alane ^ ((q & 1) * 64))), (LAS unsigned*)(l + 16384 + q * 1024), 16, 0, GAUX);
            }
            if (sdl == 0) { if (skc < 3) ++skc; else if (!part) { skc = 0; sdl = 1; } }
            else if (sdl < 7) ++sdl;
            else if (skc < 3) { ++skc; sdl = 1; }
        };
        stage(0); stage(1); stage(2);
        WAIT_VM(24);
        __builtin_amdgcn_s_barrier();
        int buf = 0;
        for (int n = 0; n < nsteps; ++n) {
            WAIT_VM(12);
            __builtin_amdgcn_s_barrier();
            asm volatile("" ::: "memory");
            stage(buf);
            buf = buf == 2 ? 0 : buf + 1;
        }
        WAIT_VM(0);
        __builtin_amdgcn_s_barrier();
        __builtin_amdgcn_s_barrier();
        asm volatile("" ::: "memory");
#pragma unroll
        for (int e = 0; e < 16; ++e) {
            const f32x4 v = *(const LAS f32x4*)(xch + e * 1024);
#pragma unroll
            for (int c = 0; c < 4; ++c) acc[e >> 2][4 * (e & 3) + c] = v[c];
        }
    } else {
        f32x16 acd[4];
#pragma unroll
        for (int c = 0; c < 4; ++c)
#pragma unroll
            for (int e = 0; e < 16; ++e) { acc[c][e] = 0.f; acd[c][e] = 0.f; }
        const int aoff = w4 * 4096 + (lane & 31) * 128 + (((lane >> 5) ^ ((lane >> 1) & 7)) * 16);
        const int boff = 32768 + (lane >> 5) * 2048 + (lane & 31) * 16;
        auto frag = [&](const LAS char* p0, const LAS char* p1) -> i32x8 {
            const u32x4 lo = *(const LAS u32x4*)p0, hi = *(const LAS u32x4*)p1;
            return (i32x8){(int)lo[0], (int)lo[1], (int)lo[2], (int)lo[3], (int)hi[0], (int)hi[1], (int)hi[2], (int)hi[3]};
        };
        i32x8 a0[2], b0[4], a1[2], b1[4];
        auto read_half = [&](int buf, int kk, i32x8 (&af)[2], i32x8 (&bf)[4]) {
            const LAS char* s = lds + buf * STAGE_BYTES;
#pragma unroll
            for (int u = 0; u < 2; ++u) af[u] = frag(s + u * 16384 + (aoff ^ (kk * 64)), s + u * 16384 + (aoff ^ (kk * 64 + 32)));
#pragma unroll
            for (int c = 0; c < 4; ++c) bf[c] = frag(s + boff + kk * 8192 + c * 512, s + boff + kk * 8192 + 4096 + c * 512);
        };
#define MFMA(D, c, a, b) D[c] = __builtin_amdgcn_mfma_scale_f32_32x32x64_f8f6f4(a, b[c], D[c], 0, 0, 0, 0, 0, 0)
        const unsigned skn0 = sk0 ? 0x204081u << (3 + sk0) : 0u, skn1 = sk1 ? 0x204081u << (3 + sk1) : 0u;
        auto blank = [&](int n, i32x8 (&af)[2]) {
            if ((skn0 >> n) & 1) { asm volatile("" ::: "memory"); af[0] = (i32x8){0, 0, 0, 0, 0, 0, 0, 0}; }
            if ((skn1 >> n) & 1) { asm volatile("" ::: "memory"); af[1] = (i32x8){0, 0, 0, 0, 0, 0, 0, 0}; }
        };
        __builtin_amdgcn_s_barrier();
        asm volatile("" ::: "memory");
        read_half(0, 0, a0, b0);
        int buf = 0;
        for (int n = 0; n < nsteps; ++n) {
            WAIT_LDS();
            blank(n, a0);
            MFMA(acc, 0, a0[0], b0);
            asm volatile("" : "+v"(acc[0])); FENCE();
            read_half(buf, 1, a1, b1); FENCE();
            MFMA(acd, 0, a0[1], b0);
#pragma unroll
            for (int c = 1; c < 4; ++c) { MFMA(acc, c, a0[0], b0); MFMA(acd, c, a0[1], b0); }
            FENCE();
            WAIT_LDS();
            blank(n, a1);
            MFMA(acc, 0, a1[0], b1);
            asm volatile("" : "+v"(acc[0])); FENCE();
            __builtin_amdgcn_s_barrier();
            asm volatile("" ::: "memory");
            buf = buf == 2 ? 0 : buf + 1;
            read_half(buf, 0, a0, b0); FENCE();
            MFMA(acd, 0, a1[1], b1);
#pragma unroll
            for (int c = 1; c < 4; ++c) { MFMA(acc, c, a1[0], b1); MFMA(acd, c, a1[1], b1); }
            FENCE();
        }
        WAIT_LDS();
        __builtin_amdgcn_s_barrier();
        asm volatile("" ::: "memory");
#pragma unroll
        for (int e = 0; e < 16; ++e) *(LAS f32x4*)(xch + e * 1024) = (f32x4){acd[e >> 2][4 * (e & 3)], acd[e >> 2][4 * (e & 3) + 1], acd[e >> 2][4 * (e & 3) + 2], acd[e >> 2][4 * (e & 3) + 3]};
        WAIT_LDS();
        __builtin_amdgcn_s_barrier();
    }
    asm volatile("" : "+v"(lane));
    const int grow = (wave >= 4 ? gu1 : gu0) * 32, delta = wave >= 4 ? dl1 : dl0;
    STAMP(3 + so);

    const unsigned tmov = ld_agent(myws + OFF_TMO);
    asm volatile("s_nop 15\n\ts_nop 15" : "+v"(acc[0]), "+v"(acc[1]), "+v"(acc[2]), "+v"(acc[3]));
    float sum = 0.f, mx = 0.f;
#pragma unroll
    for (int e = 0; e < 16; ++e) {
        sum += __builtin_fabsf(acc[e & 3][e]);
        asm("v_max3_f32 %0, |%1|, |%2|, %0" : "+v"(mx) : "v"(acc[0][e]), "v"(acc[1][e]));
        asm("v_max3_f32 %0, |%1|, |%2|, %0" : "+v"(mx) : "v"(acc[2][e]), "v"(acc[3][e]));
    }
    sum += __uint_as_float(__builtin_amdgcn_update_dpp(0u, __float_as_uint(sum), 0x111, 0xf, 0xf, true));
    sum += __uint_as_float(__builtin_amdgcn_update_dpp(0u, __float_as_uint(sum), 0x112, 0xf, 0xf, true));
    sum += __uint_as_float(__builtin_amdgcn_update_dpp(0u, __float_as_uint(sum), 0x114, 0xf, 0xf, true));
    sum += __uint_as_float(__builtin_amdgcn_update_dpp(0u, __float_as_uint(sum), 0x118, 0xf, 0xf, true));
    const int tb = min(max((int)((unsigned)__builtin_amdgcn_readlane(__float_as_uint(sum * (3.13f / 256.f)), 15) >> 23) + 1, 32), 254);
    const bool wide = __any((int)(__float_as_uint(mx) >> 23) >= tb);
    const bool huge = __any((int)(__float_as_uint(mx) >> 23) > tb);
    asm volatile("" ::: "memory");
    const bool bad = __builtin_amdgcn_readfirstlane(tmov) == call;
    WAIT_VM(0);
    const int g = lane >> 5, i = lane & 31;
    unsigned desc = ((grow & 511) >> 5) | (ct << 4);
    const char* dst;
    bool raw = true;
    if (delta == 0) { dst = (const char*)(p.out + (size_t)(grow & 511) * N_ + ct * 128); }
    else {
        dst = myws + OFF_SEND + par * SEND_BYTES + out_slot(delta, desc);
        raw = bad || huge;
        if (!raw) {
            const float up = __uint_as_float((unsigned)(280 - tb) << 23);
            unsigned P[4][2] = {}, bad_bits = 0;
#pragma unroll
            for (int b = 0; b < 4; ++b) {
                unsigned y[4][4], D[12];
#pragma unroll
                for (int c = 0; c < 4; ++c) {
                    const f32x4 z = (f32x4){acc[c][4 * b], acc[c][4 * b + 1], acc[c][4 * b + 2], acc[c][4 * b + 3]} * up;
#pragma unroll
                    for (int e = 0; e < 4; ++e) {
                        int q;
                        const float ze = z[e];
                        asm("v_cvt_i32_f32 %0, %1" : "=v"(q) : "v"(ze));
                        bad_bits |= __float_as_uint(__builtin_amdgcn_fractf(ze));
                        y[e][c] = (unsigned)q << 5;
                    }
                }
#pragma unroll
                for (int e = 0; e < 4; ++e) {
                    D[3 * e]     = __builtin_amdgcn_perm(y[e][3], y[e][0], 0x03020105u);
                    D[3 * e + 1] = __builtin_amdgcn_perm(y[e][3], y[e][1], 0x03020106u);
                    D[3 * e + 2] = __builtin_amdgcn_perm(y[e][3], y[e][2], 0x03020107u);
                    const unsigned L = __builtin_amdgcn_perm(__builtin_amdgcn_perm(y[e][3], y[e][2], 0x04000400u), __builtin_amdgcn_perm(y[e][1], y[e][0], 0x04000400u), 0x05040100u);
                    const int s = (4 * b + e) & 7;
#pragma unroll
                    for (int k = 0; k < 3; ++k) P[k][b >> 1] |= shl(L, s - (7 - k)) & (0x01010101u << s);
                }
#pragma unroll
                for (int q = 0; q < 3; ++q)
                    st16_wt(dst, (4 * b + q) * 1024 + lane * 16, (u32x4){D[4 * q], D[4 * q + 1], D[4 * q + 2], D[4 * q + 3]});
                asm volatile("" : "+v"(bad_bits), "+v"(P[0][b >> 1]), "+v"(P[1][b >> 1]), "+v"(P[2][b >> 1]));
            }
            if (wide) {
                asm volatile("" ::: "memory");
                const float lim = __uint_as_float((unsigned)tb << 23);
#pragma unroll
                for (int e = 0; e < 16; ++e)
#pragma unroll
                    for (int c = 0; c < 4; ++c) {
                        unsigned f = (acc[c][e] >= lim || acc[c][e] < -lim) ? 1u : 0u;
                        asm("" : "+v"(f));
                        P[3][e >> 3] |= f << (8 * c + (e & 7));
                    }
            }
            const unsigned nz[2] = {P[0][0] | P[1][0] | P[2][0] | P[3][0], P[0][1] | P[1][1] | P[2][1] | P[3][1]};
            LAS unsigned* cnt = (LAS unsigned*)(lds + wave * 4096);
            LAS unsigned short* list = (LAS unsigned short*)(lds + STAGE_BYTES + wave * 4096);
            *cnt = 0;
            *(LAS u32x4*)(list + lane * 8) = (u32x4){0, 0, 0, 0};
            unsigned pos = lds_add(cnt, __builtin_popcount(nz[0]) + __builtin_popcount(nz[1]));
            const unsigned nexc = __builtin_amdgcn_readfirstlane(*(volatile LAS unsigned*)cnt);
            if (!__any(bad_bits != 0) && nexc <= (unsigned)EXC_MAX) {
#pragma unroll
                for (int h = 0; h < 2; ++h)
                    for (unsigned mm = nz[h]; mm; mm &= mm - 1) {
                        const unsigned b = __builtin_ctz(mm);
                        const unsigned val = (((P[0][h] >> b) & 1u) << 3) | (((P[1][h] >> b) & 1u) << 2) | (((P[2][h] >> b) & 1u) << 1) | ((P[3][h] >> b) & 1u);
                        list[pos++] = (unsigned short)(((h * 2048 + (b & 4u) * 256 + lane * 16 + (b & 3u) * 4 + (b >> 3)) << 4) | val);
                    }
                asm volatile("" ::: "memory");
                st16_wt(dst, 15 * 1024 + lane * 16, *(LAS u32x4*)(list + lane * 8));
                desc |= ((unsigned)tb << 12) | (nexc << 20) | (wide ? D_WIDE : 0u);
            } else { raw = true; WAIT_VM(0); }
        }
    }
    if (raw && !(part && wave >= SU_S)) {
        const float nan = __builtin_nanf("");
#pragma unroll
        for (int e = 0; e < 16; ++e) {
            f32x4 v = {acc[0][e], acc[1][e], acc[2][e], acc[3][e]};
            if (delta == 0 && !part) v = dq_silu(v, scale);
            if (bad) v = (f32x4){nan, nan, nan, nan};
            st16_wt(dst, delta == 0 ? ((e & 3) + 8 * (e >> 2) + 4 * g) * (unsigned)(N_ * 4) + i * 16 : e * 1024 + lane * 16, __builtin_bit_cast(u32x4, v));
        }
        desc |= D_RAW;
        if (part) {
            WAIT_VM(0);
            if (lane == 0 && __hip_atomic_fetch_add((GAS unsigned*)(myws + OFF_CNT + 15 * 128), 1u, __ATOMIC_RELAXED, __HIP_MEMORY_SCOPE_AGENT) + 1 == call * (96u * SU_S))
                st8_sys(myws + OFF_CNT + 16 * 128, (unsigned long long)call << 32);
        }
    }
    if (delta != 0) {
        const unsigned nout = (delta & 1) ? 24 * SU_O : 32 * SU_S;
        unsigned tk = 0;
        if (lane == 0) tk = __hip_atomic_fetch_add((GAS unsigned*)(myws + OFF_CNT + (8 + delta - 1) * 128), 1u, __ATOMIC_RELAXED, __HIP_MEMORY_SCOPE_AGENT) - (call - 1) * nout;
        WAIT_VM(0);
        if (lane == 0) st8_sys(p.ws[delta] + OFF_RQ + ((par * NR + delta) * RQN + E_OUT + min(tk, nout - 1)) * 8, ((unsigned long long)call << 32) | desc);
    }
    STAMP(4 + so);
}

__global__ void __launch_bounds__(512, 2) fused_gemm_rs(Params p) {
    extern __shared__ __attribute__((aligned(16))) char smem[];
    const int lane = threadIdx.x & 63;
    const int wave = __builtin_amdgcn_readfirstlane(threadIdx.x >> 6);
    const int r = __builtin_amdgcn_readfirstlane(p.rank) & 7;
    char* myws = p.ws[0];
    char* cp = myws + OFF_CALL + (blockIdx.x * 8 + wave) * 4;
    if (blockIdx.x < NENG) { engine(p, r, myws, cp, (LAS char*)smem, wave, lane); return; }
    const unsigned callv = ld_agent(cp);
    STAMP(0);
    const unsigned long long tstart = __builtin_amdgcn_s_memrealtime();
    const int e = blockIdx.x & 7, slot = (blockIdx.x - NENG) >> 3, m = e >> 1;
    for (int task = blockIdx.x - NENG; task < 160; task = (task >= 32 && task < 64) ? task - 32 + NGEMM : 160) {
        if (task >= 32) while (__builtin_amdgcn_s_memrealtime() < tstart + (task < 64 ? PACE_X : PACE_W + (task - 64) * PACE_WS / 16)) __builtin_amdgcn_s_sleep(4);
        pack_operands(p, r, myws, callv, (LAS char*)smem, wave, lane, task, tstart + PACE_W0 + (blockIdx.x - NENG) * PACE_W1 / 16);
    }
    const unsigned call = __builtin_amdgcn_readfirstlane(callv) + 1;
    st_agent(cp, call);
    float scale = p.sx[0] * p.sw[0];
    asm volatile("" : "+v"(scale));
    STAMP(1);
    const int pt = (NGEMM / 8 - 1 - slot) * 8 + e;
    for (int id = pt < 96 ? slot - NGEMM / 8 : slot; id < 32; id += NGEMM / 8) {
        int ct, rb;
        if (id < 0) while (__builtin_amdgcn_s_memrealtime() < tstart + PACE_P0 + (95 - pt) * PACE_P1 / 16) __builtin_amdgcn_s_sleep(8);
        if (id < 0) { const int pm = pt / 24, q = (pt % 24) >> 3, mine = col_block(r, pm) & 3; ct = (pm * 4 + q + (q >= mine)) * 8 + (pt & 7); rb = 6; }
        else if (id < 24) { ct = col_block(r, m) * 8 + id / 3; rb = 3 * (e & 1) + id % 3; }
        else { ct = col_block(r, m) * 8 + id - 24; rb = 6 + (e & 1); }
        gemm_tile(p, r, myws, call, (LAS char*)smem, wave, lane, ct, rb, id < 0, scale, id < 0 ? 10 : id == slot ? 0 : 5);
    }
}

}

extern "C" void kernel_launch(int rank, int nranks, ncclComm_t comm,
                              hipStream_t stream,
                              void* const* d_in, const int* in_sizes, int n_in,
                              void* d_out, int out_size,
                              const int* in_dtypes, int out_dtype,
                              void* const* d_out_all,
                              void* const* d_ws_all, size_t ws_bytes) {
    (void)comm; (void)in_dtypes; (void)out_dtype; (void)d_out_all;
    if (nranks != NR || (unsigned)rank >= (unsigned)NR || n_in < 4 || ws_bytes < WS_TOTAL || out_size != 512 * N_ ||
        in_sizes[0] != 4096 * KS || in_sizes[1] != KS * N_) {
        fprintf(stderr, "kernel_launch: unexpected problem geometry\n");
        return;
    }
    static bool attr_set[NR] = {};
    if (!attr_set[rank]) {
        HIP_CHECK(hipFuncSetAttribute((const void*)fused_gemm_rs, hipFuncAttributeMaxDynamicSharedMemorySize, LDS_BYTES));
        attr_set[rank] = true;
    }
    Params p;
    p.x = (const unsigned char*)d_in[0]; p.w = (const unsigned char*)d_in[1];
    p.sx = (const float*)d_in[2]; p.sw = (const float*)d_in[3];
    for (int i = 0; i < NR; ++i) p.ws[i] = (char*)d_ws_all[rank ^ i];
    p.out = (float*)d_out;
    p.rank = rank; p.pad = 0;
    hipLaunchKernelGGL(fused_gemm_rs, dim3(NWG), dim3(512), LDS_BYTES, stream, p);
    HIP_CHECK(hipGetLastError());
}
```

```cpp
#include <hip/hip_runtime.h>
#include <rccl/rccl.h>
#include <cstdio>
#include <cstdint>

#define HIP_CHECK(expr) do { hipError_t _e = (expr); if (_e != hipSuccess) { \
    fprintf(stderr, "HIP error %s at %s:%d\n", hipGetErrorString(_e), __FILE__, __LINE__); \
    return; } } while (0)

namespace {

typedef unsigned u32x4 __attribute__((ext_vector_type(4)));
typedef int i32x8 __attribute__((ext_vector_type(8)));
typedef float f32x16 __attribute__((ext_vector_type(16)));
typedef float f32x4 __attribute__((ext_vector_type(4)));
#define LAS __attribute__((address_space(3)))
#define GAS __attribute__((address_space(1)))

constexpr int NR = 8;
constexpr int N_ = 16384, KS = 512;
#define NENG_ 88
#define EPW 1
constexpr int NWG = 256, NENG = NENG_, NGEMM = NWG - NENG;
#define EW 12
#define WIN 6
#define PWIN 6
#define GAUX 17
constexpr int SU_S = 7, SU_O = 9;
constexpr int NSU_REMOTE = 3 * SU_S + 4 * SU_O;
constexpr int NSU = NSU_REMOTE + 16;
constexpr int ITEM = 16384;
constexpr int NOUT_MAX = 4 * 8 * SU_S;
#define NPUSH_ 18
constexpr int NPUSH = NPUSH_;
#define PACE_X 0
#define PACE_W 300
#define PACE_WS 80
#define PACE_W0 0
#define PACE_W1 0
#define PACE_P0 1900
#define PACE_P1 220
#define JX0 16
constexpr int E_X = 128, E_OUT = 208, RQN = 512;
constexpr int DENSE = 12288;
constexpr int EXC_MAX = 512;

constexpr size_t OFF_CALL  = 0;
constexpr size_t OFF_TMO   = 16384;
constexpr size_t OFF_CNT   = 16384 + 128;
constexpr size_t OFF_LFLAG = 20480;
constexpr size_t OFF_WFLAG = 24576;
constexpr size_t OFF_XFLAG = 32768;
constexpr size_t OFF_RQ    = 65536;
constexpr size_t OFF_DBG   = 262144;
constexpr size_t SBW_BYTES = (size_t)512 * ITEM;
constexpr size_t SBX_BYTES = (size_t)128 * ITEM;
constexpr size_t OB_BYTES  = (size_t)7 * 32 * SU_O * ITEM;
constexpr size_t SEND_BYTES = SBW_BYTES + SBX_BYTES + OB_BYTES;
constexpr size_t XS_BYTES  = (size_t)NR * NSU * ITEM;
constexpr size_t WS_BYTES  = (size_t)NR * 32 * 65536;
constexpr size_t OFF_SEND  = 1u << 20;
constexpr size_t OFF_XS    = OFF_SEND + 2 * SEND_BYTES;
constexpr size_t OFF_WS    = OFF_XS + 2 * XS_BYTES;
constexpr size_t OFF_WL    = OFF_WS + WS_BYTES;
constexpr size_t WS_TOTAL  = OFF_WL + (size_t)128 * 65536;
static_assert(OFF_CNT + 17 * 128 <= OFF_LFLAG && OFF_LFLAG + 2048 <= OFF_WFLAG && OFF_XFLAG + NSU * 128 <= OFF_RQ && OFF_RQ + 2 * NR * RQN * 8 <= OFF_DBG, "flags overlap");
static_assert(EW <= (NENG / 7) * EPW && NENG % 8 == 0 && EPW <= 8 && NPUSH <= 2 * EW && NPUSH >= EW && NGEMM >= 128 && E_OUT + NOUT_MAX <= RQN, "geometry");
constexpr unsigned D_RAW = 1u << 11, D_WIDE = 1u << 30;
__device__ __forceinline__ size_t out_slot(int delta, unsigned desc) {
    const unsigned ru = desc & 15u, ct = (desc >> 4) & 127u;
    return SBW_BYTES + SBX_BYTES + (size_t)((((delta - 1) * 32 + (ct >> 5) * 8 + (ct & 7u)) * SU_O) + (ru >= SU_S ? ru - SU_S : ru)) * ITEM;
}

constexpr int STAGE_BYTES = 49152, NSTAGE = 3, LDS_BYTES = NSTAGE * STAGE_BYTES;
constexpr unsigned long long DEADLINE = 50000000ull;

struct Params {
    const unsigned char* x; const unsigned char* w; const float* sx; const float* sw;
    char* ws[NR];
    float* out;
    int rank; int pad;
};

__device__ __forceinline__ unsigned ld_sys(const void* p) { return __hip_atomic_load((const GAS unsigned*)p, __ATOMIC_RELAXED, __HIP_MEMORY_SCOPE_SYSTEM); }
__device__ __forceinline__ void st_sys(void* p, unsigned v) { __hip_atomic_store((GAS unsigned*)p, v, __ATOMIC_RELAXED, __HIP_MEMORY_SCOPE_SYSTEM); }
__device__ __forceinline__ unsigned ld_agent(const void* p) { return __hip_atomic_load((const GAS unsigned*)p, __ATOMIC_RELAXED, __HIP_MEMORY_SCOPE_AGENT); }
__device__ __forceinline__ void st_agent(void* p, unsigned v) { __hip_atomic_store((GAS unsigned*)p, v, __ATOMIC_RELAXED, __HIP_MEMORY_SCOPE_AGENT); }
__device__ __forceinline__ unsigned long long ld8_sys(const void* p) { return __hip_atomic_load((const GAS unsigned long long*)p, __ATOMIC_RELAXED, __HIP_MEMORY_SCOPE_SYSTEM); }
__device__ __forceinline__ void st8_sys(void* p, unsigned long long v) { __hip_atomic_store((GAS unsigned long long*)p, v, __ATOMIC_RELAXED, __HIP_MEMORY_SCOPE_SYSTEM); }
__device__ __forceinline__ void st16_wt(const void* base, unsigned off, u32x4 v) {
    asm volatile("s_nop 4\n\tglobal_store_dwordx4 %0, %1, %2 sc0 sc1\n\ts_nop 1" :: "v"(off), "v"(v), "s"(base) : "memory");
}
#define WAIT_VM(n) asm volatile("s_waitcnt vmcnt(%0)" :: "i"(n) : "memory")
__device__ __forceinline__ unsigned bfi(unsigned mask, unsigned a, unsigned b) { return (a & mask) | (b & ~mask); }
__device__ __forceinline__ unsigned shl(unsigned v, int s) { return s >= 0 ? v << s : v >> -s; }

__device__ __forceinline__ bool give_up(char* myws, unsigned call, unsigned long long t0, unsigned& polls) {
    bool dead = __builtin_amdgcn_s_memrealtime() - t0 > DEADLINE;
    if ((++polls & 63u) == 0u) dead |= __builtin_amdgcn_readfirstlane(ld_agent(myws + OFF_TMO)) == call;
    if (dead) st_agent(myws + OFF_TMO, call);
    return dead;
}

__device__ __forceinline__ int col_block(int r, int m) { return m * 4 + ((r & 1) ? ((r >> 1) ^ m) : (r >> 1)); }
__device__ __forceinline__ int unit_of(int r, int su) {
    if (su >= NSU_REMOTE) return r * 16 + su - NSU_REMOTE;
    const int g = su / 19, pos = su % 19;
    if (pos < 2 || (pos >= 10 && pos < 15)) { const int sg = g * 7 + (pos < 2 ? pos : pos - 8); return (r ^ (2 * (sg % 3 + 1))) * 16 + sg / 3; }
    const int jj = pos < 10 ? (pos - 2) >> 1 : pos - 15, idx = g * 3 + (pos < 10 ? (pos & 1) : 2);
    return (r ^ (2 * jj + 1)) * 16 + SU_S + idx;
}
__device__ __forceinline__ int staged_of(int r, int gu) {
    const int rel = (gu >> 4) ^ r, u = gu & 15;
    if (rel == 0) return NSU_REMOTE + u;
    if (((rel & 1) != 0) != (u >= SU_S)) return -1;
    if (u < SU_S) { const int sg = u * 3 + (rel >> 1) - 1, rho = sg % 7; return (sg / 7) * 19 + (rho < 2 ? rho : rho + 8); }
    const int idx = u - SU_S, rho = idx % 3, jj = rel >> 1;
    return (idx / 3) * 19 + (rho < 2 ? 2 + 2 * jj + rho : 15 + jj);
}

__device__ __forceinline__ int kept_su(int delta, int i) {
    if (delta & 1) return i;
    int su = i;
#pragma unroll
    for (int u = 0; u < SU_S; ++u) { const int sg = u * 3 + (delta >> 1) - 1, rho = sg % 7; su += ((sg / 7) * 19 + (rho < 2 ? rho : rho + 8)) <= su; }
    return su;
}

#define STAMP(slot) do {} while (0)

__device__ __forceinline__ f32x4 dq_silu(f32x4 a, float s) {
    constexpr float L2E = 1.44269504088896341f;
    const f32x4 y = a * s, t = y * -L2E;
    f32x4 e;
#pragma unroll
    for (int c = 0; c < 4; ++c) e[c] = __builtin_amdgcn_exp2f(__builtin_amdgcn_fmed3f(t[c], -60.0f * L2E, 60.0f * L2E));
    e += 1.0f;
#pragma unroll
    for (int c = 0; c < 4; ++c) e[c] = __builtin_amdgcn_rcpf(e[c]);
    return y * e;
}
__device__ __forceinline__ unsigned half_scan(unsigned v) {
    v += __builtin_amdgcn_update_dpp(0u, v, 0x111, 0xf, 0xf, true);
    v += __builtin_amdgcn_update_dpp(0u, v, 0x112, 0xf, 0xf, true);
    v += __builtin_amdgcn_update_dpp(0u, v, 0x114, 0xf, 0xf, true);
    v += __builtin_amdgcn_update_dpp(0u, v, 0x118, 0xf, 0xf, true);
    v += __builtin_amdgcn_update_dpp(0u, v, 0x142, 0xa, 0xf, false);
    return v;
}
__device__ __forceinline__ unsigned wave_scan(unsigned v) {
    v = half_scan(v);
    v += __builtin_amdgcn_update_dpp(0u, v, 0x143, 0xc, 0xf, false);
    return v;
}
__device__ __forceinline__ unsigned lds_add(LAS unsigned* p, unsigned v) {
    unsigned old;
    asm volatile("ds_add_rtn_u32 %0, %1, %2\n\ts_waitcnt lgkmcnt(0)" : "=v"(old) : "v"((unsigned)(size_t)p), "v"(v) : "memory");
    return old;
}

constexpr int NSLOT = 6;
constexpr int L_MAP = NSLOT * ITEM, L_CTL = L_MAP + 4 * 1024;
enum { K_NONE, K_RAWIN, K_RAWOUT, K_IN, K_OUT, K_STOP };

__device__ __forceinline__ void engine(const Params& p, int r, char* myws, char* callp, LAS char* lds, int wave, int lane) {
    const int link = blockIdx.x % 7, k = blockIdx.x / 7;
    if (k >= EW) return;
    const int gw = k * 7 + link; (void)gw;
    const int delta = link + 1, d = (r ^ delta) & 7;
    char* dws = p.ws[delta];
    const bool same = (delta & 1) == 0;
    const unsigned lin = lane * 16;
    auto srd = [](const void* base, int bytes) { return __builtin_amdgcn_make_buffer_rsrc((void*)base, 0, bytes, 0x00020000); };
    volatile LAS unsigned* ctl = (volatile LAS unsigned*)(lds + L_CTL);
    if (wave == 1) {
        const int su0 = kept_su(delta, k), su1 = kept_su(delta, k + EW < NPUSH ? k + EW : k);
        const bool two = k + EW < NPUSH;
        u32x4 A[16], B[16];
        unsigned callv;
        const __amdgpu_buffer_rsrc_t sa = srd(p.x + (size_t)unit_of(d, su0) * ITEM, ITEM), sb = srd(p.x + (size_t)unit_of(d, su1) * ITEM, two ? ITEM : 0);
        auto ld = [&](u32x4& v, int c, __amdgpu_buffer_rsrc_t src) {
            asm volatile("s_nop 4\n\tbuffer_load_dwordx4 %0, %1, %2, 0 offen offset:%3" : "=&v"(v) : "v"(lin + (c >> 2) * 4096), "s"(src), "i"((c & 3) * 1024) : "memory");
        };
#define PIN4(a, b) asm volatile("" : "+v"(a[4 * (b)]), "+v"(a[4 * (b) + 1]), "+v"(a[4 * (b) + 2]), "+v"(a[4 * (b) + 3]) :: "memory")
        asm volatile("s_nop 4\n\tglobal_load_dword %0, %1, %2 sc1" : "=&v"(callv) : "v"(0u), "s"(callp) : "memory");
#pragma unroll
        for (int c = 0; c < 16; ++c) ld(A[c], c, sa);
#pragma unroll
        for (int c = 0; c < 16; ++c) ld(B[c], c, sb);
        __builtin_amdgcn_s_barrier();
        WAIT_VM(24);
        asm volatile("" : "+v"(callv) :: "memory");
        PIN4(A, 0); PIN4(A, 1);
        const unsigned call = __builtin_amdgcn_readfirstlane(callv) + 1;
        STAMP(0);
        const char* dst = dws + OFF_XS + (size_t)(call & 1) * XS_BYTES + (size_t)delta * NSU * ITEM;
#pragma unroll
        for (int c = 0; c < 16; ++c) {
            if (c == 8) { PIN4(A, 2); PIN4(A, 3); PIN4(B, 0); PIN4(B, 1); PIN4(B, 2); PIN4(B, 3); }
            st16_wt(dst, lin + c * 1024 + su0 * ITEM, A[c]);
            if (c >= PWIN - 1) WAIT_VM(PWIN - 1);
        }
        if (two) {
#pragma unroll
            for (int c = 0; c < 16; ++c) {
                st16_wt(dst, lin + c * 1024 + su1 * ITEM, B[c]);
                WAIT_VM(PWIN - 1);
                if (c == PWIN - 2 && lane == 0) st_sys(dws + OFF_XFLAG + su0 * 128 + delta * 4, call);
            }
        }
        WAIT_VM(0);
        if (lane == 0) st_sys(dws + OFF_XFLAG + su1 * 128 + delta * 4, call);
        st_agent(callp, call);
        STAMP(1);
        return;
    }
    static_assert(PWIN <= 8 && WIN <= 12, "windows");
    if (wave == 0) { ctl[lane] = 0; __builtin_amdgcn_s_waitcnt(0xC07F); }
    __builtin_amdgcn_s_barrier();
    if (wave >= 6) return;
    const unsigned call = __builtin_amdgcn_readfirstlane(ld_agent(callp)) + 1;
    st_agent(callp, call);
    const size_t par = call & 1;

    if (wave == 0) {
        const int jW1 = JX0 + (same ? NSU_REMOTE - SU_S : NSU_REMOTE) - NPUSH, jXO = jW1 + 128 - JX0, jOUT = jXO + 16;
        const int nitems = jOUT + (same ? 32 * SU_S : 24 * SU_O);
        auto entry_of = [&](int j) {
            if (j >= jOUT) return j < nitems ? E_OUT + j - jOUT : -1;
            if (j >= jXO) return E_X + NSU_REMOTE + j - jXO;
            if (j >= jW1) return j - jW1 + JX0;
            return j < JX0 ? j : E_X + kept_su(delta, j - JX0 + NPUSH);
        };
        const char* claimp = myws + OFF_CNT + link * 128;
        const char* send = dws + OFF_SEND + par * SEND_BYTES;
        const unsigned cbase = (call - 1) * (unsigned)nitems;
        unsigned long long win = 0;
        auto look = [&](int e) {
            const unsigned off = lane == 1 ? (unsigned)(OFF_CNT + 16 * 128) : (unsigned)(OFF_RQ + ((par * NR + delta) * RQN + (e & (RQN - 1))) * 8);
            asm volatile("s_nop 4\n\tglobal_load_dwordx2 %0, %1, %2 sc0 sc1" : "=&v"(win) : "v"(off), "s"(myws) : "memory");
        };
        int e = entry_of(k), en = entry_of(k + EW);
        unsigned ops = WIN;
        bool dead = false;
        int slot = 0;
        WAIT_VM(0);
        for (unsigned i = 0;; ++i, slot = slot == NSLOT - 1 ? 0 : slot + 1) {
            const bool is_out = e >= E_OUT;
            auto there = [&]() { return (unsigned)__builtin_amdgcn_readfirstlane((unsigned)(win >> 32)) == call &&
                                        (!(same && is_out) || (unsigned)__builtin_amdgcn_readlane((unsigned)(win >> 32), 1) == call); };
            if (e < 0 || !there()) {
                WAIT_VM(0);
                ctl[0] = ops; ops += WIN;
                if (e >= 0 && !dead) {
                    const unsigned long long t0 = __builtin_amdgcn_s_memrealtime(); unsigned polls = 0;
                    for (;;) {
                        look(e);
                        asm volatile("s_waitcnt vmcnt(0)" : "+v"(win) :: "memory");
                        if (there()) break;
                        if (give_up(myws, call, t0, polls)) { dead = true; break; }
                        __builtin_amdgcn_s_sleep(2);
                    }
                }
            }
            const unsigned desc = __builtin_amdgcn_readfirstlane((unsigned)win);
            const char* src; const char* dst; unsigned flag = 0;
            if (is_out) {
                src = send + out_slot(delta, desc);
                dst = (const char*)(p.out + (size_t)(desc & 15) * 32 * N_ + ((desc >> 4) & 127) * 128);
            } else if (e < E_X) {
                const int t = (e >> 5) & 3, G = (e >> 4) & 1, kc = (e >> 2) & 3, m = e & 3, tc = (m * 4 + t) * 2 + G;
                src = send + (size_t)(((col_block(r, m) * 4 + t) * 2 + G) * 4 + kc) * ITEM;
                dst = myws + OFF_WS + ((size_t)delta * 32 + tc) * 65536 + kc * ITEM;
                flag = OFF_WFLAG + tc * 128 + (kc * 8 + delta) * 4;
            } else {
                const int su = e - E_X;
                src = send + SBW_BYTES + (size_t)unit_of(r, su) * ITEM;
                dst = myws + OFF_XS + par * XS_BYTES + ((size_t)delta * NSU + su) * ITEM;
                flag = OFF_XFLAG + su * 128 + delta * 4;
            }
            const int kind = e < 0 ? K_STOP : dead ? K_NONE : ((desc & D_RAW) ? K_RAWIN : K_IN) + (is_out ? 1 : 0);
            for (;;) {
                const unsigned m = min(min(ctl[4], ctl[5]), min(ctl[6], ctl[7]));
                if ((unsigned)__builtin_amdgcn_readfirstlane(m) + NSLOT > i) break;
                __builtin_amdgcn_s_sleep(1);
            }
            *(volatile LAS u32x4*)(ctl + 16 + 8 * slot) = (u32x4){(unsigned)(size_t)dst, (unsigned)((size_t)dst >> 32), flag, desc};
            *(volatile LAS u32x4*)(ctl + 20 + 8 * slot) = (u32x4){ops + 2, (unsigned)kind, 0u, i + 1};
            if (e < 0) break;
            unsigned cl; unsigned long long keep;
            asm volatile("s_mov_b64 %1, exec\n\ts_mov_b64 exec, 1\n\ts_nop 2\n\tglobal_atomic_add %0, %2, %3, %4 sc0\n\ts_mov_b64 exec, %1"
                         : "=&v"(cl), "=&s"(keep) : "v"(0u), "v"(1u), "s"(claimp) : "memory");
            look(en);
            ops += 2;
            const unsigned tail = kind >= K_IN ? min((desc >> 20) & 1023u, 256u) * 16 : ITEM - DENSE;
            const __amdgpu_buffer_rsrc_t ssrc = srd(src, kind == K_NONE ? 0 : DENSE + tail);
            const int naux = (tail + 1023) >> 10;
            for (int n = 1; n <= naux + 12; ++n) {
                const int c = n <= naux ? 11 + n : n - 1 - naux;
                asm volatile("s_mov_b32 m0, %0\n\ts_nop 3\n\tbuffer_load_dwordx4 %1, %2, 0 offen sc0 sc1 lds"
                             :: "s"((unsigned)(size_t)lds + slot * ITEM + c * 1024), "v"(lin + c * 1024), "s"(ssrc) : "memory");
                ++ops;
                if (n < WIN) { WAIT_VM(WIN + 1); ctl[0] = ops - (WIN + 1); }
                else { WAIT_VM(WIN - 1); ctl[0] = ops - (WIN - 1); }
            }
            asm volatile("" : "+v"(cl), "+v"(win) :: "memory");
            const unsigned cn = __builtin_amdgcn_readfirstlane(cl) - cbase;
            e = en;
            en = cn < (unsigned)nitems ? entry_of(2 * EW + (int)cn) : -1;
        }
        if (dead && lane == 0) p.out[(size_t)col_block(r, 0) * 1024] = __builtin_nanf("");
        return;
    }

    const int b = wave - 2;
    const float scale = p.sx[0] * p.sw[0];
    LAS unsigned* bm = (LAS unsigned*)(lds + L_MAP + b * 1024);
    const unsigned rowl = ((lane >> 5) * 4 + 8 * b) * (unsigned)(N_ * 4) + (lane & 31) * 16;
    int slot = 0;
    for (unsigned i = 0;; ++i, slot = slot == NSLOT - 1 ? 0 : slot + 1) {
        u32x4 h1;
        for (;;) {
            h1 = *(volatile LAS u32x4*)(ctl + 20 + 8 * slot);
            if ((unsigned)__builtin_amdgcn_readfirstlane(h1[3]) == i + 1) break;
            __builtin_amdgcn_s_sleep(1);
        }
        const int kind = __builtin_amdgcn_readfirstlane(h1[1]);
        if (kind == K_STOP) break;
        const unsigned start = __builtin_amdgcn_readfirstlane(h1[0]);
        const u32x4 h0 = *(volatile LAS u32x4*)(ctl + 16 + 8 * slot);
        const char* dst = (const char*)(((size_t)(unsigned)__builtin_amdgcn_readfirstlane(h0[1]) << 32) | (unsigned)__builtin_amdgcn_readfirstlane(h0[0]));
        const unsigned flag = __builtin_amdgcn_readfirstlane(h0[2]), desc = __builtin_amdgcn_readfirstlane(h0[3]);
        const LAS char* in = lds + slot * ITEM + lin;
        auto landed = [&](unsigned n) {
            while ((unsigned)__builtin_amdgcn_readfirstlane(ctl[0]) < start + n) __builtin_amdgcn_s_sleep(1);
            asm volatile("" ::: "memory");
        };
        auto chunk = [&](int c) { return *(const LAS u32x4*)(in + c * 1024); };
        const unsigned tail16 = min((desc >> 20) & 1023u, 256u), naux = (tail16 + 63) >> 6;
        if (kind == K_OUT || kind == K_RAWOUT) {
            const __amdgpu_buffer_rsrc_t sp = srd(dst, same ? 31 * N_ * 4 + 512 : 0);
            u32x4 P[4];
#pragma unroll
            for (int q = 0; q < 4; ++q) P[q] = __builtin_amdgcn_raw_buffer_load_b128(sp, rowl + q * (unsigned)(N_ * 4), 0, 17);
            f32x4 v[4];
            float unit = 1.0f;
            if (kind == K_OUT) {
                *(LAS u32x4*)(bm + lane * 4) = (u32x4){0, 0, 0, 0};
                landed(naux);
                if ((unsigned)lane < tail16) {
                    const u32x4 aux = chunk(12);
#pragma unroll
                    for (int q = 0; q < 8; ++q) {
                        const unsigned x = (aux[q >> 1] >> (16 * (q & 1))) & 0xffffu;
                        if ((x >> 14) == (unsigned)b)
                            __hip_atomic_fetch_or(bm + ((x >> 6) & 255u), (((x << 4) & 0xe0u) | (x & 1u)) << ((x >> 1) & 24u), __ATOMIC_RELAXED, __HIP_MEMORY_SCOPE_WAVEFRONT);
                    }
                }
                asm volatile("" ::: "memory");
                const u32x4 L = *(LAS u32x4*)(bm + lane * 4);
                unit = __uint_as_float((((desc >> 12) & 255u) - 31u) << 23);
                landed(naux + 3 + 3 * b);
                const u32x4 R[3] = {chunk(3 * b), chunk(3 * b + 1), chunk(3 * b + 2)};
                u32x4 y[4];
#pragma unroll
                for (int q = 0; q < 4; ++q) {
                    const unsigned d0 = R[(3 * q) >> 2][(3 * q) & 3], d1 = R[(3 * q + 1) >> 2][(3 * q + 1) & 3], d2 = R[(3 * q + 2) >> 2][(3 * q + 2) & 3];
                    const unsigned T = L[q] & 0xe0e0e0e0u;
                    y[q] = (u32x4){__builtin_amdgcn_perm(T, d0, 0x03020104u), __builtin_amdgcn_perm(T, d1, 0x03020105u), __builtin_amdgcn_perm(T, d2, 0x03020106u),
                                   __builtin_amdgcn_perm(__builtin_amdgcn_perm(d2, d1, 0x04000400u), __builtin_amdgcn_perm(T, d0, 0x00070007u), 0x05040100u)};
#pragma unroll
                    for (int c = 0; c < 4; ++c) v[q][c] = (float)(int)y[q][c];
                }
                if (desc & D_WIDE) {
#pragma unroll
                    for (int q = 0; q < 4; ++q)
#pragma unroll
                        for (int c = 0; c < 4; ++c)
                            v[q][c] += __uint_as_float((0xcf800000u ^ (y[q][c] & 0x80000000u)) & (unsigned)__builtin_amdgcn_sbfe((int)L[q], 8 * c, 1));
                }
            } else {
                landed(b == 3 ? 4 : 8 + 4 * b);
#pragma unroll
                for (int q = 0; q < 4; ++q) v[q] = __builtin_bit_cast(f32x4, chunk(4 * b + q));
            }
#pragma unroll
            for (int q = 0; q < 4; ++q)
                v[q] = dq_silu(__builtin_elementwise_fma(v[q], (f32x4){unit, unit, unit, unit}, __builtin_bit_cast(f32x4, P[q])), scale);
#pragma unroll
            for (int q = 0; q < 4; ++q) st16_wt(dst, rowl + q * (unsigned)(N_ * 4), __builtin_bit_cast(u32x4, v[q]));
        } else if (kind == K_IN) {
            landed(naux + 3 + 3 * b);
            u32x4 x[4] = {chunk(3 * b), chunk(3 * b + 1), chunk(3 * b + 2), {0, 0, 0, 0}}, m[4];
            unsigned n = 0;
#pragma unroll
            for (int c = 0; c < 4; ++c) {
                x[3][c] = ((x[0][c] >> 6) & 0x03030303u) | ((x[1][c] >> 4) & 0x0c0c0c0cu) | ((x[2][c] >> 2) & 0x30303030u);
#pragma unroll
                for (int q = 0; q < 4; ++q) { m[q][c] = (x[q][c] >> 3) & (x[q][c] >> 4) & 0x01010101u; n += __builtin_popcount(m[q][c]); }
            }
            const LAS unsigned* tl = (const LAS unsigned*)(lds + slot * ITEM + DENSE);
            unsigned pos = min(tl[b], 1023u) * 32 + 3 * (wave_scan(n) - n);
#pragma unroll
            for (int q = 0; q < 4; ++q) {
                u32x4 o;
#pragma unroll
                for (int cp = 0; cp < 4; cp += 2) {
                    const unsigned wi = min(pos >> 5, 1022u);
                    unsigned w = __builtin_amdgcn_alignbit(tl[wi + 1], tl[wi], pos & 31);
#pragma unroll
                    for (int c = cp; c < cp + 2; ++c) {
                        const unsigned mm = m[q][c], n01 = 3 * __builtin_popcount(mm & 0x0101u), n4 = 3 * __builtin_popcount(mm);
                        const unsigned T = __builtin_amdgcn_perm(w >> n01, w, 0x0c040c00u);
                        const unsigned me = mm & 0x00010001u;
                        const unsigned O = bfi((me << 3) - me, T >> 3, T);
                        const unsigned c3 = ((T & 0x00070007u) | ((O & 0x00070007u) << 8)) & ((mm << 3) - mm);
                        o[c] = (((x[q][c] & 0x1f1f1f1fu) + 0x28282828u) ^ __builtin_amdgcn_perm(0x60585048u, 0x40100800u, c3)) | ((x[q][c] << 2) & 0x80808080u);
                        w >>= n4; pos += n4;
                    }
                }
                st16_wt(dst, b * 4096 + ((lane >> 5) * 4 + q) * 512 + (lane & 31) * 16, o);
            }
        } else if (kind == K_RAWIN) {
            landed(b == 3 ? 4 : 8 + 4 * b);
#pragma unroll
            for (int q = 0; q < 4; ++q) st16_wt(dst, lin + (4 * b + q) * 1024, chunk(4 * b + q));
        } else landed(16);
        if (kind == K_IN || kind == K_RAWIN) {
            WAIT_VM(0);
            unsigned n = 0;
            if (lane == 0) n = lds_add((LAS unsigned*)(ctl + 8 + slot), 1u);
            if (lane == 0 && (n & 3u) == 3u) st_sys(myws + flag, call);
        }
        asm volatile("" ::: "memory");
        ctl[4 + b] = i + 1;
    }
    WAIT_VM(0);
    STAMP(4);
}

__device__ __forceinline__ void pack_operands(const Params& p, int r, char* myws, unsigned callv, LAS char* lds, int wave, int lane, int task, unsigned long long later) {
    const int pair = (task < 32 ? task : task < 64 ? task + 96 : task - 32) * 2 + (wave >> 2), pr = wave & 3, G = lane >> 5, i = lane & 31;
    LAS unsigned* tot = (LAS unsigned*)lds;
    LAS unsigned* tails = (LAS unsigned*)(lds + 64);
    for (int q = threadIdx.x; q < 1028; q += 512) ((LAS u32x4*)tails)[q] = (u32x4){0, 0, 0, 0};
    u32x4 o[8];
    const bool is_w = pair < 256;
    const int strip = ((pair >> 2) & 15) * 4 + ((pair >> 6) & 3), kc = pair & 3;
    if (is_w) {
        const __amdgpu_buffer_rsrc_t src = __builtin_amdgcn_make_buffer_rsrc((void*)(p.w + (size_t)(kc * 128 + pr * 32) * N_ + strip * 256), 0, 31 * N_ + 256, 0x00020000);
        unsigned v[32];
#pragma unroll
        for (int j = 0; j < 32; ++j) v[j] = __builtin_amdgcn_raw_buffer_load_b32(src, lane * 4, j * N_, 2);
        __builtin_amdgcn_sched_barrier(0);
#pragma unroll
        for (int q = 0; q < 8; ++q) {
            const unsigned t0 = __builtin_amdgcn_perm(v[4 * q + 1], v[4 * q], 0x05010400u), t1 = __builtin_amdgcn_perm(v[4 * q + 1], v[4 * q], 0x07030602u);
            const unsigned t2 = __builtin_amdgcn_perm(v[4 * q + 3], v[4 * q + 2], 0x05010400u), t3 = __builtin_amdgcn_perm(v[4 * q + 3], v[4 * q + 2], 0x07030602u);
            o[(q & 4) + 0][q & 3] = __builtin_amdgcn_perm(t2, t0, 0x05040100u); o[(q & 4) + 1][q & 3] = __builtin_amdgcn_perm(t2, t0, 0x07060302u);
            o[(q & 4) + 2][q & 3] = __builtin_amdgcn_perm(t3, t1, 0x05040100u); o[(q & 4) + 3][q & 3] = __builtin_amdgcn_perm(t3, t1, 0x07060302u);
        }
    } else {
        const char* src = (const char*)p.x + (size_t)((pair - 256) * 2 + G) * ITEM + pr * 4096 + i * 16;
#pragma unroll
        for (int q = 0; q < 8; ++q) o[q] = *(const u32x4*)(src + q * 512);
    }
    u32x4 pk[6], E[8];
    unsigned bad = 0, n = 0;
#pragma unroll
    for (int c = 0; c < 4; ++c) {
        unsigned cd[8];
#pragma unroll
        for (int q = 0; q < 8; ++q) {
            const unsigned v = o[q][c], idx = (v >> 3) & 0x07070707u, hi = (v >> 6) & 0x01010101u;
            E[q][c] = bfi((hi << 8) - hi, __builtin_amdgcn_perm(0xd8d8d8d8u, 0xd85a5958u, idx), __builtin_amdgcn_perm(0x1008005fu, 0x5e5d5c5bu, idx));
            cd[q] = (v & 0x07070707u) | (E[q][c] & 0x18181818u) | ((v >> 2) & 0x20202020u);
            bad |= E[q][c];
            n += __builtin_popcount(E[q][c] & 0x40404040u) << (q < 4 ? 0 : 16);
        }
#pragma unroll
        for (int q = 0; q < 6; ++q) pk[q][c] = cd[q + q / 3] | ((cd[(q / 3) * 4 + 3] << (6 - 2 * (q % 3))) & 0xc0c0c0c0u);
    }
    const int it = (wave >> 2) * 2 + G;
    const unsigned sc = half_scan(n);
    const unsigned nt = __shfl(sc, G * 32 + 31);
    const unsigned long long anybad = __ballot((bad & 0x80808080u) != 0);
    if (i == 0) tot[it * 4 + pr] = (unsigned)(anybad >> (G * 32)) ? 4096u : (3 * ((nt & 0xffffu) + (nt >> 16)) + 31) >> 5;
    __syncthreads();
    asm volatile("" : "+v"(callv));
    const unsigned call = __builtin_amdgcn_readfirstlane(callv) + 1;
    const size_t par = call & 1;
    const u32x4 tq = *(LAS u32x4*)(tot + it * 4);
    const unsigned words = 4 + tq[0] + tq[1] + tq[2] + tq[3];
    const bool rawv = words > 1000;
    LAS unsigned* tl = tails + it * 1028;
    if (!rawv) {
        const unsigned base = 4 + (pr > 0 ? tq[0] : 0) + (pr > 1 ? tq[1] : 0) + (pr > 2 ? tq[2] : 0);
        if (i == 0) tl[pr] = base;
#pragma unroll
        for (int h = 0; h < 2; ++h) {
            const unsigned pos = base * 32 + 3 * (h ? (nt & 0xffffu) + ((sc - n) >> 16) : (sc - n) & 0xffffu);
            unsigned wi = pos >> 5, fill = pos & 31;
            unsigned long long acc = 0;
#pragma unroll
            for (int q = 4 * h; q < 4 * h + 4; ++q)
#pragma unroll
                for (int cp = 0; cp < 4; cp += 2) {
                    unsigned val = 0, len = 0;
#pragma unroll
                    for (int c = cp; c < cp + 2; ++c) {
                        const unsigned e = E[q][c], me = (e >> 6) & 0x00010001u, O = (e >> 8) & 0x00070007u;
                        const unsigned T = (e & 0x00070007u) | bfi((me << 6) - me, O << 3, O);
                        val |= ((T & 63u) | ((T >> 16) << (3 * __builtin_popcount(e & 0x4040u)))) << len;
                        len += 3 * __builtin_popcount(e & 0x40404040u);
                    }
                    acc |= (unsigned long long)val << fill;
                    fill += len;
                    if (fill >= 32) { __hip_atomic_fetch_or(tl + wi, (unsigned)acc, __ATOMIC_RELAXED, __HIP_MEMORY_SCOPE_WORKGROUP); ++wi; acc >>= 32; fill -= 32; }
                }
            if (acc) __hip_atomic_fetch_or(tl + wi, (unsigned)acc, __ATOMIC_RELAXED, __HIP_MEMORY_SCOPE_WORKGROUP);
        }
    }
    const char* slot = myws + OFF_SEND + par * SEND_BYTES + (is_w ? (size_t)(strip * 8 + kc) * ITEM : SBW_BYTES + (size_t)(pair - 256) * 2 * ITEM);
    const unsigned ib = G * (is_w ? 4u * ITEM : (unsigned)ITEM) + i * 16;
#pragma unroll
    for (int q = 0; q < 6; ++q) st16_wt(slot, ib + (rawv ? pr * 4096 + q * 512 : (pr * 3 + q % 3) * 1024 + (q / 3) * 512), rawv ? o[q] : pk[q]);
    __syncthreads();
    const unsigned tail16 = (words + 3) >> 2;
#pragma unroll
    for (int q = 0; q < 2; ++q)
        if (rawv || (unsigned)((q * 4 + pr) * 32 + i) < tail16)
            st16_wt(slot, ib + (rawv ? pr * 4096 + (6 + q) * 512 : DENSE + (q * 4 + pr) * 512), rawv ? o[6 + q] : *(LAS u32x4*)(tl + ((q * 4 + pr) * 32 + i) * 4));
    WAIT_VM(0);
    __syncthreads();
    if (pr == 0 && i != 0 && i < NR) {
        const int d = r ^ i, m = strip >> 4;
        const int e = is_w ? ((strip >> 2) == col_block(d, m) ? (strip & 3) * 32 + G * 16 + kc * 4 + m : -1) : staged_of(d, (pair - 256) * 2 + G);
        if (e >= 0) st8_sys(p.ws[i] + OFF_RQ + ((par * NR + i) * RQN + (is_w ? e : E_X + e)) * 8, ((unsigned long long)call << 32) | (rawv ? D_RAW : tail16 << 20));
    }
    if (is_w) {
        if (task >= 32) while (__builtin_amdgcn_s_memrealtime() < later) __builtin_amdgcn_s_sleep(8);
        const char* dst = myws + OFF_WL + (size_t)strip * 2 * 65536 + kc * ITEM + pr * 4096;
#pragma unroll
        for (int q = 0; q < 8; ++q) st16_wt(dst, G * 65536 + q * 512 + i * 16, o[q]);
        WAIT_VM(0);
        __syncthreads();
        if (pr == 0 && i == 0) st_sys(myws + OFF_LFLAG + ((strip * 2 + G) * 4 + kc) * 4, call);
    }
}

__device__ __forceinline__ void gemm_tile(const Params& p, int r, char* myws, unsigned call, LAS char* lds, int wave, int lane,
                                          int ct, int rb, bool part, float scale, int so) {
    (void)so;
    const size_t par = call & 1;
    asm volatile("" : "+v"(lane));
    const int m = ct >> 5, tc = m * 8 + (ct & 7);
    auto su_of = [&](int u) {
        if (rb < 3) { const int j = (u - 2) >> 1; return rb * 19 + (u < 2 ? u : 2 + 2 * (j + (j >= m)) + (u & 1)); }
        if (rb < 6) { const int j = u - 5; return (rb - 3) * 19 + (u < 5 ? 10 + u : 15 + j + (j >= m)); }
        return NSU_REMOTE + (rb - 6) * 8 + u;
    };
    const int w4 = wave & 3;
    const int su0 = su_of(w4), su1 = su_of(w4 + 4), gu0 = unit_of(r, su0), gu1 = unit_of(r, su1);
    const int dl0 = (gu0 >> 4) ^ r, dl1 = (gu1 >> 4) ^ r;
    const int sk0 = (dl0 & 1) ? 0 : dl0, sk1 = (dl1 & 1) ? 0 : dl1;
    const int nsteps = part ? 4 : 32;
    f32x16 acc[4];
#define WAIT_LDS() __builtin_amdgcn_s_waitcnt(0xC07F)
#define FENCE() __builtin_amdgcn_sched_barrier(0)
    LAS char* xch = lds + 81920 + w4 * 16384 + lane * 16;
    __builtin_amdgcn_s_barrier();
    if (wave >= 4) {
        const unsigned alane = (lane >> 3) * 512 + (((lane & 7) ^ (lane >> 4)) * 16);
        const char* xs = myws + OFF_XS + par * XS_BYTES;
        const char* bbase = myws + OFF_WS + (size_t)tc * 65536 + w4 * 4096 + lane * 16;
        const char* bown = myws + OFF_WL + (size_t)ct * 65536 + w4 * 4096 + lane * 16;
        const char* flagp = lane >= 32 ? myws + OFF_XFLAG + ((lane & 8) ? su1 : su0) * 128 + (lane & 7) * 4
                          : (lane & 7) ? myws + OFF_WFLAG + tc * 128 + lane * 4 : myws + OFF_LFLAG + (ct * 4 + (lane >> 3)) * 4;
        unsigned ready = 0;
        bool dead = false;
        int skc = 0, sdl = 0;
        auto stage = [&](int buf) {
            const int n = skc * 8 + sdl;
            if (!((ready >> n) & 1) && !dead) {
                const unsigned long long t0 = __builtin_amdgcn_s_memrealtime(); unsigned polls = 0;
                for (;;) {
                    const unsigned long long mk = __ballot(ld_sys(flagp) >= call);
                    const unsigned xm = (((unsigned)(mk >> 32) | 1u | (1u << sk0)) & ((unsigned)(mk >> 40) | 1u | (1u << sk1))) & 0xffu;
                    ready = (unsigned)mk & (xm * 0x01010101u);
                    if ((ready >> n) & 1) break;
                    if (give_up(myws, call, t0, polls)) { dead = true; break; }
                    __builtin_amdgcn_s_sleep(8);
                }
            }
            const char* ga0 = (sdl == 0 ? (const char*)p.x + (size_t)gu0 * ITEM : xs + (size_t)(sdl * NSU + su0) * ITEM) + skc * 128;
            const char* ga1 = (sdl == 0 ? (const char*)p.x + (size_t)gu1 * ITEM : xs + (size_t)(sdl * NSU + su1) * ITEM) + skc * 128;
            const char* gb = (sdl == 0 ? bown : bbase + (size_t)sdl * (32 * 65536)) + skc * ITEM;
            LAS char* l = lds + buf * STAGE_BYTES + w4 * 4096;
#pragma unroll
            for (int q = 0; q < 4; ++q) {
                __builtin_amdgcn_global_load_lds((const GAS unsigned*)(gb + q * 1024), (LAS unsigned*)(l + 32768 + q * 1024), 16, 0, GAUX);
                __builtin_amdgcn_global_load_lds((const GAS unsigned*)(ga0 + q * 4096 + (alane ^ ((q & 1) * 64))), (LAS unsigned*)(l + q * 1024), 16, 0, GAUX);
                __builtin_amdgcn_global_load_lds((const GAS unsigned*)(ga1 + q * 4096 + (alane ^ ((q & 1) * 64))), (LAS unsigned*)(l + 16384 + q * 1024), 16, 0, GAUX);
            }
            if (sdl == 0) { if (skc < 3) ++skc; else if (!part) { skc = 0; sdl = 1; } }
            else if (sdl < 7) ++sdl;
            else if (skc < 3) { ++skc; sdl = 1; }
        };
        stage(0); stage(1); stage(2);
        WAIT_VM(24);
        __builtin_amdgcn_s_barrier();
        int buf = 0;
        for (int n = 0; n < nsteps; ++n) {
            WAIT_VM(12);
            __builtin_amdgcn_s_barrier();
            asm volatile("" ::: "memory");
            stage(buf);
            buf = buf == 2 ? 0 : buf + 1;
        }
        WAIT_VM(0);
        __builtin_amdgcn_s_barrier();
        __builtin_amdgcn_s_barrier();
        asm volatile("" ::: "memory");
#pragma unroll
        for (int e = 0; e < 16; ++e) {
            const f32x4 v = *(const LAS f32x4*)(xch + e * 1024);
#pragma unroll
            for (int c = 0; c < 4; ++c) acc[e >> 2][4 * (e & 3) + c] = v[c];
        }
    } else {
        f32x16 acd[4];
#pragma unroll
        for (int c = 0; c < 4; ++c)
#pragma unroll
            for (int e = 0; e < 16; ++e) { acc[c][e] = 0.f; acd[c][e] = 0.f; }
        const int aoff = w4 * 4096 + (lane & 31) * 128 + (((lane >> 5) ^ ((lane >> 1) & 7)) * 16);
        const int boff = 32768 + (lane >> 5) * 2048 + (lane & 31) * 16;
        auto frag = [&](const LAS char* p0, const LAS char* p1) -> i32x8 {
            const u32x4 lo = *(const LAS u32x4*)p0, hi = *(const LAS u32x4*)p1;
            return (i32x8){(int)lo[0], (int)lo[1], (int)lo[2], (int)lo[3], (int)hi[0], (int)hi[1], (int)hi[2], (int)hi[3]};
        };
        i32x8 a0[2], b0[4], a1[2], b1[4];
        auto read_half = [&](int buf, int kk, i32x8 (&af)[2], i32x8 (&bf)[4]) {
            const LAS char* s = lds + buf * STAGE_BYTES;
#pragma unroll
            for (int u = 0; u < 2; ++u) af[u] = frag(s + u * 16384 + (aoff ^ (kk * 64)), s + u * 16384 + (aoff ^ (kk * 64 + 32)));
#pragma unroll
            for (int c = 0; c < 4; ++c) bf[c] = frag(s + boff + kk * 8192 + c * 512, s + boff + kk * 8192 + 4096 + c * 512);
        };
#define MFMA(D, c, a, b) D[c] = __builtin_amdgcn_mfma_scale_f32_32x32x64_f8f6f4(a, b[c], D[c], 0, 0, 0, 0, 0, 0)
        const unsigned skn0 = sk0 ? 0x204081u << (3 + sk0) : 0u, skn1 = sk1 ? 0x204081u << (3 + sk1) : 0u;
        auto blank = [&](int n, i32x8 (&af)[2]) {
            if ((skn0 >> n) & 1) { asm volatile("" ::: "memory"); af[0] = (i32x8){0, 0, 0, 0, 0, 0, 0, 0}; }
            if ((skn1 >> n) & 1) { asm volatile("" ::: "memory"); af[1] = (i32x8){0, 0, 0, 0, 0, 0, 0, 0}; }
        };
        __builtin_amdgcn_s_barrier();
        asm volatile("" ::: "memory");
        read_half(0, 0, a0, b0);
        int buf = 0;
        for (int n = 0; n < nsteps; ++n) {
            WAIT_LDS();
            blank(n, a0);
            MFMA(acc, 0, a0[0], b0);
            asm volatile("" : "+v"(acc[0])); FENCE();
            read_half(buf, 1, a1, b1); FENCE();
            MFMA(acd, 0, a0[1], b0);
#pragma unroll
            for (int c = 1; c < 4; ++c) { MFMA(acc, c, a0[0], b0); MFMA(acd, c, a0[1], b0); }
            FENCE();
            WAIT_LDS();
            blank(n, a1);
            MFMA(acc, 0, a1[0], b1);
            asm volatile("" : "+v"(acc[0])); FENCE();
            __builtin_amdgcn_s_barrier();
            asm volatile("" ::: "memory");
            buf = buf == 2 ? 0 : buf + 1;
            read_half(buf, 0, a0, b0); FENCE();
            MFMA(acd, 0, a1[1], b1);
#pragma unroll
            for (int c = 1; c < 4; ++c) { MFMA(acc, c, a1[0], b1); MFMA(acd, c, a1[1], b1); }
            FENCE();
        }
        WAIT_LDS();
        __builtin_amdgcn_s_barrier();
        asm volatile("" ::: "memory");
#pragma unroll
        for (int e = 0; e < 16; ++e) *(LAS f32x4*)(xch + e * 1024) = (f32x4){acd[e >> 2][4 * (e & 3)], acd[e >> 2][4 * (e & 3) + 1], acd[e >> 2][4 * (e & 3) + 2], acd[e >> 2][4 * (e & 3) + 3]};
        WAIT_LDS();
        __builtin_amdgcn_s_barrier();
    }
    asm volatile("" : "+v"(lane));
    const int grow = (wave >= 4 ? gu1 : gu0) * 32, delta = wave >= 4 ? dl1 : dl0;
    STAMP(3 + so);

    const unsigned tmov = ld_agent(myws + OFF_TMO);
    asm volatile("s_nop 15\n\ts_nop 15" : "+v"(acc[0]), "+v"(acc[1]), "+v"(acc[2]), "+v"(acc[3]));
    float sum = 0.f, mx = 0.f;
#pragma unroll
    for (int e = 0; e < 16; ++e) {
        sum += __builtin_fabsf(acc[e & 3][e]);
        asm("v_max3_f32 %0, |%1|, |%2|, %0" : "+v"(mx) : "v"(acc[0][e]), "v"(acc[1][e]));
        asm("v_max3_f32 %0, |%1|, |%2|, %0" : "+v"(mx) : "v"(acc[2][e]), "v"(acc[3][e]));
    }
    sum += __uint_as_float(__builtin_amdgcn_update_dpp(0u, __float_as_uint(sum), 0x111, 0xf, 0xf, true));
    sum += __uint_as_float(__builtin_amdgcn_update_dpp(0u, __float_as_uint(sum), 0x112, 0xf, 0xf, true));
    sum += __uint_as_float(__builtin_amdgcn_update_dpp(0u, __float_as_uint(sum), 0x114, 0xf, 0xf, true));
    sum += __uint_as_float(__builtin_amdgcn_update_dpp(0u, __float_as_uint(sum), 0x118, 0xf, 0xf, true));
    const int tb = min(max((int)((unsigned)__builtin_amdgcn_readlane(__float_as_uint(sum * (3.13f / 256.f)), 15) >> 23) + 1, 32), 254);
    const bool wide = __any((int)(__float_as_uint(mx) >> 23) >= tb);
    const bool huge = __any((int)(__float_as_uint(mx) >> 23) > tb);
    asm volatile("" ::: "memory");
    const bool bad = __builtin_amdgcn_readfirstlane(tmov) == call;
    WAIT_VM(0);
    const int g = lane >> 5, i = lane & 31;
    unsigned desc = ((grow & 511) >> 5) | (ct << 4);
    const char* dst;
    bool raw = true;
    if (delta == 0) { dst = (const char*)(p.out + (size_t)(grow & 511) * N_ + ct * 128); }
    else {
        dst = myws + OFF_SEND + par * SEND_BYTES + out_slot(delta, desc);
        raw = bad || huge;
        if (!raw) {
            const float up = __uint_as_float((unsigned)(280 - tb) << 23);
            unsigned P[4][2] = {}, bad_bits = 0;
#pragma unroll
            for (int b = 0; b < 4; ++b) {
                unsigned y[4][4], D[12];
#pragma unroll
                for (int c = 0; c < 4; ++c) {
                    const f32x4 z = (f32x4){acc[c][4 * b], acc[c][4 * b + 1], acc[c][4 * b + 2], acc[c][4 * b + 3]} * up;
#pragma unroll
                    for (int e = 0; e < 4; ++e) {
                        int q;
                        const float ze = z[e];
                        asm("v_cvt_i32_f32 %0, %1" : "=v"(q) : "v"(ze));
                        bad_bits |= __float_as_uint(__builtin_amdgcn_fractf(ze));
                        y[e][c] = (unsigned)q << 5;
                    }
                }
#pragma unroll
                for (int e = 0; e < 4; ++e) {
                    D[3 * e]     = __builtin_amdgcn_perm(y[e][3], y[e][0], 0x03020105u);
                    D[3 * e + 1] = __builtin_amdgcn_perm(y[e][3], y[e][1], 0x03020106u);
                    D[3 * e + 2] = __builtin_amdgcn_perm(y[e][3], y[e][2], 0x03020107u);
                    const unsigned L = __builtin_amdgcn_perm(__builtin_amdgcn_perm(y[e][3], y[e][2], 0x04000400u), __builtin_amdgcn_perm(y[e][1], y[e][0], 0x04000400u), 0x05040100u);
                    const int s = (4 * b + e) & 7;
#pragma unroll
                    for (int k = 0; k < 3; ++k) P[k][b >> 1] |= shl(L, s - (7 - k)) & (0x01010101u << s);
                }
#pragma unroll
                for (int q = 0; q < 3; ++q)
                    st16_wt(dst, (3 * b + q) * 1024 + lane * 16, (u32x4){D[4 * q], D[4 * q + 1], D[4 * q + 2], D[4 * q + 3]});
                asm volatile("" : "+v"(bad_bits), "+v"(P[0][b >> 1]), "+v"(P[1][b >> 1]), "+v"(P[2][b >> 1]));
            }
            if (wide) {
                asm volatile("" ::: "memory");
                const float lim = __uint_as_float((unsigned)tb << 23);
#pragma unroll
                for (int e = 0; e < 16; ++e)
#pragma unroll
                    for (int c = 0; c < 4; ++c) {
                        unsigned f = (acc[c][e] >= lim || acc[c][e] < -lim) ? 1u : 0u;
                        asm("" : "+v"(f));
                        P[3][e >> 3] |= f << (8 * c + (e & 7));
                    }
            }
            const unsigned nz[2] = {P[0][0] | P[1][0] | P[2][0] | P[3][0], P[0][1] | P[1][1] | P[2][1] | P[3][1]};
            LAS unsigned* cnt = (LAS unsigned*)(lds + wave * 4096);
            LAS unsigned short* list = (LAS unsigned short*)(lds + STAGE_BYTES + wave * 4096);
            *cnt = 0;
            *(LAS u32x4*)(list + lane * 8) = (u32x4){0, 0, 0, 0};
            unsigned pos = lds_add(cnt, __builtin_popcount(nz[0]) + __builtin_popcount(nz[1]));
            const unsigned nexc = __builtin_amdgcn_readfirstlane(*(volatile LAS unsigned*)cnt);
            if (!__any(bad_bits != 0) && nexc <= (unsigned)EXC_MAX) {
#pragma unroll
                for (int h = 0; h < 2; ++h)
                    for (unsigned mm = nz[h]; mm; mm &= mm - 1) {
                        const unsigned b = __builtin_ctz(mm);
                        const unsigned val = (((P[0][h] >> b) & 1u) << 3) | (((P[1][h] >> b) & 1u) << 2) | (((P[2][h] >> b) & 1u) << 1) | ((P[3][h] >> b) & 1u);
                        list[pos++] = (unsigned short)(((h * 2048 + (b & 4u) * 256 + lane * 16 + (b & 3u) * 4 + (b >> 3)) << 4) | val);
                    }
                asm volatile("" ::: "memory");
                st16_wt(dst, DENSE + lane * 16, *(LAS u32x4*)(list + lane * 8));
                desc |= ((unsigned)tb << 12) | (((nexc + 7) >> 3) << 20) | (wide ? D_WIDE : 0u);
            } else { raw = true; WAIT_VM(0); }
        }
    }
    if (raw && !(part && wave >= SU_S)) {
        const float nan = __builtin_nanf("");
#pragma unroll
        for (int e = 0; e < 16; ++e) {
            f32x4 v = {acc[0][e], acc[1][e], acc[2][e], acc[3][e]};
            if (delta == 0 && !part) v = dq_silu(v, scale);
            if (bad) v = (f32x4){nan, nan, nan, nan};
            st16_wt(dst, delta == 0 ? ((e & 3) + 8 * (e >> 2) + 4 * g) * (unsigned)(N_ * 4) + i * 16 : e * 1024 + lane * 16, __builtin_bit_cast(u32x4, v));
        }
        desc |= D_RAW;
        if (part) {
            WAIT_VM(0);
            if (lane == 0 && __hip_atomic_fetch_add((GAS unsigned*)(myws + OFF_CNT + 15 * 128), 1u, __ATOMIC_RELAXED, __HIP_MEMORY_SCOPE_AGENT) + 1 == call * (96u * SU_S))
                st8_sys(myws + OFF_CNT + 16 * 128, (unsigned long long)call << 32);
        }
    }
    if (delta != 0) {
        const unsigned nout = (delta & 1) ? 24 * SU_O : 32 * SU_S;
        unsigned tk = 0;
        if (lane == 0) tk = __hip_atomic_fetch_add((GAS unsigned*)(myws + OFF_CNT + (8 + delta - 1) * 128), 1u, __ATOMIC_RELAXED, __HIP_MEMORY_SCOPE_AGENT) - (call - 1) * nout;
        WAIT_VM(0);
        if (lane == 0) st8_sys(p.ws[delta] + OFF_RQ + ((par * NR + delta) * RQN + E_OUT + min(tk, nout - 1)) * 8, ((unsigned long long)call << 32) | desc);
    }
    STAMP(4 + so);
}

__global__ void __launch_bounds__(512, 2) fused_gemm_rs(Params p) {
    extern __shared__ __attribute__((aligned(16))) char smem[];
    const int lane = threadIdx.x & 63;
    const int wave = __builtin_amdgcn_readfirstlane(threadIdx.x >> 6);
    const int r = __builtin_amdgcn_readfirstlane(p.rank) & 7;
    char* myws = p.ws[0];
    char* cp = myws + OFF_CALL + (blockIdx.x * 8 + wave) * 4;
    if (blockIdx.x < NENG) { engine(p, r, myws, cp, (LAS char*)smem, wave, lane); return; }
    const unsigned callv = ld_agent(cp);
    STAMP(0);
    const unsigned long long tstart = __builtin_amdgcn_s_memrealtime();
    const int e = blockIdx.x & 7, slot = (blockIdx.x - NENG) >> 3, m = e >> 1;
    for (int task = blockIdx.x - NENG; task < 160; task = (task >= 32 && task < 64) ? task - 32 + NGEMM : 160) {
        if (task >= 32) while (__builtin_amdgcn_s_memrealtime() < tstart + (task < 64 ? PACE_X : PACE_W + (task - 64) * PACE_WS / 16)) __builtin_amdgcn_s_sleep(4);
        pack_operands(p, r, myws, callv, (LAS char*)smem, wave, lane, task, tstart + PACE_W0 + (blockIdx.x - NENG) * PACE_W1 / 16);
    }
    const unsigned call = __builtin_amdgcn_readfirstlane(callv) + 1;
    st_agent(cp, call);
    float scale = p.sx[0] * p.sw[0];
    asm volatile("" : "+v"(scale));
    STAMP(1);
    const int pt = (NGEMM / 8 - 1 - slot) * 8 + e;
    for (int id = pt < 96 ? slot - NGEMM / 8 : slot; id < 32; id += NGEMM / 8) {
        int ct, rb;
        if (id < 0) while (__builtin_amdgcn_s_memrealtime() < tstart + PACE_P0 + (95 - pt) * PACE_P1 / 16) __builtin_amdgcn_s_sleep(8);
        if (id < 0) { const int pm = pt / 24, q = (pt % 24) >> 3, mine = col_block(r, pm) & 3; ct = (pm * 4 + q + (q >= mine)) * 8 + (pt & 7); rb = 6; }
        else if (id < 24) { ct = col_block(r, m) * 8 + id / 3; rb = 3 * (e & 1) + id % 3; }
        else { ct = col_block(r, m) * 8 + id - 24; rb = 6 + (e & 1); }
        gemm_tile(p, r, myws, call, (LAS char*)smem, wave, lane, ct, rb, id < 0, scale, id < 0 ? 10 : id == slot ? 0 : 5);
    }
}

}

extern "C" void kernel_launch(int rank, int nranks, ncclComm_t comm,
                              hipStream_t stream,
                              void* const* d_in, const int* in_sizes, int n_in,
                              void* d_out, int out_size,
                              const int* in_dtypes, int out_dtype,
                              void* const* d_out_all,
                              void* const* d_ws_all, size_t ws_bytes) {
    (void)comm; (void)in_dtypes; (void)out_dtype; (void)d_out_all;
    if (nranks != NR || (unsigned)rank >= (unsigned)NR || n_in < 4 || ws_bytes < WS_TOTAL || out_size != 512 * N_ ||
        in_sizes[0] != 4096 * KS || in_sizes[1] != KS * N_) {
        fprintf(stderr, "kernel_launch: unexpected problem geometry\n");
        return;
    }
    static bool attr_set[NR] = {};
    if (!attr_set[rank]) {
        HIP_CHECK(hipFuncSetAttribute((const void*)fused_gemm_rs, hipFuncAttributeMaxDynamicSharedMemorySize, LDS_BYTES));
        attr_set[rank] = true;
    }
    Params p;
    p.x = (const unsigned char*)d_in[0]; p.w = (const unsigned char*)d_in[1];
    p.sx = (const float*)d_in[2]; p.sw = (const float*)d_in[3];
    for (int i = 0; i < NR; ++i) p.ws[i] = (char*)d_ws_all[rank ^ i];
    p.out = (float*)d_out;
    p.rank = rank; p.pad = 0;
    hipLaunchKernelGGL(fused_gemm_rs, dim3(NWG), dim3(512), LDS_BYTES, stream, p);
    HIP_CHECK(hipGetLastError());
}
```
